```python
import numpy as np

kernel_name = "gpu_gemm_a2a_m4096_k4096_n4096_w8a8_dequant"


ABSMAX_RTOL = 0.001

COMPUTE_DTYPE = "int8"
EPILOGUE = "dequant"
IN_QUANT = "int8"
OUT_QUANT = None
IN_DTYPES = {"x": "int8", "w_mat": "int8",
             "scale_x": "float32", "scale_w": "float32"}
WORLD_SIZE = 8

def setup_inputs():

    x = np.random.randint(-127, 128, size=(4096, 4096)).astype(np.int8)
    w_mat = np.random.randint(-127, 128, size=(4096, 4096)).astype(np.int8)
    scale_x = (0.02 * (1.0 + np.random.rand(1))).astype(np.float32)
    scale_w = (0.02 * (1.0 + np.random.rand(1))).astype(np.float32)
    return {"x": x, "w_mat": w_mat, "scale_x": scale_x, "scale_w": scale_w}

def reference(x, w_mat, scale_x, scale_w):
    acc = x.astype(np.float64) @ w_mat.astype(np.float64)
    y = (acc.astype(np.float32) * (scale_x * scale_w)).astype(np.float32)
    return y
```

```cpp
#include <hip/hip_runtime.h>
#include <rccl/rccl.h>
#include <cstdio>

#define HIP_CHECK(expr) do { hipError_t _e = (expr); if (_e != hipSuccess) { \
    fprintf(stderr, "HIP error %s at %s:%d\n", hipGetErrorString(_e), __FILE__, __LINE__); \
    return; } } while (0)

typedef int v2i __attribute__((ext_vector_type(2)));
typedef int v4i __attribute__((ext_vector_type(4)));
typedef int v16i __attribute__((ext_vector_type(16)));
typedef unsigned u32x4 __attribute__((ext_vector_type(4)));
typedef float f32x4 __attribute__((ext_vector_type(4)));
#define LDS_AS __attribute__((address_space(3)))

constexpr int NR = 8;
constexpr int MS = 512;
constexpr int KD = 4096;
constexpr int ND = 4096;
constexpr int NS = 512;
constexpr int BM = 128, BN = 64, BK = 128;
constexpr int NSTAGE = 4;
constexpr int A_BYTES = BM * BK;
constexpr int B_BYTES = BK * BN;
constexpr int STAGE_BYTES = A_BYTES + B_BYTES;
constexpr int LDS_BYTES = NSTAGE * STAGE_BYTES;
constexpr int GRID = (MS / BM) * (ND / BN);
constexpr unsigned long long DEADLINE_TICKS = 40000000ull;

constexpr size_t WS_CNT = 0;
constexpr size_t WS_TMO = WS_CNT + GRID * 128;
constexpr size_t WS_FLAG = WS_TMO + 128;
constexpr size_t WS_END = WS_FLAG + NR * 32 * 128;

struct Params {
    const signed char* x;
    const signed char* w;
    const float* sx;
    const float* sw;
    float* out[NR];
    char* ws[NR];
    int rank;
    int pad;
};

__global__ void __launch_bounds__(256, 1) fused_kernel(Params p) {
    extern __shared__ __attribute__((aligned(16))) char smem[];
    const int tid = threadIdx.x, lane = tid & 63, wid = __builtin_amdgcn_readfirstlane(tid >> 6);
    const int b = blockIdx.x;
    const int grp = b & 7, idx = b >> 3;
    const int dst = (p.rank + 1 + grp) & 7;
    const int mt = idx & 3, nt = idx >> 2;
    const int m0 = mt * BM, n0 = dst * NS + nt * BN;
    char* myws = p.ws[p.rank];

    unsigned call = 0;
    if (tid == 0)
        call = __hip_atomic_fetch_add((unsigned*)(myws + WS_CNT + b * 128), 1u, __ATOMIC_RELAXED,
                                      __HIP_MEMORY_SCOPE_AGENT) + 1u;

    const signed char* asrc[4];
    const signed char* bsrc[2];
#pragma unroll
    for (int i = 0; i < 4; ++i) {
        int row = (wid * 4 + i) * 8 + (lane >> 3), ch = (lane & 7) ^ ((row >> 1) & 7);
        asrc[i] = p.x + (size_t)(m0 + row) * KD + ch * 16;
    }
#pragma unroll
    for (int i = 0; i < 2; ++i) {
        int row = (wid * 2 + i) * 16 + (lane >> 2), ch = (lane & 3) ^ (((row >> 2) & 1) << 1);
        bsrc[i] = p.w + (size_t)row * ND + n0 + ch * 16;
    }
    auto stage = [&](int t) {
        char* base = smem + (t % NSTAGE) * STAGE_BYTES;
#pragma unroll
        for (int i = 0; i < 4; ++i)
            __builtin_amdgcn_global_load_lds((const __attribute__((address_space(1))) void*)(asrc[i] + t * BK),
                                             (LDS_AS void*)(base + (wid * 4 + i) * 1024), 16, 0, 0);
#pragma unroll
        for (int i = 0; i < 2; ++i)
            __builtin_amdgcn_global_load_lds((const __attribute__((address_space(1))) void*)(bsrc[i] + (size_t)t * BK * ND),
                                             (LDS_AS void*)(base + A_BYTES + (wid * 2 + i) * 1024), 16, 0, 0);
    };

    v16i acc[2] = {};
    const int arow = wid * 32 + (lane & 31);
    const int aswz = (arow >> 1) & 7;
    const int g = lane >> 4, q = (lane & 15) >> 1, ph = lane & 1;
    constexpr int NT = KD / BK;
    stage(0); stage(1); stage(2);
    for (int t = 0; t < NT; ++t) {
        if (t + 2 < NT) asm volatile("s_waitcnt vmcnt(12)" ::: "memory");
        else if (t + 1 < NT) asm volatile("s_waitcnt vmcnt(6)" ::: "memory");
        else asm volatile("s_waitcnt vmcnt(0)" ::: "memory");
        __builtin_amdgcn_s_barrier();
        if (t + 3 < NT) stage(t + 3);
        const char* base = smem + (t % NSTAGE) * STAGE_BYTES;
#pragma unroll
        for (int kk = 0; kk < 4; ++kk) {
            v4i xa = *(const v4i*)(base + arow * 128 + (((kk * 2 + (lane >> 5)) ^ aswz) << 4));
#pragma unroll
            for (int nb = 0; nb < 2; ++nb) {
                int r0 = kk * 32 + 16 * (g >> 1) + q, r1 = r0 + 8;
                int ch = nb * 2 + (g & 1);
                const char* bb = base + A_BYTES;
                v2i lo = __builtin_amdgcn_ds_read_tr8_b64_v2i32((LDS_AS v2i*)(bb + r0 * 64 + ((ch ^ (((r0 >> 2) & 1) << 1)) << 4) + ph * 8));
                v2i hi = __builtin_amdgcn_ds_read_tr8_b64_v2i32((LDS_AS v2i*)(bb + r1 * 64 + ((ch ^ (((r1 >> 2) & 1) << 1)) << 4) + ph * 8));
                v4i wb = {lo.x, lo.y, hi.x, hi.y};
                acc[nb] = __builtin_amdgcn_mfma_i32_32x32x32_i8(wb, xa, acc[nb], 0, 0, 0);
            }
        }
        asm volatile("s_waitcnt lgkmcnt(0)" ::: "memory");
    }
    __syncthreads();
    const float scale = p.sx[0] * p.sw[0];
    {
        char* crow = smem + arow * 272;
#pragma unroll
        for (int nb = 0; nb < 2; ++nb)
#pragma unroll
            for (int j = 0; j < 4; ++j) {
                f32x4 v = {(float)acc[nb][4 * j] * scale, (float)acc[nb][4 * j + 1] * scale,
                           (float)acc[nb][4 * j + 2] * scale, (float)acc[nb][4 * j + 3] * scale};
                *(f32x4*)(crow + (nb * 32 + 8 * j + 4 * (lane >> 5)) * 4) = v;
            }
    }
    __syncthreads();
    float* obase = p.out[dst] + (size_t)(p.rank * MS + m0) * NS + nt * BN;
#pragma unroll
    for (int i = 0; i < 8; ++i) {
        int row = wid * 32 + i * 4 + (lane >> 4), c = lane & 15;
        u32x4 v = *(const u32x4*)(smem + row * 272 + c * 16);
        float* ptr = obase + (size_t)row * NS + c * 4;
        asm volatile("global_store_dwordx4 %0, %1, off sc0 sc1\n\ts_nop 1" :: "v"(ptr), "v"(v) : "memory");
    }
    asm volatile("s_waitcnt vmcnt(0)" ::: "memory");
    __syncthreads();
    if (tid == 0) {
        if (dst != p.rank)
            __hip_atomic_store((unsigned*)(p.ws[dst] + WS_FLAG + (p.rank * 32 + idx) * 128), call,
                               __ATOMIC_RELAXED, __HIP_MEMORY_SCOPE_SYSTEM);
        int src = b >> 5;
        if (src != p.rank) {
            unsigned* flag = (unsigned*)(myws + WS_FLAG + b * 128);
            unsigned* tmo = (unsigned*)(myws + WS_TMO);
            unsigned long long t0 = __builtin_amdgcn_s_memrealtime();
            while (__hip_atomic_load(flag, __ATOMIC_RELAXED, __HIP_MEMORY_SCOPE_SYSTEM) < call) {
                if (__hip_atomic_load(tmo, __ATOMIC_RELAXED, __HIP_MEMORY_SCOPE_AGENT) == call
                    || __builtin_amdgcn_s_memrealtime() - t0 > DEADLINE_TICKS) {
                    __hip_atomic_store(tmo, call, __ATOMIC_RELAXED, __HIP_MEMORY_SCOPE_AGENT);
                    break;
                }
                __builtin_amdgcn_s_sleep(2);
            }
            if (__hip_atomic_load(tmo, __ATOMIC_RELAXED, __HIP_MEMORY_SCOPE_AGENT) == call)
                __hip_atomic_store((unsigned*)(p.out[p.rank] + (size_t)p.rank * MS * NS), 0x7fc00000u,
                                   __ATOMIC_RELAXED, __HIP_MEMORY_SCOPE_SYSTEM);
        }
    }
}

extern "C" void kernel_launch(int rank, int nranks, ncclComm_t comm,
                              hipStream_t stream,
                              void* const* d_in, const int* in_sizes, int n_in,
                              void* d_out, int out_size,
                              const int* in_dtypes, int out_dtype,
                              void* const* d_out_all,
                              void* const* d_ws_all, size_t ws_bytes) {
    if (nranks != NR || (unsigned)rank >= (unsigned)NR || ws_bytes < WS_END) return;
    static bool init = false;
    if (!init) {
        HIP_CHECK(hipFuncSetAttribute((const void*)fused_kernel, hipFuncAttributeMaxDynamicSharedMemorySize, LDS_BYTES));
        init = true;
    }
    Params p;
    p.x = (const signed char*)d_in[0];
    p.w = (const signed char*)d_in[1];
    p.sx = (const float*)d_in[2];
    p.sw = (const float*)d_in[3];
    for (int i = 0; i < NR; ++i) { p.out[i] = (float*)d_out_all[i]; p.ws[i] = (char*)d_ws_all[i]; }
    p.rank = rank;
    p.pad = 0;
    hipLaunchKernelGGL(fused_kernel, dim3(GRID), dim3(256), LDS_BYTES, stream, p);
    HIP_CHECK(hipGetLastError());
}
```

```cpp
#include <hip/hip_runtime.h>
#include <rccl/rccl.h>
#include <cstdio>

#define HIP_CHECK(expr) do { hipError_t _e = (expr); if (_e != hipSuccess) { \
    fprintf(stderr, "HIP error %s at %s:%d\n", hipGetErrorString(_e), __FILE__, __LINE__); \
    return; } } while (0)

typedef int v2i __attribute__((ext_vector_type(2)));
typedef int v4i __attribute__((ext_vector_type(4)));
typedef int v16i __attribute__((ext_vector_type(16)));
typedef unsigned u32x4 __attribute__((ext_vector_type(4)));
typedef float f32x4 __attribute__((ext_vector_type(4)));
typedef unsigned long long u64;
#define LDS_AS __attribute__((address_space(3)))
#define GLB_AS __attribute__((address_space(1)))
#define CST_AS __attribute__((address_space(4)))

#define AGU 3
#define R_CNT   0x33210220
#define R_FIRST 0x63100b90
#define R_PUSH  0x70
#define A_CNT   0x00012000
#define A_FIRST 0x00020000
#define PF_MASK 0x0e
#define PF_XU   9
#define PF2_CLS 3
#define PF2_FIRST 9
#define PF2_XU 4
#define XW_FIRST 2
#define XW_XU 10
#define XW2_FIRST 12
#define XW2_XU 4
#define PULL_ORDER 0x127
#define L_CNT   0x33330022
#define L_FIRST 0x963000ec
#define START   0x00000000
#define LPF_MASK 0x0f
#define L_START 0x44440000
#define POLL_SLEEP 1
#define LA1 3
#define LA2 3
#define LA3 2
#define XW_INFL 1
#define XW_WAVES 2
#ifndef M_CLS
#define M_CLS 0x06570000
#endif
#define AGG 4
#define PF_NX 0
#define PF2_N 4
#define GO_CLS 6
#define GO_ITEM 7
#define WL_CLS 9
#define AG_NT 0
#define WARM_DRAIN 0
#define AGLD glds16
#define X_POL "sc0 sc1"
#define T_CNT   0x00021000
#define T_FIRST 0x00010000
#define AGD 4
#define PUSH_A 1
#define PUSH_B 3
#define PULL_D 2
#define OUT_SC "sc0 sc1"

constexpr int NR = 8, MS = 512, KD = 4096, ND = 4096, NS = 512;
constexpr int GRID = 256, BLOCK = 512;
constexpr int NSLOT = 4;
constexpr int SLOT_BYTES = 32768;
constexpr int CTL_OFF = NSLOT * SLOT_BYTES;
constexpr int JUNK_OFF = CTL_OFF + 256;
constexpr int LDS_BYTES = JUNK_OFF + 256;
constexpr int BIAS = 1 << 21;
constexpr u64 DEADLINE_TICKS = 40000000ull;

constexpr size_t WS_CNT = 0;
constexpr size_t WS_TMO = WS_CNT + GRID * 128;
constexpr int SHORT_BYTES = 2816;
constexpr size_t Y_LONG = 256 * SHORT_BYTES;
constexpr size_t Y_BYTES = Y_LONG + 256 * 2048;
constexpr size_t WS_IN = 2 << 20;
constexpr size_t IN_BYTES = 2 << 20;
constexpr size_t IN_X = Y_BYTES;
constexpr size_t IN_FLAG = IN_X + 128 * KD;
constexpr size_t IN_XFLAG = IN_FLAG + 32 * 128;
constexpr size_t WS_OUT = WS_IN + 2 * NR * IN_BYTES;
constexpr size_t OUT_BYTES = Y_BYTES;
constexpr size_t WS_END = WS_OUT + 2 * NR * OUT_BYTES;
static_assert(IN_XFLAG + 16 * 128 <= IN_BYTES, "input block overflow");
static_assert(AGU + (PF_XU + 1) / 2 * 2 <= 16 && AGU + PF2_FIRST + PF2_XU <= 16 && PF2_XU % 2 == 0, "warm-up rows out of range");
constexpr bool plan_ok() {
    unsigned r = 0, a = 0, l = 0, tt = 0;
    for (int c = 0; c < 8; ++c) {
        const int rc = (R_CNT >> 4 * c) & 15, ac = (A_CNT >> 4 * c) & 15, lc = (L_CNT >> 4 * c) & 15;
        if (rc > 3 || ac > 2 || lc > 3) return false;
        if ((c && !rc && !ac && !((PF_MASK >> c) & 1)) || (!lc && !((LPF_MASK >> c) & 1))) return false;
        r += ((1u << rc) - 1) << ((R_FIRST >> 4 * c) & 15);
        a += ((1u << ac) - 1) << ((A_FIRST >> 4 * c) & 15);
        const int tc = (T_CNT >> 4 * c) & 15;
        if (tc > 2 || !tc != !ac) return false;
        tt += ((1u << tc) - 1) << ((T_FIRST >> 4 * c) & 15);
        l += ((1u << lc) - 1) << (((unsigned)L_FIRST >> 4 * c) & 15);
        bool pulled = false;
        for (int t = 0; t < 4; ++t) pulled |= c && ((PULL_ORDER >> 4 * t) & 15) == c;
        if (rc && pulled == (bool)((R_PUSH >> c) & 1)) return false;
        if (!rc && (pulled || ((R_PUSH >> c) & 1))) return false;
        if ((((M_CLS >> 4 * c) & 15) != 0) != (bool)((R_PUSH >> c) & 1) || ((M_CLS >> 4 * c) & 15) > 7) return false;
    }
    return r == (1u << (16 - AGU)) - 1 && a == (1u << AGU) - 1 && tt == a && l == 0xffffu && !(R_CNT & 15) && !(A_CNT & 15) && !(PF_MASK & 1);
}
static_assert(XW_XU % 2 == 0 && XW2_XU % 2 == 0 && XW_FIRST + XW_XU <= 16 && XW2_FIRST + XW2_XU <= 16, "warm-up rows out of range");
static_assert(plan_ok(), "the tiles must cover every unit once");
static_assert(AGU >= 1 && AGU <= 4, "gathered rows");
#define STAMP(i) do { } while (0)

constexpr int pull_units() {
    int n = 0;
    for (int t = 0; t < 4; ++t) n += (R_CNT >> 4 * ((PULL_ORDER >> 4 * t) & 15)) & 15;
    return n;
}
constexpr u64 pull_table() {
    u64 v = 0; int n = 0;
    for (int t = 0; t < 4; ++t) {
        const int c = (PULL_ORDER >> 4 * t) & 15;
        for (int u = 0; u < ((R_CNT >> 4 * c) & 15); ++u) v |= (u64)(t << 6 | u << 4 | (AGU + ((R_FIRST >> 4 * c) & 15) + u)) << 8 * n++;
    }
    return v;
}

struct Group { const char* wblk; char* wsd; char* wss; int dst; int src; };
struct Params {
    const char* x;
    char* myws;
    float* out;
    const char* wloc;
    const float* sx;
    const float* sw;
    int rank;
    int pad[3];
    Group g[NR];
};

#define WAIT_LDS() __builtin_amdgcn_s_waitcnt(0xC07F)
#define WAIT_VM(n) asm volatile("s_waitcnt vmcnt(%0)" :: "n"(n) : "memory")
#define PIN() __builtin_amdgcn_sched_barrier(0)
#define BARRIER() do { asm volatile("" ::: "memory"); __builtin_amdgcn_s_barrier(); asm volatile("" ::: "memory"); } while (0)

__device__ __forceinline__ void glds16(const char* sbase, unsigned voff, unsigned lds) {
    asm volatile("s_mov_b32 m0, %2\n\ts_nop 0\n\tglobal_load_lds_dwordx4 %0, %1" :: "v"(voff), "s"(sbase), "s"(lds) : "memory");
}
__device__ __forceinline__ void glds16_sys(const char* sbase, unsigned voff, unsigned lds) {
    asm volatile("s_mov_b32 m0, %2\n\ts_nop 0\n\tglobal_load_lds_dwordx4 %0, %1 sc0 sc1" :: "v"(voff), "s"(sbase), "s"(lds) : "memory");
}
__device__ __forceinline__ void glds4(const char* sbase, unsigned voff) {
    asm volatile("s_mov_b32 m0, %2\n\ts_nop 0\n\tglobal_load_lds_dword %0, %1" :: "v"(voff), "s"(sbase), "s"(JUNK_OFF) : "memory");
}
__device__ __forceinline__ void store_sys(char* sbase, unsigned voff, u32x4 v) {
    asm volatile("global_store_dwordx4 %0, %1, %2 sc0 sc1\n\ts_nop 1" :: "v"(voff), "v"(v), "s"(sbase) : "memory");
}
__device__ __forceinline__ void store_out(char* sbase, unsigned voff, u32x4 v) {
    asm volatile("global_store_dwordx4 %0, %1, %2 " OUT_SC "\n\ts_nop 1" :: "v"(voff), "v"(v), "s"(sbase) : "memory");
}
__device__ __forceinline__ void set_flag(char* flag, unsigned call, unsigned bits) {
    __hip_atomic_store((GLB_AS u64*)flag, ((u64)call << 32) | bits, __ATOMIC_RELAXED, __HIP_MEMORY_SCOPE_SYSTEM);
}

__device__ __forceinline__ void glds16_x(const char* sbase, unsigned voff, unsigned lds) {
    asm volatile("s_mov_b32 m0, %2\n\ts_nop 0\n\tglobal_load_lds_dwordx4 %0, %1 " X_POL :: "v"(voff), "s"(sbase), "s"(lds) : "memory");
}
__device__ __forceinline__ void load_stage(const char* wp, const char* xp, int cnt, unsigned woff, unsigned xoff, unsigned wslot, unsigned xslot) {
#pragma unroll
    for (int j = 0; j < 4; ++j) glds16(wp + j * 8 * ND, woff, wslot + j * 1024);
    glds16_x(xp, xoff, xslot);
    if (cnt > 1) glds16_x(xp + 32 * KD, xoff, xslot + 4096);
    if (cnt > 2) glds16_x(xp + 64 * KD, xoff, xslot + 8192);
}
__device__ __forceinline__ unsigned w_offset(int lane, int wid) {
    return (unsigned)(wid * 32 + (lane >> 3)) * ND + (((lane & 7) ^ (((lane >> 4) & 3) << 1)) << 4);
}
__device__ __forceinline__ unsigned x_offset(int lane, int wid) {
    return (unsigned)(wid * 8 + (lane >> 3)) * KD + (((lane & 7) ^ ((wid * 4 + (lane >> 4)) & 7)) << 4);
}
template <int U, int LA>
__device__ __forceinline__ void load_tile(const char* wp, const char* xp, int lane, int wid) {
    static_assert(LA < NSLOT, "ring too small");
    const unsigned woff = w_offset(lane, wid), xoff = x_offset(lane, wid);
    unsigned slot = 0;
#pragma unroll 1
    for (int s = 0; s < 32; ++s) {
        if (s >= LA) {
            WAIT_VM((LA - 1) * (4 + U));
            BARRIER();
        }
        load_stage(wp + (size_t)s * 128 * ND, xp + s * 128, U, woff, xoff, slot + wid * 4096, slot + 16384 + wid * 1024);
        slot = (slot + SLOT_BYTES) & (NSLOT * SLOT_BYTES - 1);
    }
#pragma unroll
    for (int r = LA - 1; r >= 0; --r) {
        WAIT_VM(r * (4 + U));
        BARRIER();
    }
}
__device__ __forceinline__ void load_tile_n(const char* wp, const char* xp, int cnt, int lane, int wid) {
    if (cnt == 1) load_tile<1, LA1>(wp, xp, lane, wid);
    else if (cnt == 2) load_tile<2, LA2>(wp, xp, lane, wid);
    else load_tile<3, LA3>(wp, xp, lane, wid);
}

__device__ __forceinline__ void glds16_nt(const char* sbase, unsigned voff, unsigned lds) {
    asm volatile("s_mov_b32 m0, %2\n\ts_nop 0\n\tglobal_load_lds_dwordx4 %0, %1 nt" :: "v"(voff), "s"(sbase), "s"(lds) : "memory");
}
__device__ __forceinline__ void load_tile_nt(const char* wp, const char* xp, int cnt, int lane, int wid) {
    const unsigned woff = w_offset(lane, wid), xoff = x_offset(lane, wid);
    unsigned slot = 0;
#pragma unroll 1
    for (int s = 0; s < 32; ++s) {
        if (s >= 3) {
            WAIT_VM(10);
            BARRIER();
        }
#pragma unroll
        for (int j = 0; j < 4; ++j) glds16_nt(wp + (size_t)(s * 128 + j * 8) * ND, woff, slot + wid * 4096 + j * 1024);
        glds16_sys(xp + s * 128, xoff, slot + 16384 + wid * 1024);
        if (cnt > 1) glds16_sys(xp + 32 * KD + s * 128, xoff, slot + 16384 + 4096 + wid * 1024);
        slot = (slot + SLOT_BYTES) & (NSLOT * SLOT_BYTES - 1);
    }
    WAIT_VM(10);
    BARRIER();
    WAIT_VM(5);
    BARRIER();
    WAIT_VM(0);
    BARRIER();
}
__device__ __forceinline__ bool tight(const char* p) { return (((unsigned)(size_t)p >> 7) & 7) == 3 || (((unsigned)(size_t)p >> 7) & 31) == 31; }

template <int U> struct Frag { v4i w[4]; v4i x[U][4]; };

template <int U>
__device__ __forceinline__ void read_frags(Frag<U>& f, char* smem, unsigned slot, unsigned wad, unsigned xad) {
#pragma unroll
    for (int kk = 0; kk < 4; ++kk) {
        v2i lo = __builtin_amdgcn_ds_read_tr8_b64_v2i32((LDS_AS v2i*)(smem + slot + wad + kk * 4096));
        v2i hi = __builtin_amdgcn_ds_read_tr8_b64_v2i32((LDS_AS v2i*)(smem + slot + wad + kk * 4096 + 1024));
        f.w[kk] = v4i{lo.x, lo.y, hi.x, hi.y};
#pragma unroll
        for (int u = 0; u < U; ++u) f.x[u][kk] = *(const v4i*)(smem + slot + (xad ^ (kk << 5)) + 16384 + u * 4096);
    }
}
template <int U>
__device__ __forceinline__ void mma(const Frag<U>& f, v16i (&acc)[U]) {
#pragma unroll
    for (int kk = 0; kk < 4; ++kk)
#pragma unroll
        for (int u = 0; u < U; ++u) acc[u] = __builtin_amdgcn_mfma_i32_32x32x32_i8(f.w[kk], f.x[u][kk], acc[u], 0, 0, 0);
}
template <int U>
__device__ __forceinline__ void interleave() {
#pragma unroll
    for (int i = 0; i < 4 * U; ++i) {
        __builtin_amdgcn_sched_group_barrier(0x8, 1, 0);
        __builtin_amdgcn_sched_group_barrier(0x2, 1, 0);
        __builtin_amdgcn_sched_group_barrier(0x100, U == 1 ? 3 : 2, 0);
    }
    PIN();
}

template <int U>
__device__ __forceinline__ void tile_compute(char* smem, int lane, int cw, unsigned keep0, float keep1, float keep2) {
    const int g = lane >> 4, q = (lane & 15) >> 1;
    const unsigned wad = (16 * (g >> 1) + q) * 128 + (((2 * cw + (g & 1)) ^ (((q >> 1) & 3) << 1)) << 4) + (lane & 1) * 8;
    const unsigned xad = (lane & 31) * 128 + (((lane >> 5) ^ ((lane >> 1) & 7)) << 4);
    v16i acc[U];
#pragma unroll
    for (int u = 0; u < U; ++u)
#pragma unroll
        for (int i = 0; i < 16; ++i) acc[u][i] = BIAS;
    Frag<U> fa, fb;
    unsigned slot = 0;
    BARRIER();
    read_frags<U>(fa, smem, slot, wad, xad);
    asm volatile("" :: "s"(keep0), "s"(keep1), "s"(keep2));
#pragma unroll 1
    for (int s = 1; s < 31; s += 2) {
        slot = (slot + SLOT_BYTES) & (NSLOT * SLOT_BYTES - 1);
        WAIT_LDS();
        BARRIER();
        read_frags<U>(fb, smem, slot, wad, xad);
        mma<U>(fa, acc);
        interleave<U>();
        slot = (slot + SLOT_BYTES) & (NSLOT * SLOT_BYTES - 1);
        WAIT_LDS();
        BARRIER();
        read_frags<U>(fa, smem, slot, wad, xad);
        mma<U>(fb, acc);
        interleave<U>();
    }
    WAIT_LDS();
    BARRIER();
    read_frags<U>(fb, smem, 3 * SLOT_BYTES, wad, xad);
    mma<U>(fa, acc);
    interleave<U>();
    mma<U>(fb, acc);
    char* area = smem + cw * 4096 + (lane & 31) * 128;
#pragma unroll
    for (int u = 0; u < U; ++u)
#pragma unroll
        for (int j = 0; j < 4; ++j)
            *(v4i*)(area + u * 16384 + (((2 * j + (lane >> 5)) ^ (lane & 7)) << 4)) =
                v4i{acc[u][4 * j], acc[u][4 * j + 1], acc[u][4 * j + 2], acc[u][4 * j + 3]};
    WAIT_LDS();
}

__device__ __forceinline__ void write_out(const u32x4 (&t)[4], float scale, char* obase, int lane) {
    const unsigned off = (lane >> 3) * (NS * 4) + (lane & 7) * 16;
#pragma unroll
    for (int i = 0; i < 4; ++i) {
        f32x4 v;
#pragma unroll
        for (int e = 0; e < 4; ++e) v[e] = (float)(int)(t[i][e] - BIAS) * scale;
        store_out(obase + i * 8 * (NS * 4), off, __builtin_bit_cast(u32x4, v));
    }
}

__device__ __forceinline__ unsigned send_subtile(const u32x4 (&t)[4], char* base, int n, int lane, int pace, char* go, unsigned call) {
    const unsigned off = lane * 16;
    char* const dst = base + n * SHORT_BYTES;
    u32x4 lo[2], hi[2];
#pragma unroll
    for (int i = 0; i < 2; ++i)
        lo[i] = u32x4{__builtin_amdgcn_perm(t[2 * i][1], t[2 * i][0], 0x05040100u), __builtin_amdgcn_perm(t[2 * i][3], t[2 * i][2], 0x05040100u),
                      __builtin_amdgcn_perm(t[2 * i + 1][1], t[2 * i + 1][0], 0x05040100u), __builtin_amdgcn_perm(t[2 * i + 1][3], t[2 * i + 1][2], 0x05040100u)};
    if (pace == 2) WAIT_VM(PUSH_B);
    store_sys(dst, off, lo[0]);
    if (pace == 2) WAIT_VM(PUSH_B);
    store_sys(dst + 1024, off, lo[1]);
    if (go && lane == 0) set_flag(go, call, 0);
#pragma unroll
    for (int i = 0; i < 2; ++i)
        hi[i] = u32x4{__builtin_amdgcn_perm(t[2 * i][1], t[2 * i][0], 0x07060302u), __builtin_amdgcn_perm(t[2 * i][3], t[2 * i][2], 0x07060302u),
                      __builtin_amdgcn_perm(t[2 * i + 1][1], t[2 * i + 1][0], 0x07060302u), __builtin_amdgcn_perm(t[2 * i + 1][3], t[2 * i + 1][2], 0x07060302u)};
    const u32x4 o = hi[0] | hi[1];
    if (__any(((o[0] | o[1] | o[2] | o[3]) & 0xffc0ffc0u) != 0)) {
        store_sys(base + Y_LONG + n * 2048, off, hi[0]);
        store_sys(base + Y_LONG + n * 2048 + 1024, off, hi[1]);
        return 1u;
    }
    unsigned b[4];
#pragma unroll
    for (int i = 0; i < 4; ++i) b[i] = __builtin_amdgcn_perm(hi[i >> 1][(i & 1) * 2 + 1], hi[i >> 1][(i & 1) * 2], 0x06040200u);
    const unsigned h0 = b[0] | ((b[3] << 6) & 0xc0c0c0c0u), h1 = b[1] | ((b[3] << 4) & 0xc0c0c0c0u), h2 = b[2] | ((b[3] << 2) & 0xc0c0c0c0u);
    auto s1 = __builtin_amdgcn_permlane16_swap(h0, h1, false, false);
    auto s2 = __builtin_amdgcn_permlane16_swap(h2, h2, false, false);
    auto s3 = __builtin_amdgcn_permlane32_swap(s1[1], s2[1], false, false);
    const u32x4 h = {h0, h1, h2, s3[1]};
    if (pace == 1) WAIT_VM(PUSH_A); else if (pace == 2) WAIT_VM(PUSH_B);
    if (lane < 48) store_sys(dst + 2048, off, h);
    return 0u;
}

constexpr unsigned READY = 1u << 31, FAILED = 1u << 30;

template <int D>
__device__ __forceinline__ void fetch_subtile(const char* base, int n, unsigned buf, bool raw, int lane) {
    const char* const src = base + n * SHORT_BYTES;
    WAIT_VM(D);
    glds16_sys(src, lane * 16, buf);
    WAIT_VM(D);
    glds16_sys(src + 1024, lane * 16, buf + 1024);
    WAIT_VM(D);
    glds16_sys(raw ? base + Y_LONG + n * 2048 : src + 2048, min(lane, raw ? 63 : 47) * 16, buf + 2048);
    if (raw) glds16_sys(base + Y_LONG + n * 2048 + 1024, lane * 16, buf + 3072);
}
__device__ __forceinline__ void finish_subtile(char* smem, unsigned buf, bool raw, bool failed, float scale, char* obase, int lane) {
    u32x4 t[4];
    const u32x4 lo[2] = {*(const u32x4*)(smem + buf + lane * 16), *(const u32x4*)(smem + buf + 1024 + lane * 16)};
    if (raw) {
#pragma unroll
        for (int i = 0; i < 2; ++i) {
            const u32x4 hi = *(const u32x4*)(smem + buf + 2048 + i * 1024 + lane * 16);
#pragma unroll
            for (int e = 0; e < 4; ++e) {
                t[2 * i + (e >> 1)][(e & 1) * 2] = __builtin_amdgcn_perm(hi[e], lo[i][e], 0x05040100u);
                t[2 * i + (e >> 1)][(e & 1) * 2 + 1] = __builtin_amdgcn_perm(hi[e], lo[i][e], 0x07060302u);
            }
        }
    } else {
        const unsigned ha = buf + 2048 + (lane < 48 ? lane * 16 : (lane - 48) * 16 + 12), hs = lane < 48 ? 4 : 256;
        const unsigned h0 = *(const unsigned*)(smem + ha), h1 = *(const unsigned*)(smem + ha + hs), h2 = *(const unsigned*)(smem + ha + 2 * hs);
        unsigned b[4];
        b[0] = h0 & 0x3f3f3f3fu; b[1] = h1 & 0x3f3f3f3fu; b[2] = h2 & 0x3f3f3f3fu;
        b[3] = ((h0 >> 6) & 0x03030303u) | ((h1 >> 4) & 0x0c0c0c0cu) | ((h2 >> 2) & 0x30303030u);
#pragma unroll
        for (int i = 0; i < 4; ++i) {
            const unsigned l0 = lo[i >> 1][(i & 1) * 2], l1 = lo[i >> 1][(i & 1) * 2 + 1];
            t[i][0] = __builtin_amdgcn_perm(b[i], l0, 0x0c040100u);
            t[i][1] = __builtin_amdgcn_perm(b[i], l0, 0x0c050302u);
            t[i][2] = __builtin_amdgcn_perm(b[i], l1, 0x0c060100u);
            t[i][3] = __builtin_amdgcn_perm(b[i], l1, 0x0c070302u);
        }
    }
    write_out(t, failed ? __builtin_nanf("") : scale, obase, lane);
}

__device__ __forceinline__ void poll_flags(const char* flag, bool active, LDS_AS unsigned* word, char* tmo_, unsigned call) {
    GLB_AS unsigned* tm = (GLB_AS unsigned*)tmo_;
    const u64 t0 = __builtin_amdgcn_s_memrealtime();
    bool waiting = active;
    while (true) {
        const u64 v = __hip_atomic_load((GLB_AS u64*)flag, __ATOMIC_RELAXED, __HIP_MEMORY_SCOPE_SYSTEM);
        if (waiting && (unsigned)(v >> 32) >= call) {
            __hip_atomic_store(word, (unsigned)v | READY, __ATOMIC_RELEASE, __HIP_MEMORY_SCOPE_WORKGROUP);
            waiting = false;
        }
        if (!__any(waiting)) break;
        const u64 dt = __builtin_amdgcn_s_memrealtime() - t0;
        if (dt > 10000 && (dt > DEADLINE_TICKS || __hip_atomic_load(tm, __ATOMIC_RELAXED, __HIP_MEMORY_SCOPE_AGENT) == call)) {
            __hip_atomic_store(tm, call, __ATOMIC_RELAXED, __HIP_MEMORY_SCOPE_AGENT);
            if (waiting) __hip_atomic_store(word, READY | FAILED, __ATOMIC_RELEASE, __HIP_MEMORY_SCOPE_WORKGROUP);
            break;
        }
        __builtin_amdgcn_s_sleep(POLL_SLEEP);
    }
}

template <int PER, int NW, int STEP, int INFL = 63>
__device__ __forceinline__ void warm_l2(const char* wblk, const char* xf, int me, int lane) {
    const unsigned wo = (lane >> 2) * ND + (lane & 3) * 128;
#pragma unroll 1
    for (int n = me; n < 32 * PER; n += STEP) {
        const int kb = n / PER, j = n - kb * PER;
        const bool isw = j < NW;
        if (INFL < 63) WAIT_VM(INFL);
        glds4(isw ? wblk + (size_t)(kb * 128 + j * 16) * ND : xf + (size_t)(j - NW) * 64 * KD + kb * 128, isw ? wo : (unsigned)lane * KD);
    }
}

__device__ __forceinline__ void warm_remote(const char* wblk, const char* x, int me, int lane) {
    constexpr int NWARM = __builtin_popcount(PF_MASK) * 8 + (4 - AGG) * 2;
    if (PF_NX == 0) {
        warm_l2<8 + (PF_XU + 1) / 2, 8, NWARM>(wblk, x + AGU * 32 * KD, me, lane);
        if (PF2_XU && me < PF2_N * 2) warm_l2<PF2_XU ? PF2_XU / 2 : 1, 0, PF2_N * 2>(wblk, x + (AGU + PF2_FIRST) * 32 * KD, me, lane);
    } else if (me >= 2 * PF_NX) {
        warm_l2<8, 8, NWARM - 2 * PF_NX>(wblk, x, me - 2 * PF_NX, lane);
    } else {
        warm_l2<(PF_XU + 1) / 2, 0, PF_NX ? 2 * PF_NX : 1>(wblk, x + AGU * 32 * KD, me, lane);
        if (PF2_XU && me < PF2_N * 2) warm_l2<PF2_XU ? PF2_XU / 2 : 1, 0, PF2_N * 2>(wblk, x + (AGU + PF2_FIRST) * 32 * KD, me, lane);
    }
}
__device__ __forceinline__ void advance(LDS_AS unsigned* ctl, char* counter, unsigned call, int lane) {
    asm volatile("" :: "s"(call) : "memory");
    if (lane == 0 && __hip_atomic_fetch_add(ctl + 12, 1u, __ATOMIC_RELAXED, __HIP_MEMORY_SCOPE_WORKGROUP) == 7)
        __hip_atomic_store((GLB_AS unsigned*)counter, call, __ATOMIC_RELAXED, __HIP_MEMORY_SCOPE_AGENT);
}

typedef const CST_AS Params* KArgs;
#define FRESH() KArgs kp = kp0; int b = b0, wid = wid0, lane = lane0; \
    asm volatile("" : "+s"(kp), "+s"(b), "+s"(wid) :: "memory"); asm volatile("" : "+v"(lane))
#define CALL() (*(const CST_AS unsigned*)(kp->myws + WS_CNT + b * 128) + 1u)
#define SCALE() (*(const CST_AS float*)kp->sx * *(const CST_AS float*)kp->sw)

__global__ void __launch_bounds__(BLOCK, 2) fused_kernel(Params) {
    extern __shared__ __attribute__((aligned(16))) char smem[];
    const KArgs kp = (KArgs)__builtin_amdgcn_kernarg_segment_ptr(), kp0 = kp;
    const int lane = threadIdx.x & 63, wid = __builtin_amdgcn_readfirstlane(threadIdx.x >> 6), b = blockIdx.x;
    const int lane0 = lane, wid0 = wid, b0 = b;
    const u64 tstart = __builtin_amdgcn_s_memrealtime();
#define grp (b & 7)
#define cls (b >> 5)
#define pan ((b >> 3) & 3)
#define local ((b & 7) == 7)
#define NIB(word) (((unsigned)(word) >> (4 * cls)) & 15)
    LDS_AS unsigned* ctl = (LDS_AS unsigned*)(smem + CTL_OFF);
    const bool link = cls == 0 && !local;

    if (link) {
        constexpr int PPU = 32 / AGG, NP = AGU * PPU;
        static_assert(AGD >= 2 && AGD <= 8 && AGD <= PPU && (AGG == 2 || AGG == 4), "staging ring");
        const char* const x = kp->x;
        const bool ag = pan < AGG;
        const unsigned agoff = (pan * 4 + (wid & 3)) * 1024 + lane * 16;
        if (ag && wid < 4) {
#pragma unroll
            for (int i = 0; i < AGD; ++i) AGLD(x + i * (AGG * 4096), agoff, (wid * 8 + i) * 1024);
        }
        if (wid == 7) {
            if (lane < 32) ctl[lane] = 0;
            WAIT_LDS();
        }
        BARRIER();
        if (!ag && (wid >> 1) == 2) warm_remote(kp->g[grp].wblk, x, (pan - AGG) * 2 + (wid & 1), lane);
        const unsigned call = CALL(), par = call & 1;
        const float scale = SCALE();
        const int rank = kp->rank, src = kp->g[grp].src;
        char* const din = kp->g[grp].wsd + WS_IN + (size_t)(par * NR + rank) * IN_BYTES;
        STAMP(1);
        if (ag && wid < 4) {
            int r = 0;
#pragma unroll 1
            for (int i = 0; i < NP; ++i) {
                if (i >= AGD - 1) WAIT_VM(2 * AGD - 2); else WAIT_VM(AGD - 1);
                if (i >= AGD && ((i - AGD) & (PPU - 1)) == PPU - 1 && lane == 0)
                    __hip_atomic_fetch_add(ctl + 8 + (i - AGD) / PPU, 1u, __ATOMIC_RELAXED, __HIP_MEMORY_SCOPE_WORKGROUP);
                const u32x4 v = *(const u32x4*)(smem + (wid * 8 + r) * 1024 + lane * 16);
                store_sys(din + IN_X + i * (AGG * 4096), agoff, v);
                AGLD(x + (i + AGD < NP ? i + AGD : 0) * (AGG * 4096), agoff, (wid * 8 + r) * 1024);
                r = r == AGD - 1 ? 0 : r + 1;
            }
            WAIT_VM(0);
            if (lane == 0) __hip_atomic_fetch_add(ctl + 8 + AGU - 1, 1u, __ATOMIC_RELAXED, __HIP_MEMORY_SCOPE_WORKGROUP);
            STAMP(3);
        }
        advance(ctl, kp->myws + WS_CNT + b * 128, call, lane);
        constexpr int NPULL = PULL_ORDER >> 12 ? 4 : (PULL_ORDER >> 8 ? 3 : (PULL_ORDER >> 4 ? 2 : 1));
        constexpr unsigned PULL_BUF = 0;
        constexpr int NPU = pull_units();
        static_assert(PULL_BUF + NPU * 16384 <= CTL_OFF && PULL_D <= 2, "pull buffers");
        LDS_AS unsigned* landed = ctl + 16;
        static_assert(NPULL < 4, "lane 3 watches the go-ahead");
        if (wid == 7) {
            for (int u = 0; ag && u < AGU; ++u) {
                while (__hip_atomic_load(ctl + 8 + u, __ATOMIC_RELAXED, __HIP_MEMORY_SCOPE_WORKGROUP) < 4) __builtin_amdgcn_s_sleep(2);
                if (lane == 0) set_flag(din + IN_XFLAG + (u * 4 + pan) * 128, call, 0);
            }
        } else if (wid == 6) {
            const int c = (PULL_ORDER >> (4 * (lane & 3))) & 15;
            while (__hip_atomic_load(ctl + 12, __ATOMIC_RELAXED, __HIP_MEMORY_SCOPE_WORKGROUP) < 8) __builtin_amdgcn_s_sleep(4);
            poll_flags(kp->myws + WS_IN + (size_t)(par * NR + src) * IN_BYTES + IN_FLAG + (c * 4 + pan) * 128, lane < NPULL || (GO_CLS && lane == 3),
                       ctl + 4 + (lane & 3), kp->myws + WS_TMO, call);
        } else if (wid < 4) {
            if (GO_CLS) while (!(__hip_atomic_load(ctl + 7, __ATOMIC_RELAXED, __HIP_MEMORY_SCOPE_WORKGROUP) & READY)) __builtin_amdgcn_s_sleep(1);
            const char* const sout = kp->g[grp].wss + WS_OUT + (unsigned)(par * NR + rank) * (unsigned)OUT_BYTES;
#pragma unroll 1
            for (int j = 0; j < NPU; ++j) {
                const int e = (int)(pull_table() >> (8 * j)) & 255, t = e >> 6, u = (e >> 4) & 3, unit = e & 15;
                unsigned word = __hip_atomic_load(ctl + 4 + t, __ATOMIC_ACQUIRE, __HIP_MEMORY_SCOPE_WORKGROUP);
                if (!(word & READY)) {
                    WAIT_VM(0);
                    if (lane == 0) __hip_atomic_store(landed + wid, (unsigned)j, __ATOMIC_RELEASE, __HIP_MEMORY_SCOPE_WORKGROUP);
                    while (!((word = __hip_atomic_load(ctl + 4 + t, __ATOMIC_ACQUIRE, __HIP_MEMORY_SCOPE_WORKGROUP)) & READY)) __builtin_amdgcn_s_sleep(1);
                }
                word = __builtin_amdgcn_readfirstlane(word);
                if (word & FAILED) WAIT_VM(0);
                else fetch_subtile<PULL_D>(sout, (unit * 4 + pan) * 4 + wid, PULL_BUF + (j * 4 + wid) * 4096, (word >> (u * 4 + wid)) & 1, lane);
                if (lane == 0) __hip_atomic_store(landed + wid, (unsigned)j, __ATOMIC_RELEASE, __HIP_MEMORY_SCOPE_WORKGROUP);
            }
            WAIT_VM(0);
            if (lane == 0) __hip_atomic_store(landed + wid, (unsigned)NPU, __ATOMIC_RELEASE, __HIP_MEMORY_SCOPE_WORKGROUP);
        }
        char* const out = (char*)kp->out + ((size_t)src * MS * NS + pan * 128) * 4;
        while (true) {
            unsigned q = 0;
            if (lane == 0) q = __hip_atomic_fetch_add(ctl + 13, 1u, __ATOMIC_RELAXED, __HIP_MEMORY_SCOPE_WORKGROUP);
            q = __builtin_amdgcn_readfirstlane(q);
            if (q >= NPU * 4) break;
            const int j = q >> 2, cw = q & 3;
            const int e = (int)(pull_table() >> (8 * j)) & 255, t = e >> 6, u = (e >> 4) & 3, unit = e & 15;
            while (__hip_atomic_load(landed + cw, __ATOMIC_ACQUIRE, __HIP_MEMORY_SCOPE_WORKGROUP) <= (unsigned)j) __builtin_amdgcn_s_sleep(2);
            const unsigned word = __builtin_amdgcn_readfirstlane(ctl[4 + t]);
            finish_subtile(smem, PULL_BUF + q * 4096, (word >> (u * 4 + cw)) & 1, word & FAILED, scale,
                           out + (unit * 32 * NS + cw * 32) * 4, lane);
        }
    } else {
        int cnt = NIB(local ? L_CNT : R_CNT);
        const char* wp = kp->g[grp].wblk;
        const char* xp = kp->x;
        if (wid == 7) {
            if (lane < 32) ctl[lane] = 0;
            WAIT_LDS();
        }
        if (((local ? LPF_MASK : PF_MASK) >> cls) & 1) {
            if (wid == 6) asm volatile("" :: "s"(CALL()), "v"(SCALE()));
            if ((wid >> 1) == 2) {
                const int me = (__builtin_popcount((local ? LPF_MASK : PF_MASK) & ((1 << cls) - 1)) * 4 + pan) * 2 + (wid & 1);
                if (local) {
                    warm_l2<8, 0, __builtin_popcount(LPF_MASK) * 8>(wp, xp, me, lane);
                    warm_l2<8, 8, __builtin_popcount(LPF_MASK) * 8>(wp, xp, me, lane);
                } else {
                    warm_remote(wp, xp, (4 - AGG) * 2 + me, lane);
                    if (cls == WL_CLS) {
                        const char* wl = kp->wloc;
#pragma unroll 1
                        for (int p = 0; p < 4; ++p)
                            if (tight(wl + p * 128))
#pragma unroll 1
                                for (int n = pan * 2 + (wid & 1); n < 64; n += 8) glds4(wl + p * 128 + (size_t)n * 64 * ND, lane * ND);
                    }
                }
                if (WARM_DRAIN) WAIT_VM(0);
            }
            BARRIER();
        }
        if (NIB(local ? L_START : START))
            while (__builtin_amdgcn_s_memrealtime() - tstart < (u64)(NIB(local ? L_START : START) * 64)) __builtin_amdgcn_s_sleep(2);
        xp += (size_t)(local ? NIB(L_FIRST) : AGU + NIB(R_FIRST)) * 32 * KD;
        int task = cnt ? 0 : 1;
        const int ntask = !local && NIB(A_CNT) ? 2 : 1;
#pragma unroll 1
        for (; task < ntask; ++task) {
            unsigned bad = 0, afirst = 0;
            if (task == 1) {
                BARRIER();
                FRESH();
                const unsigned call = CALL();
                const char* sin = kp->myws + WS_IN + (size_t)((call & 1) * NR + kp->g[grp].src) * IN_BYTES;
                wp = kp->wloc;
                const bool tp = tight(wp + pan * 128);
                cnt = NIB(tp ? T_CNT : A_CNT);
                afirst = NIB(tp ? T_FIRST : A_FIRST);
                if (wid == 1) poll_flags(sin + IN_XFLAG + ((afirst + cnt - 1) * 4 + (lane & 3)) * 128, lane < AGG,
                                         ctl + 4 + (lane & 3), kp->myws + WS_TMO, call);
                WAIT_LDS();
                BARRIER();
                bad = __builtin_amdgcn_readfirstlane((ctl[4] | ctl[5] | ctl[6] | ctl[7]) & FAILED);
                xp = sin + IN_X + (size_t)afirst * 32 * KD;
                STAMP(4);
            }
            if (wid < 4 && !bad) {
                FRESH();
                if (AG_NT && task == 1 && !tight(wp + pan * 128)) load_tile_nt(wp + pan * 128, xp, cnt, lane, wid);
                else load_tile_n(wp + pan * 128, xp, cnt, lane, wid);
            }
            FRESH();
            const bool direct = local || task == 1, push = (R_PUSH >> cls) & 1;
            char *myws = kp->myws, *out = (char*)kp->out, *din = kp->g[grp].wsd;
            const char *sxp = (const char*)kp->sx, *swp = (const char*)kp->sw;
            unsigned rank = kp->rank, src = kp->g[grp].src, dst = kp->g[grp].dst;
            asm volatile("" : "+s"(myws), "+s"(out), "+s"(din), "+s"(sxp), "+s"(swp), "+s"(rank), "+s"(src), "+s"(dst));
            const unsigned done = *(const CST_AS unsigned*)(myws + WS_CNT + b * 128);
            const float sx = *(const CST_AS float*)sxp, sw = *(const CST_AS float*)swp;
            din += WS_IN + rank * (unsigned)IN_BYTES;
            char* base = direct ? out + ((task == 1 ? src * 16 + afirst : rank * 16 + NIB(L_FIRST)) * 32 * NS + pan * 128) * 4
                       : push ? din : myws + WS_OUT + dst * (unsigned)OUT_BYTES;
            unsigned odd = direct ? 0u : push ? (unsigned)(NR * IN_BYTES) : (unsigned)(NR * OUT_BYTES);
            asm volatile("" : "+s"(base), "+s"(din), "+s"(odd));
            if (wid >= 4 && !bad) {
                if (cnt == 1) tile_compute<1>(smem, lane, wid - 4, done, sx, sw);
                else if (cnt == 2) tile_compute<2>(smem, lane, wid - 4, done, sx, sw);
                else tile_compute<3>(smem, lane, wid - 4, done, sx, sw);
            }
            if (cnt > 1) BARRIER();
            const unsigned call = done + 1;
            const float scale = bad ? __builtin_nanf("") : sx * sw;
            if (call & 1) { base += odd; din += NR * IN_BYTES; }
            unsigned rawbits = 0;
#pragma unroll 1
            for (int i = wid ^ 4; i < cnt * 4; i += 8) {
                u32x4 t[4];
#pragma unroll
                for (int j = 0; j < 4; ++j)
                    t[j] = *(const u32x4*)(smem + i * 4096 + (8 * j + (lane >> 3)) * 128 + (((lane & 7) ^ (lane >> 3)) << 4));
                if (GO_ITEM >= 8 && !direct && cls == GO_CLS && i == GO_ITEM && lane == 0) set_flag(din + IN_FLAG + pan * 128, call, 0);
                if (direct) write_out(t, scale, base + (i >> 2) * (32 * NS * 4) + (i & 3) * 128, lane);
                else rawbits |= send_subtile(t, base, ((AGU + NIB(R_FIRST) + (i >> 2)) * 4 + pan) * 4 + (i & 3), lane, push && cnt > 1 ? 1 + (i >> 3) : 0,
                                             GO_ITEM < 8 && cls == GO_CLS && i == GO_ITEM ? din + IN_FLAG + pan * 128 : nullptr, call) << i;
            }
            if (!direct) {
                if (rawbits && lane == 0) __hip_atomic_fetch_or(ctl, rawbits, __ATOMIC_RELAXED, __HIP_MEMORY_SCOPE_WORKGROUP);
                asm volatile("s_waitcnt vmcnt(0) lgkmcnt(0)" ::: "memory");
                BARRIER();
                STAMP(3);
                if (wid == 7 && lane == 0) set_flag(din + IN_FLAG + (b >> 3) * 128, call, ctl[0]);
            }
        }
        FRESH();
        const unsigned call = CALL();
        advance(ctl, kp->myws + WS_CNT + b * 128, call, lane);
        if (!local) {
            const int src = kp->g[grp].src;
            const char* sin = kp->myws + WS_IN + (size_t)((call & 1) * NR + src) * IN_BYTES;
            char* const out = (char*)kp->out + ((size_t)src * MS * NS + pan * 128) * 4;
            const float scale = SCALE();
#pragma unroll 1
            for (int c = 1; c < 8; ++c) {
                if (((M_CLS >> (4 * c)) & 15) != cls) continue;
                BARRIER();
                if (wid == 1) poll_flags(sin + IN_FLAG + (c * 4 + pan) * 128, lane == 0, ctl + 16 + c, kp->myws + WS_TMO, call);
                WAIT_LDS();
                BARRIER();
                const unsigned word = __builtin_amdgcn_readfirstlane(ctl[16 + c]);
                for (int i = wid; i < ((R_CNT >> (4 * c)) & 15) * 4; i += 8) {
                    const int unit = AGU + ((R_FIRST >> (4 * c)) & 15) + (i >> 2), cw = i & 3;
                    const bool raw = (word >> i) & 1, failed = word & FAILED;
                    if (!failed) fetch_subtile<63>(sin, (unit * 4 + pan) * 4 + cw, i * 4096, raw, lane);
                    WAIT_VM(0);
                    finish_subtile(smem, i * 4096, raw, failed, scale, out + (unit * 32 * NS + cw * 32) * 4, lane);
                }
            }
        }
    }
}
#undef grp
#undef cls
#undef pan
#undef local

extern "C" void kernel_launch(int rank, int nranks, ncclComm_t comm,
                              hipStream_t stream,
                              void* const* d_in, const int* in_sizes, int n_in,
                              void* d_out, int out_size,
                              const int* in_dtypes, int out_dtype,
                              void* const* d_out_all,
                              void* const* d_ws_all, size_t ws_bytes) {
    if (nranks != NR || (unsigned)rank >= (unsigned)NR || ws_bytes < WS_END) return;
    static bool init[NR] = {};
    if (!init[rank]) {
        HIP_CHECK(hipFuncSetAttribute((const void*)fused_kernel, hipFuncAttributeMaxDynamicSharedMemorySize, LDS_BYTES));
        init[rank] = true;
    }
    Params p;
    p.x = (const char*)d_in[0];
    p.myws = (char*)d_ws_all[rank];
    p.out = (float*)d_out;
    p.wloc = (const char*)d_in[1] + rank * NS;
    p.sx = (const float*)d_in[2];
    p.sw = (const float*)d_in[3];
    p.rank = rank;
    p.pad[0] = p.pad[1] = p.pad[2] = 0;
    for (int g = 0; g < NR; ++g) {
        const int dst = (rank + 1 + g) & 7, src = (rank + 7 - g) & 7;
        p.g[g].wblk = (const char*)d_in[1] + dst * NS;
        p.g[g].wsd = (char*)d_ws_all[dst];
        p.g[g].wss = (char*)d_ws_all[src];
        p.g[g].dst = dst;
        p.g[g].src = src;
    }
    hipLaunchKernelGGL(fused_kernel, dim3(GRID), dim3(BLOCK), LDS_BYTES, stream, p);
    HIP_CHECK(hipGetLastError());
}
```

```cpp
#include <hip/hip_runtime.h>
#include <rccl/rccl.h>
#include <cstdio>

#define HIP_CHECK(expr) do { hipError_t _e = (expr); if (_e != hipSuccess) { \
    fprintf(stderr, "HIP error %s at %s:%d\n", hipGetErrorString(_e), __FILE__, __LINE__); \
    return; } } while (0)

typedef int v2i __attribute__((ext_vector_type(2)));
typedef int v4i __attribute__((ext_vector_type(4)));
typedef int v16i __attribute__((ext_vector_type(16)));
typedef unsigned u32x4 __attribute__((ext_vector_type(4)));
typedef float f32x4 __attribute__((ext_vector_type(4)));
typedef unsigned long long u64;
#define LDS_AS __attribute__((address_space(3)))
#define GLB_AS __attribute__((address_space(1)))
#define CST_AS __attribute__((address_space(4)))

#define AGU 3
#define R_CNT   0x33210220
#define R_FIRST 0x63100b90
#define R_PUSH  0x70
#define A_CNT   0x00012000
#define A_FIRST 0x00020000
#define PF_MASK 0x0e
#define PF_XU   9
#define PF2_CLS 3
#define PF2_FIRST 9
#define PF2_XU 4
#define XW_FIRST 2
#define XW_XU 10
#define XW2_FIRST 12
#define XW2_XU 4
#define PULL_ORDER 0x127
#define L_CNT   0x33330022
#define L_FIRST 0x963000ec
#define START   0x00000000
#define LPF_MASK 0x0f
#define L_START 0x44440000
#define POLL_SLEEP 8
#define LA1 3
#define LA2 3
#define LA3 2
#define XW_INFL 1
#define XW_WAVES 2
#ifndef M_CLS
#define M_CLS 0x06570000
#endif
#define AGG 4
#define PF_NX 0
#define PF2_N 3
#define GO_CLS 6
#define GO_ITEM 7
#define GO_LATE
#define WL_CLS 9
#define AG_NT 0
#define WARM_DRAIN 0
#define AGLD glds16
#define X_POL "sc0 sc1"
#define T_CNT   0x00021000
#define T_FIRST 0x00010000
#define LOCAL_GRP 3
#define TOPO 1
#define FSP 16
#define MIR_SLEEP 12
#define SPREAD 1
#define PF2_LO 8
#define M16 6
#define LW_START 0
#define A_START 880
#define AGD 4
#define PUSH_A 1
#define PUSH_B 3
#define PULL_D 2
#define OUT_SC "sc0 sc1"

constexpr int NR = 8, MS = 512, KD = 4096, ND = 4096, NS = 512;
constexpr int GRID = 256, BLOCK = 512;
constexpr int NSLOT = 4;
constexpr int SLOT_BYTES = 32768;
constexpr int CTL_OFF = NSLOT * SLOT_BYTES;
constexpr int MIR_OFF = CTL_OFF + 256;
constexpr int JUNK_OFF = MIR_OFF + 1024;
constexpr int LDS_BYTES = JUNK_OFF + 256;
constexpr int BIAS = 1 << 21;
constexpr u64 DEADLINE_TICKS = 40000000ull;

constexpr size_t WS_CNT = 0;
constexpr size_t WS_TMO = WS_CNT + GRID * 128;
constexpr int SHORT_BYTES = 2816;
constexpr size_t Y_LONG = 256 * SHORT_BYTES;
constexpr size_t Y_BYTES = Y_LONG + 256 * 2048;
constexpr size_t WS_IN = 2 << 20;
constexpr size_t IN_BYTES = 2 << 20;
constexpr size_t IN_X = Y_BYTES;
constexpr size_t IN_FLAG = IN_X + 128 * KD;
constexpr size_t IN_XFLAG = IN_FLAG + 32 * 128;
constexpr size_t WS_OUT = WS_IN + 2 * NR * IN_BYTES;
constexpr size_t OUT_BYTES = Y_BYTES;
constexpr size_t WS_END = WS_OUT + 2 * NR * OUT_BYTES;
static_assert(IN_XFLAG + 16 * 128 <= IN_BYTES, "input block overflow");
static_assert(AGU + (PF_XU + 1) / 2 * 2 <= 16 && AGU + PF2_FIRST + PF2_XU <= 16 && PF2_XU % 2 == 0, "warm-up rows out of range");
constexpr bool plan_ok() {
    unsigned r = 0, a = 0, l = 0, tt = 0;
    for (int c = 0; c < 8; ++c) {
        const int rc = (R_CNT >> 4 * c) & 15, ac = (A_CNT >> 4 * c) & 15, lc = (L_CNT >> 4 * c) & 15;
        if (rc > 3 || ac > 2 || lc > 3) return false;
        if ((c && !rc && !ac && !((PF_MASK >> c) & 1)) || (!lc && !((LPF_MASK >> c) & 1))) return false;
        r += ((1u << rc) - 1) << ((R_FIRST >> 4 * c) & 15);
        a += ((1u << ac) - 1) << ((A_FIRST >> 4 * c) & 15);
        const int tc = (T_CNT >> 4 * c) & 15;
        if (tc > 2 || !tc != !ac) return false;
        tt += ((1u << tc) - 1) << ((T_FIRST >> 4 * c) & 15);
        l += ((1u << lc) - 1) << (((unsigned)L_FIRST >> 4 * c) & 15);
        bool pulled = false;
        for (int t = 0; t < 4; ++t) pulled |= c && ((PULL_ORDER >> 4 * t) & 15) == c;
        if (rc && pulled == (bool)((R_PUSH >> c) & 1)) return false;
        if (!rc && (pulled || ((R_PUSH >> c) & 1))) return false;
        if ((((M_CLS >> 4 * c) & 15) != 0) != (bool)((R_PUSH >> c) & 1) || ((M_CLS >> 4 * c) & 15) > 7) return false;
    }
    return r == (1u << (16 - AGU)) - 1 && a == (1u << AGU) - 1 && tt == a && l == 0xffffu && !(R_CNT & 15) && !(A_CNT & 15) && !(PF_MASK & 1);
}
static_assert(XW_XU % 2 == 0 && XW2_XU % 2 == 0 && XW_FIRST + XW_XU <= 16 && XW2_FIRST + XW2_XU <= 16, "warm-up rows out of range");
static_assert(plan_ok(), "the tiles must cover every unit once");
static_assert(AGU >= 1 && AGU <= 4, "gathered rows");
#define STAMP(i) do { } while (0)

constexpr int pull_units() {
    int n = 0;
    for (int t = 0; t < 4; ++t) n += (R_CNT >> 4 * ((PULL_ORDER >> 4 * t) & 15)) & 15;
    return n;
}
constexpr u64 pull_table() {
    u64 v = 0; int n = 0;
    for (int t = 0; t < 4; ++t) {
        const int c = (PULL_ORDER >> 4 * t) & 15;
        for (int u = 0; u < ((R_CNT >> 4 * c) & 15); ++u) v |= (u64)(t << 6 | u << 4 | (AGU + ((R_FIRST >> 4 * c) & 15) + u)) << 8 * n++;
    }
    return v;
}

struct Group {
    const char* x;
    const char* wblk;
    char* myws;
    char* peerws;
    char* out;
    const char* sx;
    const char* sw;
    int rank;
    int peer;
};
struct Params { Group g[NR]; };
static_assert(sizeof(Group) == 64, "one line");

#define WAIT_LDS() __builtin_amdgcn_s_waitcnt(0xC07F)
#define WAIT_VM(n) asm volatile("s_waitcnt vmcnt(%0)" :: "n"(n) : "memory")
#define PIN() __builtin_amdgcn_sched_barrier(0)
#define BARRIER() do { asm volatile("" ::: "memory"); __builtin_amdgcn_s_barrier(); asm volatile("" ::: "memory"); } while (0)

__device__ __forceinline__ void glds16(const char* sbase, unsigned voff, unsigned lds) {
    asm volatile("s_mov_b32 m0, %2\n\ts_nop 0\n\tglobal_load_lds_dwordx4 %0, %1" :: "v"(voff), "s"(sbase), "s"(lds) : "memory");
}
__device__ __forceinline__ void glds16_sys(const char* sbase, unsigned voff, unsigned lds) {
    asm volatile("s_mov_b32 m0, %2\n\ts_nop 0\n\tglobal_load_lds_dwordx4 %0, %1 sc0 sc1" :: "v"(voff), "s"(sbase), "s"(lds) : "memory");
}
__device__ __forceinline__ void glds4(const char* sbase, unsigned voff) {
    asm volatile("s_mov_b32 m0, %2\n\ts_nop 0\n\tglobal_load_lds_dword %0, %1" :: "v"(voff), "s"(sbase), "s"(JUNK_OFF) : "memory");
}
__device__ __forceinline__ void store_sys(char* sbase, unsigned voff, u32x4 v) {
    asm volatile("global_store_dwordx4 %0, %1, %2 sc0 sc1\n\ts_nop 1" :: "v"(voff), "v"(v), "s"(sbase) : "memory");
}
__device__ __forceinline__ void store_out(char* sbase, unsigned voff, u32x4 v) {
    asm volatile("global_store_dwordx4 %0, %1, %2 " OUT_SC "\n\ts_nop 1" :: "v"(voff), "v"(v), "s"(sbase) : "memory");
}
__device__ __forceinline__ void set_flag(char* flag, unsigned call, unsigned bits) {
    __hip_atomic_store((GLB_AS u64*)flag, ((u64)call << 32) | (call << 16) | bits, __ATOMIC_RELAXED, __HIP_MEMORY_SCOPE_SYSTEM);
}

__device__ __forceinline__ void glds16_x(const char* sbase, unsigned voff, unsigned lds) {
    asm volatile("s_mov_b32 m0, %2\n\ts_nop 0\n\tglobal_load_lds_dwordx4 %0, %1 " X_POL :: "v"(voff), "s"(sbase), "s"(lds) : "memory");
}
__device__ __forceinline__ void load_stage(const char* wp, const char* xp, int cnt, unsigned woff, unsigned xoff, unsigned wslot, unsigned xslot) {
#pragma unroll
    for (int j = 0; j < 4; ++j) glds16(wp + j * 8 * ND, M16 && j >= 2 ? woff ^ 16 : woff, wslot + j * 1024);
    glds16_x(xp, xoff, xslot);
    if (cnt > 1) glds16_x(xp + 32 * KD, xoff, xslot + 4096);
    if (cnt > 2) glds16_x(xp + 64 * KD, xoff, xslot + 8192);
}
__device__ __forceinline__ unsigned w_offset(int lane, int wid) {
    return (unsigned)(wid * 32 + (lane >> 3)) * ND + (((lane & 7) ^ (((lane >> 4) & 3) << 1)) << 4);
}
__device__ __forceinline__ unsigned x_offset(int lane, int wid) {
    return (unsigned)(wid * 8 + (lane >> 3)) * KD + (((lane & 7) ^ ((wid * 4 + (lane >> 4)) & 7)) << 4);
}
template <int U, int LA>
__device__ __forceinline__ void load_tile(const char* wp, const char* xp, unsigned woff, unsigned xoff, int wid) {
    static_assert(LA < NSLOT, "ring too small");
    unsigned slot = 0;
#pragma unroll 1
    for (int s = 0; s < 32; ++s) {
        if (s >= LA) {
            WAIT_VM((LA - 1) * (4 + U));
            BARRIER();
        }
        load_stage(wp + (size_t)s * 128 * ND, xp + s * 128, U, woff, xoff, slot + wid * 4096, slot + 16384 + wid * 1024);
        slot = (slot + SLOT_BYTES) & (NSLOT * SLOT_BYTES - 1);
    }
#pragma unroll
    for (int r = LA - 1; r >= 0; --r) {
        WAIT_VM(r * (4 + U));
        BARRIER();
    }
}
__device__ __forceinline__ void load_tile_n(const char* wp, const char* xp, int cnt, unsigned woff, unsigned xoff, int wid) {
    if (cnt == 1) load_tile<1, LA1>(wp, xp, woff, xoff, wid);
    else if (cnt == 2) load_tile<2, LA2>(wp, xp, woff, xoff, wid);
    else load_tile<3, LA3>(wp, xp, woff, xoff, wid);
}

__device__ __forceinline__ void glds16_nt(const char* sbase, unsigned voff, unsigned lds) {
    asm volatile("s_mov_b32 m0, %2\n\ts_nop 0\n\tglobal_load_lds_dwordx4 %0, %1 nt" :: "v"(voff), "s"(sbase), "s"(lds) : "memory");
}
__device__ __forceinline__ bool tight(const char* p) { return (((unsigned)(size_t)p >> 7) & 7) == 3 || (((unsigned)(size_t)p >> 7) & 31) == 31; }

template <int U> struct Frag { v4i w[4]; v4i x[U][4]; };
template <bool S> struct Acc { v16i a; };
template <> struct Acc<true> { v4i q[4]; };

template <int U, bool S>
__device__ __forceinline__ void read_frags(Frag<U>& f, char* smem, unsigned slot, unsigned wad, unsigned xad) {
#pragma unroll
    for (int kk = 0; kk < 4; ++kk) {
        const unsigned wa = slot + (S ? (wad ^ ((kk & 1) << 4)) + (kk >> 1) * 8192 : wad + kk * 4096);
        v2i lo = __builtin_amdgcn_ds_read_tr8_b64_v2i32((LDS_AS v2i*)(smem + wa));
        v2i hi = __builtin_amdgcn_ds_read_tr8_b64_v2i32((LDS_AS v2i*)(smem + wa + 1024));
        f.w[kk] = v4i{lo.x, lo.y, hi.x, hi.y};
        const unsigned xa = slot + (S ? (xad ^ ((kk >> 1) << 6)) + (kk & 1) * 2048 : xad ^ (kk << 5)) + 16384;
#pragma unroll
        for (int u = 0; u < U; ++u) f.x[u][kk] = *(const v4i*)(smem + xa + u * 4096);
    }
}
template <int U>
__device__ __forceinline__ void mma(const Frag<U>& f, Acc<false> (&acc)[U]) {
#pragma unroll
    for (int kk = 0; kk < 4; ++kk)
#pragma unroll
        for (int u = 0; u < U; ++u) acc[u].a = __builtin_amdgcn_mfma_i32_32x32x32_i8(f.w[kk], f.x[u][kk], acc[u].a, 0, 0, 0);
}
template <int U>
__device__ __forceinline__ void mma(const Frag<U>& f, Acc<true> (&acc)[U]) {
#pragma unroll
    for (int k2 = 0; k2 < 2; ++k2)
#pragma unroll
        for (int u = 0; u < U; ++u)
#pragma unroll
            for (int i = 0; i < 4; ++i)
                acc[u].q[i] = __builtin_amdgcn_mfma_i32_16x16x64_i8(f.w[k2 * 2 + (i >> 1)], f.x[u][k2 * 2 + (i & 1)], acc[u].q[i], 0, 0, 0);
}
template <int U, bool S>
__device__ __forceinline__ void interleave() {
#pragma unroll
    for (int i = 0; i < 4 * U; ++i) {
        __builtin_amdgcn_sched_group_barrier(0x8, S ? 2 : 1, 0);
        __builtin_amdgcn_sched_group_barrier(0x2, 1, 0);
        __builtin_amdgcn_sched_group_barrier(0x100, U == 1 ? 3 : 2, 0);
    }
    PIN();
}

template <int U>
__device__ __forceinline__ void tile_compute(char* smem, int lane, int cw, unsigned keep0, float keep1, float keep2) {
    constexpr bool S = (M16 >> U) & 1;
    const int g = lane >> 4, q = (lane & 15) >> 1;
    const unsigned wad = S ? (16 * g + q) * 128 + (((2 * cw) ^ (((q >> 1) & 3) << 1) ^ (g & 1)) << 4) + (lane & 1) * 8
                           : (16 * (g >> 1) + q) * 128 + (((2 * cw + (g & 1)) ^ (((q >> 1) & 3) << 1) ^ (M16 ? g >> 1 : 0)) << 4) + (lane & 1) * 8;
    const unsigned xad = S ? (lane & 15) * 128 + ((g ^ ((lane >> 1) & 7)) << 4)
                           : (lane & 31) * 128 + (((lane >> 5) ^ ((lane >> 1) & 7)) << 4);
    Acc<S> acc[U];
#pragma unroll
    for (int u = 0; u < U; ++u) {
        if constexpr (S) {
#pragma unroll
            for (int i = 0; i < 4; ++i) acc[u].q[i] = v4i{BIAS, BIAS, BIAS, BIAS};
        } else {
#pragma unroll
            for (int i = 0; i < 16; ++i) acc[u].a[i] = BIAS;
        }
    }
    Frag<U> fa, fb;
    unsigned slot = 0;
    BARRIER();
    read_frags<U, S>(fa, smem, slot, wad, xad);
    asm volatile("" :: "s"(keep0), "s"(keep1), "s"(keep2));
#pragma unroll 1
    for (int s = 1; s < 31; s += 2) {
        slot = (slot + SLOT_BYTES) & (NSLOT * SLOT_BYTES - 1);
        WAIT_LDS();
        BARRIER();
        read_frags<U, S>(fb, smem, slot, wad, xad);
        mma<U>(fa, acc);
        interleave<U, S>();
        slot = (slot + SLOT_BYTES) & (NSLOT * SLOT_BYTES - 1);
        WAIT_LDS();
        BARRIER();
        read_frags<U, S>(fa, smem, slot, wad, xad);
        mma<U>(fb, acc);
        interleave<U, S>();
    }
    WAIT_LDS();
    BARRIER();
    read_frags<U, S>(fb, smem, 3 * SLOT_BYTES, wad, xad);
    mma<U>(fa, acc);
    interleave<U, S>();
    mma<U>(fb, acc);
#pragma unroll
    for (int u = 0; u < U; ++u)
#pragma unroll
        for (int j = 0; j < 4; ++j) {
            if constexpr (S)
                *(v4i*)(smem + cw * 4096 + u * 16384 + (16 * (j & 1) + (lane & 15)) * 128 + (((4 * (j >> 1) + g) ^ (lane & 7)) << 4)) = acc[u].q[j];
            else
                *(v4i*)(smem + cw * 4096 + u * 16384 + (lane & 31) * 128 + (((2 * j + (lane >> 5)) ^ (lane & 7)) << 4)) =
                    v4i{acc[u].a[4 * j], acc[u].a[4 * j + 1], acc[u].a[4 * j + 2], acc[u].a[4 * j + 3]};
        }
    WAIT_LDS();
}

__device__ __forceinline__ void write_out(const u32x4 (&t)[4], float scale, char* obase, int lane) {
    const unsigned off = (lane >> 3) * (NS * 4) + (lane & 7) * 16;
#pragma unroll
    for (int i = 0; i < 4; ++i) {
        f32x4 v;
#pragma unroll
        for (int e = 0; e < 4; ++e) v[e] = (float)(int)(t[i][e] - BIAS) * scale;
        store_out(obase + i * 8 * (NS * 4), off, __builtin_bit_cast(u32x4, v));
    }
}

__device__ __forceinline__ unsigned send_subtile(const u32x4 (&t)[4], char* base, int n, int lane, int pace, char* go, unsigned call) {
    const unsigned off = lane * 16;
    char* const dst = base + n * SHORT_BYTES;
    u32x4 lo[2], hi[2];
#pragma unroll
    for (int i = 0; i < 2; ++i)
        lo[i] = u32x4{__builtin_amdgcn_perm(t[2 * i][1], t[2 * i][0], 0x05040100u), __builtin_amdgcn_perm(t[2 * i][3], t[2 * i][2], 0x05040100u),
                      __builtin_amdgcn_perm(t[2 * i + 1][1], t[2 * i + 1][0], 0x05040100u), __builtin_amdgcn_perm(t[2 * i + 1][3], t[2 * i + 1][2], 0x05040100u)};
    if (pace == 2) WAIT_VM(PUSH_B);
    store_sys(dst, off, lo[0]);
    if (pace == 2) WAIT_VM(PUSH_B);
    store_sys(dst + 1024, off, lo[1]);
#pragma unroll
    for (int i = 0; i < 2; ++i)
        hi[i] = u32x4{__builtin_amdgcn_perm(t[2 * i][1], t[2 * i][0], 0x07060302u), __builtin_amdgcn_perm(t[2 * i][3], t[2 * i][2], 0x07060302u),
                      __builtin_amdgcn_perm(t[2 * i + 1][1], t[2 * i + 1][0], 0x07060302u), __builtin_amdgcn_perm(t[2 * i + 1][3], t[2 * i + 1][2], 0x07060302u)};
    const u32x4 o = hi[0] | hi[1];
    if (__any(((o[0] | o[1] | o[2] | o[3]) & 0xffc0ffc0u) != 0)) {
        if (go && lane == 0) set_flag(go, call, 0);
        store_sys(base + Y_LONG + n * 2048, off, hi[0]);
        store_sys(base + Y_LONG + n * 2048 + 1024, off, hi[1]);
        return 1u;
    }
    unsigned b[4];
#pragma unroll
    for (int i = 0; i < 4; ++i) b[i] = __builtin_amdgcn_perm(hi[i >> 1][(i & 1) * 2 + 1], hi[i >> 1][(i & 1) * 2], 0x06040200u);
    const unsigned h0 = b[0] | ((b[3] << 6) & 0xc0c0c0c0u), h1 = b[1] | ((b[3] << 4) & 0xc0c0c0c0u), h2 = b[2] | ((b[3] << 2) & 0xc0c0c0c0u);
    auto s1 = __builtin_amdgcn_permlane16_swap(h0, h1, false, false);
    auto s2 = __builtin_amdgcn_permlane16_swap(h2, h2, false, false);
    auto s3 = __builtin_amdgcn_permlane32_swap(s1[1], s2[1], false, false);
    const u32x4 h = {h0, h1, h2, s3[1]};
    if (pace == 1) WAIT_VM(PUSH_A); else if (pace == 2) WAIT_VM(PUSH_B);
    if (go && lane == 0) set_flag(go, call, 0);
    if (lane < 48) store_sys(dst + 2048, off, h);
    return 0u;
}

constexpr unsigned READY = 1u << 31, FAILED = 1u << 30;

template <int D>
__device__ __forceinline__ void fetch_subtile(const char* base, int n, unsigned buf, bool raw, int lane) {
    const char* const src = base + n * SHORT_BYTES;
    WAIT_VM(D);
    glds16_sys(src, lane * 16, buf);
    WAIT_VM(D);
    glds16_sys(src + 1024, lane * 16, buf + 1024);
    WAIT_VM(D);
    glds16_sys(raw ? base + Y_LONG + n * 2048 : src + 2048, min(lane, raw ? 63 : 47) * 16, buf + 2048);
    if (raw) glds16_sys(base + Y_LONG + n * 2048 + 1024, lane * 16, buf + 3072);
}
template <int H>
__device__ __forceinline__ void finish_half(char* smem, unsigned buf, bool raw, float scale, char* obase, int lane) {
    const u32x4 lo = *(const u32x4*)(smem + buf + H * 1024 + lane * 16);
    const unsigned off = (lane >> 3) * (NS * 4) + (lane & 7) * 16;
    f32x4 v[2];
    if (raw) {
        const u32x4 hi = *(const u32x4*)(smem + buf + 2048 + H * 1024 + lane * 16);
#pragma unroll
        for (int e = 0; e < 4; ++e) {
            v[e >> 1][(e & 1) * 2] = (float)(int)(__builtin_amdgcn_perm(hi[e], lo[e], 0x05040100u) - BIAS) * scale;
            v[e >> 1][(e & 1) * 2 + 1] = (float)(int)(__builtin_amdgcn_perm(hi[e], lo[e], 0x07060302u) - BIAS) * scale;
        }
    } else {
        const unsigned ha = buf + 2048 + (lane < 48 ? lane * 16 : (lane - 48) * 16 + 12), hs = lane < 48 ? 4 : 256;
        const unsigned h0 = *(const unsigned*)(smem + ha), h1 = *(const unsigned*)(smem + ha + hs);
        unsigned b[2];
        if (H == 0) {
            b[0] = h0 & 0x3f3f3f3fu; b[1] = h1 & 0x3f3f3f3fu;
        } else {
            const unsigned h2 = *(const unsigned*)(smem + ha + 2 * hs);
            b[0] = h2 & 0x3f3f3f3fu;
            b[1] = ((h0 >> 6) & 0x03030303u) | ((h1 >> 4) & 0x0c0c0c0cu) | ((h2 >> 2) & 0x30303030u);
        }
#pragma unroll
        for (int i = 0; i < 2; ++i) {
            v[i][0] = ((float)__builtin_amdgcn_perm(b[i], lo[2 * i], 0x0c040100u) - (float)BIAS) * scale;
            v[i][1] = ((float)__builtin_amdgcn_perm(b[i], lo[2 * i], 0x0c050302u) - (float)BIAS) * scale;
            v[i][2] = ((float)__builtin_amdgcn_perm(b[i], lo[2 * i + 1], 0x0c060100u) - (float)BIAS) * scale;
            v[i][3] = ((float)__builtin_amdgcn_perm(b[i], lo[2 * i + 1], 0x0c070302u) - (float)BIAS) * scale;
        }
    }
    store_out(obase + (2 * H) * 8 * (NS * 4), off, __builtin_bit_cast(u32x4, v[0]));
    store_out(obase + (2 * H + 1) * 8 * (NS * 4), off, __builtin_bit_cast(u32x4, v[1]));
}

__device__ __forceinline__ void poll_flags(const char* flag, bool active, LDS_AS unsigned* word, char* tmo_, unsigned call) {
    GLB_AS unsigned* tm = (GLB_AS unsigned*)tmo_;
    const u64 t0 = __builtin_amdgcn_s_memrealtime();
    bool waiting = active;
    while (true) {
        const u64 v = __hip_atomic_load((GLB_AS u64*)flag, __ATOMIC_RELAXED, __HIP_MEMORY_SCOPE_SYSTEM);
        if (waiting && (unsigned)(v >> 32) >= call) {
            __hip_atomic_store(word, ((unsigned)v & 0xffffu) | READY, __ATOMIC_RELEASE, __HIP_MEMORY_SCOPE_WORKGROUP);
            waiting = false;
        }
        if (!__any(waiting)) break;
        const u64 dt = __builtin_amdgcn_s_memrealtime() - t0;
        if (dt > 10000 && (dt > DEADLINE_TICKS || __hip_atomic_load(tm, __ATOMIC_RELAXED, __HIP_MEMORY_SCOPE_AGENT) == call)) {
            __hip_atomic_store(tm, call, __ATOMIC_RELAXED, __HIP_MEMORY_SCOPE_AGENT);
            if (waiting) __hip_atomic_store(word, READY | FAILED, __ATOMIC_RELEASE, __HIP_MEMORY_SCOPE_WORKGROUP);
            break;
        }
        __builtin_amdgcn_s_sleep(POLL_SLEEP);
    }
}

template <int PER, int NW, int STEP, int INFL = 63>
__device__ __forceinline__ void warm_l2(const char* wblk, const char* xf, int me, int lane) {
    const unsigned wo = (lane >> 2) * ND + (lane & 3) * 128;
#pragma unroll 1
    for (int n = me; n < 32 * PER; n += STEP) {
        const int kb = n / PER, j = n - kb * PER;
        const bool isw = j < NW;
        if (INFL < 63) WAIT_VM(INFL);
        glds4(isw ? wblk + (size_t)(kb * 128 + j * 16) * ND : xf + (size_t)(j - NW) * 64 * KD + kb * 128, isw ? wo : (unsigned)lane * KD);
    }
}

__device__ __forceinline__ void warm_remote(const char* wblk, const char* x, int me, int lane) {
    constexpr int NWARM = __builtin_popcount(PF_MASK) * 8 + (4 - AGG) * 2;
    if (PF_NX == 0) {
        warm_l2<8 + (PF_XU + 1) / 2, 8, NWARM>(wblk, x + AGU * 32 * KD, me, lane);
        if (PF2_XU && (unsigned)(me - PF2_LO * 2) < PF2_N * 2) warm_l2<PF2_XU ? PF2_XU / 2 : 1, 0, PF2_N * 2>(wblk, x + (AGU + PF2_FIRST) * 32 * KD, me - PF2_LO * 2, lane);
    } else if (me >= 2 * PF_NX) {
        warm_l2<8, 8, NWARM - 2 * PF_NX>(wblk, x, me - 2 * PF_NX, lane);
    } else {
        warm_l2<(PF_XU + 1) / 2, 0, PF_NX ? 2 * PF_NX : 1>(wblk, x + AGU * 32 * KD, me, lane);
        if (PF2_XU && me < PF2_N * 2) warm_l2<PF2_XU ? PF2_XU / 2 : 1, 0, PF2_N * 2>(wblk, x + (AGU + PF2_FIRST) * 32 * KD, me, lane);
    }
}
__device__ __forceinline__ void advance(LDS_AS unsigned* ctl, char* counter, unsigned call, int lane) {
    asm volatile("" :: "s"(call) : "memory");
    if (lane == 0 && __hip_atomic_fetch_add(ctl + 12, 1u, __ATOMIC_RELAXED, __HIP_MEMORY_SCOPE_WORKGROUP) == 7)
        __hip_atomic_store((GLB_AS unsigned*)counter, call, __ATOMIC_RELAXED, __HIP_MEMORY_SCOPE_AGENT);
}

typedef const CST_AS Group* KArgs;
#define FRESH() KArgs kp = kp0; int b = b0, wid = wid0, lane = lane0; \
    asm volatile("" : "+s"(kp), "+s"(b), "+s"(wid) :: "memory"); asm volatile("" : "+v"(lane))
#define WLOC() (kp->wblk + (kp->rank - kp->peer) * NS)
#define CALL() (*(const CST_AS unsigned*)(kp->myws + WS_CNT + b * 128) + 1u)
#define SCALE() (*(const CST_AS float*)kp->sx * *(const CST_AS float*)kp->sw)

__global__ void __launch_bounds__(BLOCK, 2) fused_kernel(Params) {
    extern __shared__ __attribute__((aligned(16))) char smem[];
    const KArgs kp = (KArgs)__builtin_amdgcn_kernarg_segment_ptr() + (blockIdx.x & 7), kp0 = kp;
    const int lane = threadIdx.x & 63, wid = __builtin_amdgcn_readfirstlane(threadIdx.x >> 6), b = blockIdx.x;
    const int lane0 = lane, wid0 = wid, b0 = b;
    const char *x0 = kp->x, *w0 = kp->wblk;
    char* ws0 = kp->myws;
    const u64 tstart = __builtin_amdgcn_s_memrealtime();
    PIN();
#define grp (b & 7)
#define cls (b >> 5)
#define pan ((b >> 3) & 3)
#define local ((b & 7) == LOCAL_GRP)
#define NIB(word) (((unsigned)(word) >> (4 * cls)) & 15)
    LDS_AS unsigned* ctl = (LDS_AS unsigned*)(smem + CTL_OFF);
    const bool link = cls == 0 && !local;

    if (link) {
        constexpr int PPU = 32 / AGG, NP = AGU * PPU;
        static_assert(AGD >= 2 && AGD <= 8 && AGD <= PPU && (AGG == 2 || AGG == 4), "staging ring");
        const char *x = x0, *sxp = kp->sx, *swp = kp->sw;
        char *myws = ws0, *peer = kp->peerws;
        unsigned rank = kp->rank, src = kp->peer;
        const bool ag = pan < AGG;
        const unsigned agoff = (pan * 4 + (wid & 3)) * 1024 + lane * 16;
        asm volatile("" : "+s"(x), "+s"(sxp), "+s"(swp), "+s"(myws), "+s"(peer), "+s"(rank), "+s"(src));
        unsigned done = *(const CST_AS unsigned*)(myws + WS_CNT + b * 128);
        float sx = *(const CST_AS float*)sxp, sw = *(const CST_AS float*)swp;
        PIN();
        if (ag && wid < 4) {
#pragma unroll
            for (int i = 0; i < AGD; ++i) AGLD(x + i * (AGG * 4096), agoff, (wid * 8 + i) * 1024);
        }
        if (wid == 7) {
            ctl[lane] = 0;
            ctl[64 + lane] = 0;
            WAIT_LDS();
        }
        BARRIER();
        if (!ag && (wid >> 1) == 2) warm_remote(kp->wblk, x, (pan - AGG) * 2 + (wid & 1), lane);
        asm volatile("" : "+s"(done), "+s"(sx), "+s"(sw));
        const unsigned call = done + 1, par = call & 1;
        const float scale = sx * sw;
        char* const din = peer + WS_IN + (par * NR + rank) * (unsigned)IN_BYTES;
        STAMP(1);
        if (ag && wid < 4) {
            int r = 0;
#pragma unroll 1
            for (int i = 0; i < NP; ++i) {
                if (i >= AGD - 1) WAIT_VM(2 * AGD - 2); else WAIT_VM(AGD - 1);
                if (i >= AGD && ((i - AGD) & (PPU - 1)) == PPU - 1 && lane == 0)
                    __hip_atomic_fetch_add(ctl + 8 + (i - AGD) / PPU, 1u, __ATOMIC_RELAXED, __HIP_MEMORY_SCOPE_WORKGROUP);
                const u32x4 v = *(const u32x4*)(smem + (wid * 8 + r) * 1024 + lane * 16);
                store_sys(din + IN_X + i * (AGG * 4096), agoff, v);
                AGLD(x + (i + AGD < NP ? i + AGD : 0) * (AGG * 4096), agoff, (wid * 8 + r) * 1024);
                r = r == AGD - 1 ? 0 : r + 1;
            }
            WAIT_VM(0);
            if (lane == 0) __hip_atomic_fetch_add(ctl + 8 + AGU - 1, 1u, __ATOMIC_RELAXED, __HIP_MEMORY_SCOPE_WORKGROUP);
            STAMP(3);
        }
        advance(ctl, myws + WS_CNT + b * 128, call, lane);
        constexpr int NPULL = PULL_ORDER >> 12 ? 4 : (PULL_ORDER >> 8 ? 3 : (PULL_ORDER >> 4 ? 2 : 1));
        constexpr unsigned PULL_BUF = 0;
        constexpr int NPU = pull_units();
        static_assert(PULL_BUF + NPU * 16384 <= CTL_OFF && PULL_D <= 5, "pull buffers");
        LDS_AS unsigned* landed = ctl + 16;
        static_assert(NPULL < 4, "lanes 12..15 watch the go-ahead");
        constexpr int N0 = 4 * ((R_CNT >> 4 * (PULL_ORDER & 15)) & 15), N1 = N0 + 3 * (NPU - N0 / 4);
        int tp = 3;
        for (int p = 2; p >= 0; --p) if (tight(WLOC() + p * 128)) tp = p;
        auto entry = [&](int m, int& p, int& q) {
            int j;
            q = pan;
            if (!SPREAD) { j = m >> 2; p = pan; q = m & 3; }
            else if (m < N0) { j = m >> 2; p = m & 3; }
            else if (m < N1) { j = (m - N0) / 3; p = m - N0 - 3 * j; p += p >= tp; j += N0 / 4; }
            else { j = m - N1 + N0 / 4; p = tp; }
            return (int)(pull_table() >> (8 * j)) & 255;
        };
        LDS_AS u64* mirror = (LDS_AS u64*)(ctl + (MIR_OFF - CTL_OFF) / 4);
        auto flag_word = [&](int i) {
            const u64 v = __hip_atomic_load(mirror + i * 2, __ATOMIC_ACQUIRE, __HIP_MEMORY_SCOPE_WORKGROUP);
            const unsigned fail = __hip_atomic_load(ctl + 14, __ATOMIC_ACQUIRE, __HIP_MEMORY_SCOPE_WORKGROUP);
            const bool ok = (unsigned)(v >> 32) >= call && ((unsigned)v >> 16) == (call & 0xffffu);
            return (ok ? ((unsigned)v & 0xffffu) | READY : 0u) | fail;
        };
        if (wid == 7) {
            for (int u = 0; ag && u < AGU; ++u) {
                while (__hip_atomic_load(ctl + 8 + u, __ATOMIC_RELAXED, __HIP_MEMORY_SCOPE_WORKGROUP) < 4) __builtin_amdgcn_s_sleep(2);
                if (lane == 0) set_flag(din + IN_XFLAG + (u * 4 + pan) * 128, call, 0);
            }
        } else if (wid == 6) {
            const char* const flags = myws + WS_IN + (par * NR + src) * (unsigned)IN_BYTES + IN_FLAG;
            const unsigned fo = (((PULL_ORDER >> (lane & 12)) & 15) * 4 + (lane & 3)) * FSP;
            const bool watch = (lane & 15) < NPULL * 4 && (SPREAD || (lane & 3) == pan);
            GLB_AS unsigned* tm = (GLB_AS unsigned*)(myws + WS_TMO);
            while (__hip_atomic_load(ctl + 12, __ATOMIC_RELAXED, __HIP_MEMORY_SCOPE_WORKGROUP) < 8) __builtin_amdgcn_s_sleep(4);
            const u64 t0 = __builtin_amdgcn_s_memrealtime();
            while (true) {
                WAIT_VM(7);
                if (lane < 16) glds16_sys(flags, fo, MIR_OFF);
                __builtin_amdgcn_s_sleep(MIR_SLEEP);
                if (!__any((watch || (GO_CLS && (lane & 15) == 12 + pan)) && !(flag_word(lane & 15) & READY))) break;
                const u64 dt = __builtin_amdgcn_s_memrealtime() - t0;
                if (dt > 10000 && (dt > DEADLINE_TICKS || __hip_atomic_load(tm, __ATOMIC_RELAXED, __HIP_MEMORY_SCOPE_AGENT) == call)) {
                    __hip_atomic_store(tm, call, __ATOMIC_RELAXED, __HIP_MEMORY_SCOPE_AGENT);
                    __hip_atomic_store(ctl + 14, READY | FAILED, __ATOMIC_RELEASE, __HIP_MEMORY_SCOPE_WORKGROUP);
                    break;
                }
            }
        }
        int ep, eq;
        const int ee = entry(min(lane, NPU * 4 - 1), ep, eq);
        const unsigned eout = ((ee & 15) * 32 * NS + ep * 128 + eq * 32) * 4;
        char* const out = kp->out + (size_t)src * MS * NS * 4;
        constexpr int LAST = (NPU - 1) * 4;
        if (wid < 4) {
            const char* const sout = peer + WS_OUT + (par * NR + rank) * (unsigned)OUT_BYTES;
            const int en = ((ee & 15) * 4 + ep) * 4 + eq, ef = (ee >> 6) * 4 + ep, eb = ((ee >> 4) & 3) * 4 + eq;
            if (GO_CLS) while (!(flag_word(12 + pan) & READY)) __builtin_amdgcn_s_sleep(1);
            unsigned words = 0, marks = 0;
#pragma unroll 1
            for (int j = 0; j < NPU; ++j) {
                const int m = j * 4 + wid;
                unsigned word = __builtin_amdgcn_readlane(words, m);
                for (int k = 0; !(word & READY); ++k) {
                    if (k == 1) WAIT_VM(2);
                    else if (k == 2) WAIT_VM(1);
                    else if (k == 3) {
                        WAIT_VM(0);
                        if (lane == 0) __hip_atomic_store(landed + wid, marks | j, __ATOMIC_RELEASE, __HIP_MEMORY_SCOPE_WORKGROUP);
                    } else if (k) __builtin_amdgcn_s_sleep(1);
                    const unsigned w = flag_word(ef);
                    words = (w & (READY | FAILED)) | ((w >> eb) & 1);
                    word = __builtin_amdgcn_readlane(words, m);
                }
                marks |= ((word & 1) << 8 | (word & FAILED ? 1u << 16 : 0u)) << j;
                if (word & FAILED) WAIT_VM(0);
                else fetch_subtile<PULL_D>(sout, __builtin_amdgcn_readlane(en, m), PULL_BUF + m * 4096, word & 1, lane);
                static_assert(PULL_D <= 2 && NPU <= 8, "landed count");
                if (lane == 0) __hip_atomic_store(landed + wid, marks | j, __ATOMIC_RELEASE, __HIP_MEMORY_SCOPE_WORKGROUP);
            }
            char* o = out + __builtin_amdgcn_readlane(eout, LAST + wid);
            float sc = (marks >> (15 + NPU)) & 1 ? __builtin_nanf("") : scale;
            asm volatile("" : "+s"(o));
            asm volatile("" : "+v"(sc));
            WAIT_VM(0);
            if (lane == 0) __hip_atomic_store(landed + wid, marks | NPU, __ATOMIC_RELEASE, __HIP_MEMORY_SCOPE_WORKGROUP);
            finish_half<0>(smem, PULL_BUF + (LAST + wid) * 4096, (marks >> (7 + NPU)) & 1, sc, o, lane);
        } else {
            for (bool more = true; more;) {
                unsigned m = 0;
                if (lane == 0) m = __hip_atomic_fetch_add(ctl + 13, 1u, __ATOMIC_RELAXED, __HIP_MEMORY_SCOPE_WORKGROUP);
                m = __builtin_amdgcn_readfirstlane(m);
                more = m < LAST;
                if (!more) m = LAST + wid - 4;
                char* o = out + __builtin_amdgcn_readlane(eout, m);
                asm volatile("" : "+s"(o));
                unsigned w;
                while (((w = __builtin_amdgcn_readfirstlane(__hip_atomic_load(landed + (m & 3), __ATOMIC_ACQUIRE, __HIP_MEMORY_SCOPE_WORKGROUP))) & 255) <= m >> 2)
                    if (more) __builtin_amdgcn_s_sleep(1);
                w >>= m >> 2;
                const float sc = (w >> 16) & 1 ? __builtin_nanf("") : scale;
                finish_half<1>(smem, PULL_BUF + m * 4096, (w >> 8) & 1, sc, o, lane);
                if (more) finish_half<0>(smem, PULL_BUF + m * 4096, (w >> 8) & 1, sc, o, lane);
            }
        }
    } else {
        int cnt = NIB(local ? L_CNT : R_CNT);
        const char* const wp = w0;
        const char* const xp = x0;
        if (wid == 7) {
            ctl[lane] = 0;
            WAIT_LDS();
        }
        if (((local ? LPF_MASK : PF_MASK) >> cls) & 1) {
            if (wid == 6) asm volatile("" :: "s"(CALL()), "v"(SCALE()));
            if ((wid >> 1) == 2) {
                const int me = (__builtin_popcount((local ? LPF_MASK : PF_MASK) & ((1 << cls) - 1)) * 4 + pan) * 2 + (wid & 1);
                if (local) {
                    warm_l2<8, 0, __builtin_popcount(LPF_MASK) * 8>(wp, xp, me, lane);
                    if (LW_START) while (__builtin_amdgcn_s_memrealtime() - tstart < LW_START) __builtin_amdgcn_s_sleep(4);
                    warm_l2<8, 8, __builtin_popcount(LPF_MASK) * 8>(wp, xp, me, lane);
                } else {
                    warm_remote(wp, xp, (4 - AGG) * 2 + me, lane);
                    if (cls == WL_CLS) {
                        const char* wl = WLOC();
#pragma unroll 1
                        for (int p = 0; p < 4; ++p)
                            if (tight(wl + p * 128))
#pragma unroll 1
                                for (int n = pan * 2 + (wid & 1); n < 64; n += 8) glds4(wl + p * 128 + (size_t)n * 64 * ND, lane * ND);
                    }
                }
                if (WARM_DRAIN) WAIT_VM(0);
            }
            BARRIER();
        }
        if (NIB(local ? L_START : START))
            while (__builtin_amdgcn_s_memrealtime() - tstart < (u64)(NIB(local ? L_START : START) * 64)) __builtin_amdgcn_s_sleep(2);
        int task = cnt ? 0 : 1;
        const int ntask = !local && NIB(A_CNT) ? 2 : 1;
#pragma unroll 1
        for (; task < ntask; ++task) {
            unsigned bad = 0, afirst = local ? NIB(L_FIRST) : AGU + NIB(R_FIRST);
            const char *wp1 = nullptr, *xp1 = nullptr;
            if (task == 1) {
                BARRIER();
                FRESH();
                const unsigned call = CALL();
                const char* sin = kp->myws + WS_IN + (size_t)((call & 1) * NR + kp->peer) * IN_BYTES;
                wp1 = WLOC();
                const bool tp = tight(wp1 + pan * 128);
                cnt = NIB(tp ? T_CNT : A_CNT);
                afirst = NIB(tp ? T_FIRST : A_FIRST);
                if (A_START && !tp && wid == 1) while (__builtin_amdgcn_s_memrealtime() - tstart < A_START) __builtin_amdgcn_s_sleep(4);
                if (wid == 1) poll_flags(sin + IN_XFLAG + ((afirst + cnt - 1) * 4 + (lane & 3)) * 128, lane < AGG,
                                         ctl + 4 + (lane & 3), kp->myws + WS_TMO, call);
                WAIT_LDS();
                BARRIER();
                bad = __builtin_amdgcn_readfirstlane((ctl[4] | ctl[5] | ctl[6] | ctl[7]) & FAILED);
                xp1 = sin + IN_X;
                STAMP(4);
            }
            if (wid < 4 && !bad) {
                FRESH();
                unsigned woff = w_offset(lane, wid) + pan * 128, xoff = x_offset(lane, wid) + afirst * 32 * KD;
                asm volatile("" : "+v"(woff), "+v"(xoff));
                PIN();
                load_tile_n(task ? wp1 : w0, task ? xp1 : x0, cnt, woff, xoff, wid);
            }
            FRESH();
            const bool direct = local || task == 1, push = (R_PUSH >> cls) & 1;
            char *myws = kp->myws, *out = kp->out, *din = kp->peerws;
            const char *sxp = kp->sx, *swp = kp->sw;
            unsigned rank = kp->rank, src = kp->peer, dst = kp->peer;
            asm volatile("" : "+s"(myws), "+s"(out), "+s"(din), "+s"(sxp), "+s"(swp), "+s"(rank), "+s"(src), "+s"(dst));
            const unsigned done = *(const CST_AS unsigned*)(myws + WS_CNT + b * 128);
            const float sx = *(const CST_AS float*)sxp, sw = *(const CST_AS float*)swp;
            din += WS_IN + rank * (unsigned)IN_BYTES;
            char* base = direct ? out + (((task == 1 ? src : rank) * 16 + afirst) * 32 * NS + pan * 128) * 4
                       : push ? din : myws + WS_OUT + dst * (unsigned)OUT_BYTES;
            unsigned odd = direct ? 0u : push ? (unsigned)(NR * IN_BYTES) : (unsigned)(NR * OUT_BYTES);
            asm volatile("" : "+s"(base), "+s"(din), "+s"(odd));
            if (wid >= 4 && !bad) {
                if (cnt == 1) tile_compute<1>(smem, lane, wid - 4, done, sx, sw);
                else if (cnt == 2) tile_compute<2>(smem, lane, wid - 4, done, sx, sw);
                else tile_compute<3>(smem, lane, wid - 4, done, sx, sw);
            }
            if (cnt > 1) BARRIER();
            const unsigned call = done + 1;
            const float scale = bad ? __builtin_nanf("") : sx * sw;
            if (call & 1) { base += odd; din += NR * IN_BYTES; }
            unsigned rawbits = 0;
#pragma unroll 1
            for (int i = wid ^ 4; i < cnt * 4; i += 8) {
                u32x4 t[4];
#pragma unroll
                for (int j = 0; j < 4; ++j)
                    t[j] = *(const u32x4*)(smem + i * 4096 + (8 * j + (lane >> 3)) * 128 + (((lane & 7) ^ (lane >> 3)) << 4));
                if (GO_ITEM >= 8 && !direct && cls == GO_CLS && i == GO_ITEM && lane == 0) set_flag(din + IN_FLAG + pan * FSP, call, 0);
                if (direct) write_out(t, scale, base + (i >> 2) * (32 * NS * 4) + (i & 3) * 128, lane);
                else rawbits |= send_subtile(t, base, ((afirst + (i >> 2)) * 4 + pan) * 4 + (i & 3), lane, push && cnt > 1 ? 1 + (i >> 3) : 0,
                                             GO_ITEM < 8 && cls == GO_CLS && i == GO_ITEM ? din + IN_FLAG + pan * FSP : nullptr, call) << i;
            }
            if (!direct) {
                if (rawbits && lane == 0) __hip_atomic_fetch_or(ctl, rawbits, __ATOMIC_RELAXED, __HIP_MEMORY_SCOPE_WORKGROUP);
                asm volatile("s_waitcnt vmcnt(0) lgkmcnt(0)" ::: "memory");
                BARRIER();
                STAMP(3);
                if (wid == 7 && lane == 0) set_flag(din + IN_FLAG + (b >> 3) * FSP, call, ctl[0]);
            }
        }
        FRESH();
        const unsigned call = CALL();
        advance(ctl, kp->myws + WS_CNT + b * 128, call, lane);
        if (!local) {
            const int src = kp->peer;
            const char* sin = kp->myws + WS_IN + (size_t)((call & 1) * NR + src) * IN_BYTES;
            char* const out = kp->out + ((size_t)src * MS * NS + pan * 128) * 4;
            const float scale = SCALE();
#pragma unroll 1
            for (int c = 1; c < 8; ++c) {
                if (((M_CLS >> (4 * c)) & 15) != cls) continue;
                BARRIER();
                if (wid == 1) poll_flags(sin + IN_FLAG + (c * 4 + pan) * FSP, lane == 0, ctl + 16 + c, kp->myws + WS_TMO, call);
                WAIT_LDS();
                BARRIER();
                const unsigned word = __builtin_amdgcn_readfirstlane(ctl[16 + c]);
                for (int i = wid; i < ((R_CNT >> (4 * c)) & 15) * 4; i += 8) {
                    const int unit = AGU + ((R_FIRST >> (4 * c)) & 15) + (i >> 2), cw = i & 3;
                    const bool raw = (word >> i) & 1, failed = word & FAILED;
                    if (!failed) fetch_subtile<63>(sin, (unit * 4 + pan) * 4 + cw, i * 4096, raw, lane);
                    WAIT_VM(0);
                    finish_half<0>(smem, i * 4096, raw, failed ? __builtin_nanf("") : scale, out + (unit * 32 * NS + cw * 32) * 4, lane);
                    finish_half<1>(smem, i * 4096, raw, failed ? __builtin_nanf("") : scale, out + (unit * 32 * NS + cw * 32) * 4, lane);
                }
            }
        }
    }
}
#undef grp
#undef cls
#undef pan
#undef local

extern "C" void kernel_launch(int rank, int nranks, ncclComm_t comm,
                              hipStream_t stream,
                              void* const* d_in, const int* in_sizes, int n_in,
                              void* d_out, int out_size,
                              const int* in_dtypes, int out_dtype,
                              void* const* d_out_all,
                              void* const* d_ws_all, size_t ws_bytes) {
    if (nranks != NR || (unsigned)rank >= (unsigned)NR || ws_bytes < WS_END) return;
    static bool init[NR] = {};
    if (!init[rank]) {
        HIP_CHECK(hipFuncSetAttribute((const void*)fused_kernel, hipFuncAttributeMaxDynamicSharedMemorySize, LDS_BYTES));
        init[rank] = true;
    }
    Params p;
    int peer[NR];
    {
        const unsigned quad = ((0x4d >> rank) & 1) ? 0x4du : 0xb2u;
        int lo = 0, hi = 4;
        for (int i = 1; i < NR; ++i) {
            const int q = (rank + i) & 7;
            if (lo == LOCAL_GRP) ++lo;
            if (TOPO == 0) { peer[lo++] = q; continue; }
            if ((quad >> q) & 1) peer[lo++] = q; else peer[hi++] = q;
        }
        peer[LOCAL_GRP] = rank;
    }
    for (int g = 0; g < NR; ++g) {
        p.g[g].x = (const char*)d_in[0];
        p.g[g].wblk = (const char*)d_in[1] + peer[g] * NS;
        p.g[g].myws = (char*)d_ws_all[rank];
        p.g[g].peerws = (char*)d_ws_all[peer[g]];
        p.g[g].out = (char*)d_out;
        p.g[g].sx = (const char*)d_in[2];
        p.g[g].sw = (const char*)d_in[3];
        p.g[g].rank = rank;
        p.g[g].peer = peer[g];
    }
    hipLaunchKernelGGL(fused_kernel, dim3(GRID), dim3(BLOCK), LDS_BYTES, stream, p);
    HIP_CHECK(hipGetLastError());
}
```

```cpp
#include <hip/hip_runtime.h>
#include <rccl/rccl.h>
#include <cstdio>

#define HIP_CHECK(expr) do { hipError_t _e = (expr); if (_e != hipSuccess) { \
    fprintf(stderr, "HIP error %s at %s:%d\n", hipGetErrorString(_e), __FILE__, __LINE__); \
    return; } } while (0)

typedef int v2i __attribute__((ext_vector_type(2)));
typedef int v4i __attribute__((ext_vector_type(4)));
typedef int v16i __attribute__((ext_vector_type(16)));
typedef unsigned u32x4 __attribute__((ext_vector_type(4)));
typedef float f32x4 __attribute__((ext_vector_type(4)));
typedef unsigned long long u64;
#define LDS_AS __attribute__((address_space(3)))
#define GLB_AS __attribute__((address_space(1)))
#define CST_AS __attribute__((address_space(4)))

#define AGU 3
#define R_CNT   0x33210220
#define R_FIRST 0x63100b90
#define R_PUSH  0x70
#define A_CNT   0x00012000
#define A_FIRST 0x00020000
#define PF_MASK 0x0e
#define PF_XU   9
#define PF2_CLS 3
#define PF2_FIRST 9
#define PF2_XU 4
#define XW_FIRST 2
#define XW_XU 10
#define XW2_FIRST 12
#define XW2_XU 4
#define PULL_ORDER 0x127
#define L_CNT   0x33330022
#define L_FIRST 0x963000ec
#define START   0x00000000
#define LPF_MASK 0x0f
#define L_START 0x44440000
#define POLL_SLEEP 8
#define LA1 3
#define LA2 3
#define LA3 2
#define XW_INFL 1
#define XW_WAVES 2
#ifndef M_CLS
#define M_CLS 0x06570000
#endif
#define AGG 4
#define PF_NX 0
#define PF2_N 3
#define GO_CLS 6
#define GO_ITEM 7
#define GO_LATE
#define WL_CLS 9
#define AG_NT 0
#define WARM_DRAIN 0
#define AGLD glds16
#define X_POL "sc0 sc1"
#define T_CNT   0x00021000
#define T_FIRST 0x00010000
#define LOCAL_GRP 3
#define TOPO 1
#define FSP 16
#define MIR_SLEEP 12
#define SPREAD 1
#define PF2_LO 8
#define M16 6
#define LW_START 0
#define A_START 880
#define ILV16 0
#define RPM32 2
#define HOLD0 710
#define HOLD1 78
#define AGD 4
#define PUSH_A 1
#define PUSH_B 3
#define PULL_D 2
#define OUT_SC "sc0 sc1"

constexpr int NR = 8, MS = 512, KD = 4096, ND = 4096, NS = 512;
constexpr int GRID = 256, BLOCK = 512;
constexpr int NSLOT = 4;
constexpr int SLOT_BYTES = 32768;
constexpr int CTL_OFF = NSLOT * SLOT_BYTES;
constexpr int MIR_OFF = CTL_OFF + 256;
constexpr int JUNK_OFF = MIR_OFF + 1024;
constexpr int LDS_BYTES = JUNK_OFF + 256;
constexpr int BIAS = 1 << 21;
constexpr u64 DEADLINE_TICKS = 40000000ull;

constexpr size_t WS_CNT = 0;
constexpr size_t WS_TMO = WS_CNT + GRID * 128;
constexpr int SHORT_BYTES = 2816;
constexpr size_t Y_LONG = 256 * SHORT_BYTES;
constexpr size_t Y_BYTES = Y_LONG + 256 * 2048;
constexpr size_t WS_IN = 2 << 20;
constexpr size_t IN_BYTES = 2 << 20;
constexpr size_t IN_X = Y_BYTES;
constexpr size_t IN_FLAG = IN_X + 128 * KD;
constexpr size_t IN_XFLAG = IN_FLAG + 32 * 128;
constexpr size_t WS_OUT = WS_IN + 2 * NR * IN_BYTES;
constexpr size_t OUT_BYTES = Y_BYTES;
constexpr size_t WS_END = WS_OUT + 2 * NR * OUT_BYTES;
static_assert(IN_XFLAG + 16 * 128 <= IN_BYTES, "input block overflow");
static_assert(AGU + (PF_XU + 1) / 2 * 2 <= 16 && AGU + PF2_FIRST + PF2_XU <= 16 && PF2_XU % 2 == 0, "warm-up rows out of range");
constexpr bool plan_ok() {
    unsigned r = 0, a = 0, l = 0, tt = 0;
    for (int c = 0; c < 8; ++c) {
        const int rc = (R_CNT >> 4 * c) & 15, ac = (A_CNT >> 4 * c) & 15, lc = (L_CNT >> 4 * c) & 15;
        if (rc > 3 || ac > 2 || lc > 3) return false;
        if ((c && !rc && !ac && !((PF_MASK >> c) & 1)) || (!lc && !((LPF_MASK >> c) & 1))) return false;
        r += ((1u << rc) - 1) << ((R_FIRST >> 4 * c) & 15);
        a += ((1u << ac) - 1) << ((A_FIRST >> 4 * c) & 15);
        const int tc = (T_CNT >> 4 * c) & 15;
        if (tc > 2 || !tc != !ac) return false;
        tt += ((1u << tc) - 1) << ((T_FIRST >> 4 * c) & 15);
        l += ((1u << lc) - 1) << (((unsigned)L_FIRST >> 4 * c) & 15);
        bool pulled = false;
        for (int t = 0; t < 4; ++t) pulled |= c && ((PULL_ORDER >> 4 * t) & 15) == c;
        if (rc && pulled == (bool)((R_PUSH >> c) & 1)) return false;
        if (!rc && (pulled || ((R_PUSH >> c) & 1))) return false;
        if ((((M_CLS >> 4 * c) & 15) != 0) != (bool)((R_PUSH >> c) & 1) || ((M_CLS >> 4 * c) & 15) > 7) return false;
    }
    return r == (1u << (16 - AGU)) - 1 && a == (1u << AGU) - 1 && tt == a && l == 0xffffu && !(R_CNT & 15) && !(A_CNT & 15) && !(PF_MASK & 1);
}
static_assert(XW_XU % 2 == 0 && XW2_XU % 2 == 0 && XW_FIRST + XW_XU <= 16 && XW2_FIRST + XW2_XU <= 16, "warm-up rows out of range");
static_assert(plan_ok(), "the tiles must cover every unit once");
static_assert(AGU >= 1 && AGU <= 4, "gathered rows");
#define STAMP(i) do { } while (0)

constexpr int pull_units() {
    int n = 0;
    for (int t = 0; t < 4; ++t) n += (R_CNT >> 4 * ((PULL_ORDER >> 4 * t) & 15)) & 15;
    return n;
}
constexpr u64 pull_table() {
    u64 v = 0; int n = 0;
    for (int t = 0; t < 4; ++t) {
        const int c = (PULL_ORDER >> 4 * t) & 15;
        for (int u = 0; u < ((R_CNT >> 4 * c) & 15); ++u) v |= (u64)(t << 6 | u << 4 | (AGU + ((R_FIRST >> 4 * c) & 15) + u)) << 8 * n++;
    }
    return v;
}

struct Group {
    const char* x;
    const char* wblk;
    char* myws;
    char* peerws;
    char* out;
    const char* sx;
    const char* sw;
    int rank;
    int peer;
};
struct Params { Group g[NR]; };
static_assert(sizeof(Group) == 64, "one line");

#define WAIT_LDS() __builtin_amdgcn_s_waitcnt(0xC07F)
#define WAIT_VM(n) asm volatile("s_waitcnt vmcnt(%0)" :: "n"(n) : "memory")
#define PIN() __builtin_amdgcn_sched_barrier(0)
#define BARRIER() do { asm volatile("" ::: "memory"); __builtin_amdgcn_s_barrier(); asm volatile("" ::: "memory"); } while (0)

__device__ __forceinline__ void glds16(const char* sbase, unsigned voff, unsigned lds) {
    asm volatile("s_mov_b32 m0, %2\n\ts_nop 0\n\tglobal_load_lds_dwordx4 %0, %1" :: "v"(voff), "s"(sbase), "s"(lds) : "memory");
}
__device__ __forceinline__ void glds16_sys(const char* sbase, unsigned voff, unsigned lds) {
    asm volatile("s_mov_b32 m0, %2\n\ts_nop 0\n\tglobal_load_lds_dwordx4 %0, %1 sc0 sc1" :: "v"(voff), "s"(sbase), "s"(lds) : "memory");
}
__device__ __forceinline__ void glds4(const char* sbase, unsigned voff) {
    asm volatile("s_mov_b32 m0, %2\n\ts_nop 0\n\tglobal_load_lds_dword %0, %1" :: "v"(voff), "s"(sbase), "s"(JUNK_OFF) : "memory");
}
__device__ __forceinline__ void store_sys(char* sbase, unsigned voff, u32x4 v) {
    asm volatile("global_store_dwordx4 %0, %1, %2 sc0 sc1\n\ts_nop 1" :: "v"(voff), "v"(v), "s"(sbase) : "memory");
}
__device__ __forceinline__ void store_out(char* sbase, unsigned voff, u32x4 v) {
    asm volatile("global_store_dwordx4 %0, %1, %2 " OUT_SC "\n\ts_nop 1" :: "v"(voff), "v"(v), "s"(sbase) : "memory");
}
__device__ __forceinline__ void set_flag(char* flag, unsigned call, unsigned bits) {
    __hip_atomic_store((GLB_AS u64*)flag, ((u64)call << 32) | (call << 16) | bits, __ATOMIC_RELAXED, __HIP_MEMORY_SCOPE_SYSTEM);
}

__device__ __forceinline__ void glds16_x(const char* sbase, unsigned voff, unsigned lds) {
    asm volatile("s_mov_b32 m0, %2\n\ts_nop 0\n\tglobal_load_lds_dwordx4 %0, %1 " X_POL :: "v"(voff), "s"(sbase), "s"(lds) : "memory");
}
__device__ __forceinline__ void load_stage(const char* wp, const char* xp, int cnt, unsigned woff, unsigned xoff, unsigned wslot, unsigned xslot) {
#pragma unroll
    for (int j = 0; j < 4; ++j) glds16(wp + j * 8 * ND, j >= 2 ? woff ^ 16 : woff, wslot + j * 1024);
    glds16_x(xp, xoff, xslot);
    if (cnt > 1) glds16_x(xp + 32 * KD, xoff, xslot + 4096);
    if (cnt > 2) glds16_x(xp + 64 * KD, xoff, xslot + 8192);
}
__device__ __forceinline__ unsigned w_offset(int lane, int wid) {
    return (unsigned)(wid * 32 + (lane >> 3)) * ND + (((lane & 7) ^ (((lane >> 4) & 3) << 1)) << 4);
}
__device__ __forceinline__ unsigned x_offset(int lane, int wid) {
    return (unsigned)(wid * 8 + (lane >> 3)) * KD + (((lane & 7) ^ ((wid * 4 + (lane >> 4)) & 7)) << 4);
}
template <int U, int LA>
__device__ __forceinline__ void load_tile(const char* wp, const char* xp, unsigned woff, unsigned xoff, int wid) {
    static_assert(LA < NSLOT, "ring too small");
    unsigned slot = 0;
#pragma unroll 1
    for (int s = 0; s < 32; ++s) {
        if (s >= LA) {
            WAIT_VM((LA - 1) * (4 + U));
            BARRIER();
        }
        load_stage(wp + (size_t)s * 128 * ND, xp + s * 128, U, woff, xoff, slot + wid * 4096, slot + 16384 + wid * 1024);
        slot = (slot + SLOT_BYTES) & (NSLOT * SLOT_BYTES - 1);
    }
#pragma unroll
    for (int r = LA - 1; r >= 0; --r) {
        WAIT_VM(r * (4 + U));
        BARRIER();
    }
}
__device__ __forceinline__ void load_tile_n(const char* wp, const char* xp, int cnt, unsigned woff, unsigned xoff, int wid) {
    if (cnt == 1) load_tile<1, LA1>(wp, xp, woff, xoff, wid);
    else if (cnt == 2) load_tile<2, LA2>(wp, xp, woff, xoff, wid);
    else load_tile<3, LA3>(wp, xp, woff, xoff, wid);
}

__device__ __forceinline__ void glds16_nt(const char* sbase, unsigned voff, unsigned lds) {
    asm volatile("s_mov_b32 m0, %2\n\ts_nop 0\n\tglobal_load_lds_dwordx4 %0, %1 nt" :: "v"(voff), "s"(sbase), "s"(lds) : "memory");
}
__device__ __forceinline__ bool tight(const char* p) { return (((unsigned)(size_t)p >> 7) & 7) == 3 || (((unsigned)(size_t)p >> 7) & 31) == 31; }

template <int U> struct Frag { v2i w[4][2]; v4i x[U][4]; };
struct Addr { unsigned w0, w1, x0, x1; };

template <int U>
__device__ __forceinline__ void read_frag(Frag<U>& f, char* smem, const Addr& a, int r) {
    const int kk = r / (2 + U), i = r % (2 + U);
    if (i < 2) f.w[kk][i] = __builtin_amdgcn_ds_read_tr8_b64_v2i32((LDS_AS v2i*)(smem + (kk & 1 ? a.w1 : a.w0) + (kk >> 1) * 8192 + i * 1024));
    else f.x[i - 2][kk] = *(const v4i*)(smem + (kk >> 1 ? a.x1 : a.x0) + (kk & 1) * 2048 + (i - 2) * 4096);
}
template <int U>
__device__ __forceinline__ void mfma(const Frag<U>& f, v4i (&acc)[U][4], int m) {
    const int k2 = m / (4 * U), u = m / 4 % U, i = m % 4;
    const v2i lo = f.w[k2 * 2 + (i >> 1)][0], hi = f.w[k2 * 2 + (i >> 1)][1];
    asm volatile("v_mfma_i32_16x16x64_i8 %0, %1, %2, %0" : "+v"(acc[u][i]) : "v"(v4i{lo.x, lo.y, hi.x, hi.y}), "v"(f.x[u][k2 * 2 + (i & 1)]));
}
template <int U>
__device__ __forceinline__ void stage(const Frag<U>& cur, Frag<U>& nxt, v4i (&acc)[U][4], char* smem, const Addr& a) {
    constexpr int NRD = 4 * (2 + U), PER = U == 1 ? 2 : 1;
#pragma unroll
    for (int m = 0; m < 8 * U; ++m) {
        mfma<U>(cur, acc, m);
#pragma unroll
        for (int r = m * PER; r < (m + 1) * PER && r < NRD; ++r) read_frag<U>(nxt, smem, a, r);
        PIN();
    }
}

template <int U>
__device__ __forceinline__ void tile_compute(char* smem, int lane, int cw, unsigned keep0, float keep1, float keep2) {
    const int g = lane >> 4, q = (lane & 15) >> 1;
    const unsigned wad = (16 * g + q) * 128 + (((2 * cw) ^ (((q >> 1) & 3) << 1) ^ (g & 1)) << 4) + (lane & 1) * 8;
    const unsigned xad = (lane & 15) * 128 + ((g ^ ((lane >> 1) & 7)) << 4) + 16384;
    Addr a = {wad, wad ^ 16, xad, xad ^ 64};
    auto next = [&]() {
        constexpr unsigned M = NSLOT * SLOT_BYTES - 1;
        a.w0 = (a.w0 + SLOT_BYTES) & M; a.w1 = a.w0 ^ 16; a.x0 = (a.x0 + SLOT_BYTES) & M; a.x1 = a.x0 ^ 64;
        PIN();
    };
    v4i acc[U][4];
#pragma unroll
    for (int u = 0; u < U; ++u)
#pragma unroll
        for (int i = 0; i < 4; ++i) acc[u][i] = v4i{BIAS, BIAS, BIAS, BIAS};
    Frag<U> fa, fb;
    BARRIER();
#pragma unroll
    for (int r = 0; r < 4 * (2 + U); ++r) read_frag<U>(fa, smem, a, r);
    asm volatile("" :: "s"(keep0), "s"(keep1), "s"(keep2));
#pragma unroll 1
    for (int s = 1; s < 31; s += 2) {
        next();
        WAIT_LDS();
        BARRIER();
        stage<U>(fa, fb, acc, smem, a);
        next();
        WAIT_LDS();
        BARRIER();
        stage<U>(fb, fa, acc, smem, a);
    }
    next();
    WAIT_LDS();
    BARRIER();
    stage<U>(fa, fb, acc, smem, a);
    char* const d0 = smem + cw * 4096 + (lane & 15) * 128 + ((g ^ (lane & 7)) << 4);
    char* const d1 = smem + cw * 4096 + (lane & 15) * 128 + (((4 + g) ^ (lane & 7)) << 4);
#pragma unroll
    for (int u = 0; u <= U; ++u) {
        if (u == U) asm volatile("s_nop 7" ::: "memory");
#pragma unroll
        for (int i = 0; i < 8; ++i) {
            if (u < U) mfma<U>(fb, acc, (i >> 2) * 4 * U + u * 4 + (i & 3));
            if (u > 0 && i >= 4) *(v4i*)((i & 2 ? d1 : d0) + (u - 1) * 16384 + (i & 1) * 2048) = acc[u - 1][i & 3];
            PIN();
        }
    }
    WAIT_LDS();
}

__device__ __forceinline__ void write_rows(const u32x4& t, float scale, char* obase, int lane) {
    f32x4 v;
#pragma unroll
    for (int e = 0; e < 4; ++e) v[e] = (float)(int)(t[e] - BIAS) * scale;
    store_out(obase, (lane >> 3) * (NS * 4) + (lane & 7) * 16, __builtin_bit_cast(u32x4, v));
}

__device__ __forceinline__ unsigned send_subtile(const u32x4 (&t)[4], char* base, int n, int lane, int pace, char* go, unsigned call) {
    const unsigned off = lane * 16;
    char* const dst = base + n * SHORT_BYTES;
    u32x4 lo[2], hi[2];
#pragma unroll
    for (int i = 0; i < 2; ++i)
        lo[i] = u32x4{__builtin_amdgcn_perm(t[2 * i][1], t[2 * i][0], 0x05040100u), __builtin_amdgcn_perm(t[2 * i][3], t[2 * i][2], 0x05040100u),
                      __builtin_amdgcn_perm(t[2 * i + 1][1], t[2 * i + 1][0], 0x05040100u), __builtin_amdgcn_perm(t[2 * i + 1][3], t[2 * i + 1][2], 0x05040100u)};
    if (pace == 2) WAIT_VM(PUSH_B);
    store_sys(dst, off, lo[0]);
    if (pace == 2) WAIT_VM(PUSH_B);
    store_sys(dst + 1024, off, lo[1]);
#pragma unroll
    for (int i = 0; i < 2; ++i)
        hi[i] = u32x4{__builtin_amdgcn_perm(t[2 * i][1], t[2 * i][0], 0x07060302u), __builtin_amdgcn_perm(t[2 * i][3], t[2 * i][2], 0x07060302u),
                      __builtin_amdgcn_perm(t[2 * i + 1][1], t[2 * i + 1][0], 0x07060302u), __builtin_amdgcn_perm(t[2 * i + 1][3], t[2 * i + 1][2], 0x07060302u)};
    const u32x4 o = hi[0] | hi[1];
    if (__any(((o[0] | o[1] | o[2] | o[3]) & 0xffc0ffc0u) != 0)) {
        if (go && lane == 0) set_flag(go, call, 0);
        store_sys(base + Y_LONG + n * 2048, off, hi[0]);
        store_sys(base + Y_LONG + n * 2048 + 1024, off, hi[1]);
        return 1u;
    }
    unsigned b[4];
#pragma unroll
    for (int i = 0; i < 4; ++i) b[i] = __builtin_amdgcn_perm(hi[i >> 1][(i & 1) * 2 + 1], hi[i >> 1][(i & 1) * 2], 0x06040200u);
    const unsigned h0 = b[0] | ((b[3] << 6) & 0xc0c0c0c0u), h1 = b[1] | ((b[3] << 4) & 0xc0c0c0c0u), h2 = b[2] | ((b[3] << 2) & 0xc0c0c0c0u);
    auto s1 = __builtin_amdgcn_permlane16_swap(h0, h1, false, false);
    auto s2 = __builtin_amdgcn_permlane16_swap(h2, h2, false, false);
    auto s3 = __builtin_amdgcn_permlane32_swap(s1[1], s2[1], false, false);
    const u32x4 h = {h0, h1, h2, s3[1]};
    if (pace == 1) WAIT_VM(PUSH_A); else if (pace == 2) WAIT_VM(PUSH_B);
    if (go && lane == 0) set_flag(go, call, 0);
    if (lane < 48) store_sys(dst + 2048, off, h);
    return 0u;
}

constexpr unsigned READY = 1u << 31, FAILED = 1u << 30;

template <int D>
__device__ __forceinline__ void fetch_subtile(const char* base, int n, unsigned buf, bool raw, int lane) {
    const char* const src = base + n * SHORT_BYTES;
    WAIT_VM(D);
    glds16_sys(src, lane * 16, buf);
    WAIT_VM(D);
    glds16_sys(src + 1024, lane * 16, buf + 1024);
    WAIT_VM(D);
    glds16_sys(raw ? base + Y_LONG + n * 2048 : src + 2048, min(lane, raw ? 63 : 47) * 16, buf + 2048);
    if (raw) glds16_sys(base + Y_LONG + n * 2048 + 1024, lane * 16, buf + 3072);
}
template <int H>
__device__ __forceinline__ void finish_half(char* smem, unsigned buf, bool raw, float scale, char* obase, int lane) {
    const u32x4 lo = *(const u32x4*)(smem + buf + H * 1024 + lane * 16);
    const unsigned off = (lane >> 3) * (NS * 4) + (lane & 7) * 16;
    f32x4 v[2];
    if (raw) {
        const u32x4 hi = *(const u32x4*)(smem + buf + 2048 + H * 1024 + lane * 16);
#pragma unroll
        for (int e = 0; e < 4; ++e) {
            v[e >> 1][(e & 1) * 2] = (float)(int)(__builtin_amdgcn_perm(hi[e], lo[e], 0x05040100u) - BIAS) * scale;
            v[e >> 1][(e & 1) * 2 + 1] = (float)(int)(__builtin_amdgcn_perm(hi[e], lo[e], 0x07060302u) - BIAS) * scale;
        }
    } else {
        const unsigned ha = buf + 2048 + (lane < 48 ? lane * 16 : (lane - 48) * 16 + 12), hs = lane < 48 ? 4 : 256;
        const unsigned h0 = *(const unsigned*)(smem + ha), h1 = *(const unsigned*)(smem + ha + hs);
        unsigned b[2];
        if (H == 0) {
            b[0] = h0 & 0x3f3f3f3fu; b[1] = h1 & 0x3f3f3f3fu;
        } else {
            const unsigned h2 = *(const unsigned*)(smem + ha + 2 * hs);
            b[0] = h2 & 0x3f3f3f3fu;
            b[1] = ((h0 >> 6) & 0x03030303u) | ((h1 >> 4) & 0x0c0c0c0cu) | ((h2 >> 2) & 0x30303030u);
        }
#pragma unroll
        for (int i = 0; i < 2; ++i) {
            v[i][0] = ((float)__builtin_amdgcn_perm(b[i], lo[2 * i], 0x0c040100u) - (float)BIAS) * scale;
            v[i][1] = ((float)__builtin_amdgcn_perm(b[i], lo[2 * i], 0x0c050302u) - (float)BIAS) * scale;
            v[i][2] = ((float)__builtin_amdgcn_perm(b[i], lo[2 * i + 1], 0x0c060100u) - (float)BIAS) * scale;
            v[i][3] = ((float)__builtin_amdgcn_perm(b[i], lo[2 * i + 1], 0x0c070302u) - (float)BIAS) * scale;
        }
    }
    store_out(obase + (2 * H) * 8 * (NS * 4), off, __builtin_bit_cast(u32x4, v[0]));
    store_out(obase + (2 * H + 1) * 8 * (NS * 4), off, __builtin_bit_cast(u32x4, v[1]));
}

__device__ __forceinline__ void poll_flags(const char* flag, bool active, LDS_AS unsigned* word, char* tmo_, unsigned call) {
    GLB_AS unsigned* tm = (GLB_AS unsigned*)tmo_;
    const u64 t0 = __builtin_amdgcn_s_memrealtime();
    bool waiting = active;
    while (true) {
        const u64 v = __hip_atomic_load((GLB_AS u64*)flag, __ATOMIC_RELAXED, __HIP_MEMORY_SCOPE_SYSTEM);
        if (waiting && (unsigned)(v >> 32) >= call) {
            __hip_atomic_store(word, ((unsigned)v & 0xffffu) | READY, __ATOMIC_RELEASE, __HIP_MEMORY_SCOPE_WORKGROUP);
            waiting = false;
        }
        if (!__any(waiting)) break;
        const u64 dt = __builtin_amdgcn_s_memrealtime() - t0;
        if (dt > 10000 && (dt > DEADLINE_TICKS || __hip_atomic_load(tm, __ATOMIC_RELAXED, __HIP_MEMORY_SCOPE_AGENT) == call)) {
            __hip_atomic_store(tm, call, __ATOMIC_RELAXED, __HIP_MEMORY_SCOPE_AGENT);
            if (waiting) __hip_atomic_store(word, READY | FAILED, __ATOMIC_RELEASE, __HIP_MEMORY_SCOPE_WORKGROUP);
            break;
        }
        __builtin_amdgcn_s_sleep(POLL_SLEEP);
    }
}

template <int PER, int NW, int STEP, int INFL = 63>
__device__ __forceinline__ void warm_l2(const char* wblk, const char* xf, int me, int lane) {
    const unsigned wo = (lane >> 2) * ND + (lane & 3) * 128;
#pragma unroll 1
    for (int n = me; n < 32 * PER; n += STEP) {
        const int kb = n / PER, j = n - kb * PER;
        const bool isw = j < NW;
        if (INFL < 63) WAIT_VM(INFL);
        glds4(isw ? wblk + (size_t)(kb * 128 + j * 16) * ND : xf + (size_t)(j - NW) * 64 * KD + kb * 128, isw ? wo : (unsigned)lane * KD);
    }
}

__device__ __forceinline__ void warm_remote(const char* wblk, const char* x, int me, int lane) {
    constexpr int NWARM = __builtin_popcount(PF_MASK) * 8 + (4 - AGG) * 2;
    if (PF_NX == 0) {
        warm_l2<8 + (PF_XU + 1) / 2, 8, NWARM>(wblk, x + AGU * 32 * KD, me, lane);
        if (PF2_XU && (unsigned)(me - PF2_LO * 2) < PF2_N * 2) warm_l2<PF2_XU ? PF2_XU / 2 : 1, 0, PF2_N * 2>(wblk, x + (AGU + PF2_FIRST) * 32 * KD, me - PF2_LO * 2, lane);
    } else if (me >= 2 * PF_NX) {
        warm_l2<8, 8, NWARM - 2 * PF_NX>(wblk, x, me - 2 * PF_NX, lane);
    } else {
        warm_l2<(PF_XU + 1) / 2, 0, PF_NX ? 2 * PF_NX : 1>(wblk, x + AGU * 32 * KD, me, lane);
        if (PF2_XU && me < PF2_N * 2) warm_l2<PF2_XU ? PF2_XU / 2 : 1, 0, PF2_N * 2>(wblk, x + (AGU + PF2_FIRST) * 32 * KD, me, lane);
    }
}
__device__ __forceinline__ void advance(LDS_AS unsigned* ctl, char* counter, unsigned call, int lane) {
    asm volatile("" :: "s"(call) : "memory");
    if (lane == 0 && __hip_atomic_fetch_add(ctl + 12, 1u, __ATOMIC_RELAXED, __HIP_MEMORY_SCOPE_WORKGROUP) == 7)
        __hip_atomic_store((GLB_AS unsigned*)counter, call, __ATOMIC_RELAXED, __HIP_MEMORY_SCOPE_AGENT);
}

typedef const CST_AS Group* KArgs;
#define FRESH() KArgs kp = kp0; int b = b0, wid = wid0, lane = lane0; \
    asm volatile("" : "+s"(kp), "+s"(b), "+s"(wid) :: "memory"); asm volatile("" : "+v"(lane))
#define WLOC() (kp->wblk + (kp->rank - kp->peer) * NS)
#define CALL() (*(const CST_AS unsigned*)(kp->myws + WS_CNT + b * 128) + 1u)
#define SCALE() (*(const CST_AS float*)kp->sx * *(const CST_AS float*)kp->sw)

__global__ void __launch_bounds__(BLOCK, 2) fused_kernel(Params) {
    extern __shared__ __attribute__((aligned(16))) char smem[];
    const KArgs kp = (KArgs)__builtin_amdgcn_kernarg_segment_ptr() + (blockIdx.x & 7), kp0 = kp;
    const int lane = threadIdx.x & 63, wid = __builtin_amdgcn_readfirstlane(threadIdx.x >> 6), b = blockIdx.x;
    const int lane0 = lane, wid0 = wid, b0 = b;
    const char *x0 = kp->x, *w0 = kp->wblk;
    char* ws0 = kp->myws;
    const u64 tstart = __builtin_amdgcn_s_memrealtime();
    PIN();
#define grp (b & 7)
#define cls (b >> 5)
#define pan ((b >> 3) & 3)
#define local ((b & 7) == LOCAL_GRP)
#define NIB(word) (((unsigned)(word) >> (4 * cls)) & 15)
    LDS_AS unsigned* ctl = (LDS_AS unsigned*)(smem + CTL_OFF);
    const bool link = cls == 0 && !local;

    if (link) {
        constexpr int PPU = 32 / AGG, NP = AGU * PPU;
        static_assert(AGD >= 2 && AGD <= 8 && AGD <= PPU && (AGG == 2 || AGG == 4), "staging ring");
        const char *x = x0, *sxp = kp->sx, *swp = kp->sw;
        char *myws = ws0, *peer = kp->peerws;
        unsigned rank = kp->rank, src = kp->peer;
        const bool ag = pan < AGG;
        const unsigned agoff = (pan * 4 + (wid & 3)) * 1024 + lane * 16;
        asm volatile("" : "+s"(x), "+s"(sxp), "+s"(swp), "+s"(myws), "+s"(peer), "+s"(rank), "+s"(src));
        unsigned done = *(const CST_AS unsigned*)(myws + WS_CNT + b * 128);
        float sx = *(const CST_AS float*)sxp, sw = *(const CST_AS float*)swp;
        PIN();
        if (ag && wid < 4) {
#pragma unroll
            for (int i = 0; i < AGD; ++i) AGLD(x + i * (AGG * 4096), agoff, (wid * 8 + i) * 1024);
        }
        if (wid == 7) {
            ctl[lane] = 0;
            ctl[64 + lane] = 0;
            WAIT_LDS();
        }
        BARRIER();
        if (!ag && (wid >> 1) == 2) warm_remote(kp->wblk, x, (pan - AGG) * 2 + (wid & 1), lane);
        asm volatile("" : "+s"(done), "+s"(sx), "+s"(sw));
        const unsigned call = done + 1, par = call & 1;
        const float scale = sx * sw;
        char* const din = peer + WS_IN + (par * NR + rank) * (unsigned)IN_BYTES;
        STAMP(1);
        if (ag && wid < 4) {
            int r = 0;
#pragma unroll 1
            for (int i = 0; i < NP; ++i) {
                if (i >= AGD - 1) WAIT_VM(2 * AGD - 2); else WAIT_VM(AGD - 1);
                if (i >= AGD && ((i - AGD) & (PPU - 1)) == PPU - 1 && lane == 0)
                    __hip_atomic_fetch_add(ctl + 8 + (i - AGD) / PPU, 1u, __ATOMIC_RELAXED, __HIP_MEMORY_SCOPE_WORKGROUP);
                const u32x4 v = *(const u32x4*)(smem + (wid * 8 + r) * 1024 + lane * 16);
                store_sys(din + IN_X + i * (AGG * 4096), agoff, v);
                AGLD(x + (i + AGD < NP ? i + AGD : 0) * (AGG * 4096), agoff, (wid * 8 + r) * 1024);
                r = r == AGD - 1 ? 0 : r + 1;
            }
            WAIT_VM(0);
            if (lane == 0) __hip_atomic_fetch_add(ctl + 8 + AGU - 1, 1u, __ATOMIC_RELAXED, __HIP_MEMORY_SCOPE_WORKGROUP);
            STAMP(3);
        }
        advance(ctl, myws + WS_CNT + b * 128, call, lane);
        constexpr int NPULL = PULL_ORDER >> 12 ? 4 : (PULL_ORDER >> 8 ? 3 : (PULL_ORDER >> 4 ? 2 : 1));
        constexpr unsigned PULL_BUF = 0;
        constexpr int NPU = pull_units();
        static_assert(PULL_BUF + NPU * 16384 <= CTL_OFF && PULL_D <= 5, "pull buffers");
        LDS_AS unsigned* landed = ctl + 16;
        static_assert(NPULL < 4, "lanes 12..15 watch the go-ahead");
        constexpr int N0 = 4 * ((R_CNT >> 4 * (PULL_ORDER & 15)) & 15), N1 = N0 + 3 * (NPU - N0 / 4);
        int tp = 3;
        for (int p = 2; p >= 0; --p) if (tight(WLOC() + p * 128)) tp = p;
        auto entry = [&](int m, int& p, int& q) {
            int j;
            q = pan;
            if (!SPREAD) { j = m >> 2; p = pan; q = m & 3; }
            else if (m < N0) { j = m >> 2; p = m & 3; }
            else if (m < N1) { j = (m - N0) / 3; p = m - N0 - 3 * j; p += p >= tp; j += N0 / 4; }
            else { j = m - N1 + N0 / 4; p = tp; }
            return (int)(pull_table() >> (8 * j)) & 255;
        };
        LDS_AS u64* mirror = (LDS_AS u64*)(ctl + (MIR_OFF - CTL_OFF) / 4);
        auto flag_word = [&](int i) {
            const u64 v = __hip_atomic_load(mirror + i * 2, __ATOMIC_ACQUIRE, __HIP_MEMORY_SCOPE_WORKGROUP);
            const unsigned fail = __hip_atomic_load(ctl + 14, __ATOMIC_ACQUIRE, __HIP_MEMORY_SCOPE_WORKGROUP);
            const bool ok = (unsigned)(v >> 32) >= call && ((unsigned)v >> 16) == (call & 0xffffu);
            return (ok ? ((unsigned)v & 0xffffu) | READY : 0u) | fail;
        };
        if (wid == 7) {
            for (int u = 0; ag && u < AGU; ++u) {
                while (__hip_atomic_load(ctl + 8 + u, __ATOMIC_RELAXED, __HIP_MEMORY_SCOPE_WORKGROUP) < 4) __builtin_amdgcn_s_sleep(2);
                if (lane == 0) set_flag(din + IN_XFLAG + (u * 4 + pan) * 128, call, 0);
            }
        } else if (wid == 6) {
            const char* const flags = myws + WS_IN + (par * NR + src) * (unsigned)IN_BYTES + IN_FLAG;
            const unsigned fo = (((PULL_ORDER >> (lane & 12)) & 15) * 4 + (lane & 3)) * FSP;
            const bool watch = (lane & 15) < NPULL * 4 && (SPREAD || (lane & 3) == pan);
            GLB_AS unsigned* tm = (GLB_AS unsigned*)(myws + WS_TMO);
            while (__hip_atomic_load(ctl + 12, __ATOMIC_RELAXED, __HIP_MEMORY_SCOPE_WORKGROUP) < 8) __builtin_amdgcn_s_sleep(4);
            const u64 t0 = __builtin_amdgcn_s_memrealtime();
            while (true) {
                WAIT_VM(7);
                if (lane < 16) glds16_sys(flags, fo, MIR_OFF);
                __builtin_amdgcn_s_sleep(MIR_SLEEP);
                if (!__any((watch || (GO_CLS && (lane & 15) == 12 + pan)) && !(flag_word(lane & 15) & READY))) break;
                const u64 dt = __builtin_amdgcn_s_memrealtime() - t0;
                if (dt > 10000 && (dt > DEADLINE_TICKS || __hip_atomic_load(tm, __ATOMIC_RELAXED, __HIP_MEMORY_SCOPE_AGENT) == call)) {
                    __hip_atomic_store(tm, call, __ATOMIC_RELAXED, __HIP_MEMORY_SCOPE_AGENT);
                    __hip_atomic_store(ctl + 14, READY | FAILED, __ATOMIC_RELEASE, __HIP_MEMORY_SCOPE_WORKGROUP);
                    break;
                }
            }
        }
        int ep, eq;
        const int ee = entry(min(lane, NPU * 4 - 1), ep, eq);
        const unsigned eout = ((ee & 15) * 32 * NS + ep * 128 + eq * 32) * 4;
        char* const out = kp->out + (size_t)src * MS * NS * 4;
        constexpr int LAST = (NPU - 1) * 4;
        if (wid < 4) {
            const char* const sout = peer + WS_OUT + (par * NR + rank) * (unsigned)OUT_BYTES;
            const int en = ((ee & 15) * 4 + ep) * 4 + eq, ef = (ee >> 6) * 4 + ep, eb = ((ee >> 4) & 3) * 4 + eq;
            if (GO_CLS) while (!(flag_word(12 + pan) & READY)) __builtin_amdgcn_s_sleep(1);
            unsigned words = 0, marks = 0;
#pragma unroll 1
            for (int j = 0; j < NPU; ++j) {
                const int m = j * 4 + wid;
                unsigned word = __builtin_amdgcn_readlane(words, m);
                for (int k = 0; !(word & READY); ++k) {
                    if (k == 1) WAIT_VM(2);
                    else if (k == 2) WAIT_VM(1);
                    else if (k == 3) {
                        WAIT_VM(0);
                        if (lane == 0) __hip_atomic_store(landed + wid, marks | j, __ATOMIC_RELEASE, __HIP_MEMORY_SCOPE_WORKGROUP);
                    } else if (k) __builtin_amdgcn_s_sleep(1);
                    const unsigned w = flag_word(ef);
                    words = (w & (READY | FAILED)) | ((w >> eb) & 1);
                    word = __builtin_amdgcn_readlane(words, m);
                }
                marks |= ((word & 1) << 8 | (word & FAILED ? 1u << 16 : 0u)) << j;
                if (word & FAILED) WAIT_VM(0);
                else fetch_subtile<PULL_D>(sout, __builtin_amdgcn_readlane(en, m), PULL_BUF + m * 4096, word & 1, lane);
                static_assert(PULL_D <= 2 && NPU <= 8, "landed count");
                if (lane == 0) __hip_atomic_store(landed + wid, marks | j, __ATOMIC_RELEASE, __HIP_MEMORY_SCOPE_WORKGROUP);
            }
            char* o = out + __builtin_amdgcn_readlane(eout, LAST + wid);
            float sc = (marks >> (15 + NPU)) & 1 ? __builtin_nanf("") : scale;
            asm volatile("" : "+s"(o));
            asm volatile("" : "+v"(sc));
            WAIT_VM(0);
            if (lane == 0) __hip_atomic_store(landed + wid, marks | NPU, __ATOMIC_RELEASE, __HIP_MEMORY_SCOPE_WORKGROUP);
            finish_half<0>(smem, PULL_BUF + (LAST + wid) * 4096, (marks >> (7 + NPU)) & 1, sc, o, lane);
        } else {
            for (bool more = true; more;) {
                unsigned m = 0;
                if (lane == 0) m = __hip_atomic_fetch_add(ctl + 13, 1u, __ATOMIC_RELAXED, __HIP_MEMORY_SCOPE_WORKGROUP);
                m = __builtin_amdgcn_readfirstlane(m);
                more = m < LAST;
                if (!more) m = LAST + wid - 4;
                char* o = out + __builtin_amdgcn_readlane(eout, m);
                asm volatile("" : "+s"(o));
                unsigned w;
                while (((w = __builtin_amdgcn_readfirstlane(__hip_atomic_load(landed + (m & 3), __ATOMIC_ACQUIRE, __HIP_MEMORY_SCOPE_WORKGROUP))) & 255) <= m >> 2)
                    if (more) __builtin_amdgcn_s_sleep(1);
                w >>= m >> 2;
                const float sc = (w >> 16) & 1 ? __builtin_nanf("") : scale;
                finish_half<1>(smem, PULL_BUF + m * 4096, (w >> 8) & 1, sc, o, lane);
                if (more) finish_half<0>(smem, PULL_BUF + m * 4096, (w >> 8) & 1, sc, o, lane);
            }
        }
    } else {
        int cnt = NIB(local ? L_CNT : R_CNT);
        const char* const wp = w0;
        const char* const xp = x0;
        if (wid == 7) {
            ctl[lane] = 0;
            WAIT_LDS();
        }
        if (((local ? LPF_MASK : PF_MASK) >> cls) & 1) {
            if (wid == 6) asm volatile("" :: "s"(CALL()), "v"(SCALE()));
            if ((wid >> 1) == 2) {
                const int me = (__builtin_popcount((local ? LPF_MASK : PF_MASK) & ((1 << cls) - 1)) * 4 + pan) * 2 + (wid & 1);
                if (local) {
                    warm_l2<8, 0, __builtin_popcount(LPF_MASK) * 8>(wp, xp, me, lane);
                    if (LW_START) while (__builtin_amdgcn_s_memrealtime() - tstart < LW_START) __builtin_amdgcn_s_sleep(4);
                    warm_l2<8, 8, __builtin_popcount(LPF_MASK) * 8>(wp, xp, me, lane);
                } else {
                    warm_remote(wp, xp, (4 - AGG) * 2 + me, lane);
                    if (cls == WL_CLS) {
                        const char* wl = WLOC();
#pragma unroll 1
                        for (int p = 0; p < 4; ++p)
                            if (tight(wl + p * 128))
#pragma unroll 1
                                for (int n = pan * 2 + (wid & 1); n < 64; n += 8) glds4(wl + p * 128 + (size_t)n * 64 * ND, lane * ND);
                    }
                }
                if (WARM_DRAIN) WAIT_VM(0);
            }
            BARRIER();
        }
        if (NIB(local ? L_START : START))
            while (__builtin_amdgcn_s_memrealtime() - tstart < (u64)(NIB(local ? L_START : START) * 64)) __builtin_amdgcn_s_sleep(2);
        int task = cnt ? 0 : 1;
        unsigned lastcall = 0;
        const int ntask = !local && NIB(A_CNT) ? 2 : 1;
#pragma unroll 1
        for (; task < ntask; ++task) {
            unsigned bad = 0, afirst = local ? NIB(L_FIRST) : AGU + NIB(R_FIRST);
            const char *wp1 = nullptr, *xp1 = nullptr;
            if (task == 1) {
                BARRIER();
                FRESH();
                const unsigned call = CALL();
                const char* sin = kp->myws + WS_IN + (size_t)((call & 1) * NR + kp->peer) * IN_BYTES;
                wp1 = WLOC();
                const bool tp = tight(wp1 + pan * 128);
                cnt = NIB(tp ? T_CNT : A_CNT);
                afirst = NIB(tp ? T_FIRST : A_FIRST);
                if (A_START && !tp && wid == 1) while (__builtin_amdgcn_s_memrealtime() - tstart < A_START) __builtin_amdgcn_s_sleep(4);
                if (wid == 1) poll_flags(sin + IN_XFLAG + ((afirst + cnt - 1) * 4 + (lane & 3)) * 128, lane < AGG,
                                         ctl + 4 + (lane & 3), kp->myws + WS_TMO, call);
                WAIT_LDS();
                BARRIER();
                bad = __builtin_amdgcn_readfirstlane((ctl[4] | ctl[5] | ctl[6] | ctl[7]) & FAILED);
                xp1 = sin + IN_X;
                STAMP(4);
            }
            if (wid < 4 && !bad) {
                FRESH();
                unsigned woff = w_offset(lane, wid) + pan * 128, xoff = x_offset(lane, wid) + afirst * 32 * KD;
                asm volatile("" : "+v"(woff), "+v"(xoff));
                PIN();
                load_tile_n(task ? wp1 : w0, task ? xp1 : x0, cnt, woff, xoff, wid);
            }
            FRESH();
            const bool direct = local || task == 1, push = (R_PUSH >> cls) & 1;
            char *myws = kp->myws, *out = kp->out, *din = kp->peerws;
            const char *sxp = kp->sx, *swp = kp->sw;
            unsigned rank = kp->rank, src = kp->peer, dst = kp->peer;
            asm volatile("" : "+s"(myws), "+s"(out), "+s"(din), "+s"(sxp), "+s"(swp), "+s"(rank), "+s"(src), "+s"(dst));
            const unsigned done = *(const CST_AS unsigned*)(myws + WS_CNT + b * 128);
            const float sx = *(const CST_AS float*)sxp, sw = *(const CST_AS float*)swp;
            din += WS_IN + rank * (unsigned)IN_BYTES;
            char* base = direct ? out + (((task == 1 ? src : rank) * 16 + afirst) * 32 * NS + pan * 128) * 4
                       : push ? din : myws + WS_OUT + dst * (unsigned)OUT_BYTES;
            unsigned odd = direct ? 0u : push ? (unsigned)(NR * IN_BYTES) : (unsigned)(NR * OUT_BYTES);
            asm volatile("" : "+s"(base), "+s"(din), "+s"(odd));
            if (wid >= 4 && !bad) {
                if (cnt == 1) tile_compute<1>(smem, lane, wid - 4, done, sx, sw);
                else if (cnt == 2) tile_compute<2>(smem, lane, wid - 4, done, sx, sw);
                else tile_compute<3>(smem, lane, wid - 4, done, sx, sw);
            }
            if (cnt > 1 || direct) BARRIER();
            const unsigned call = done + 1;
            const float scale = bad ? __builtin_nanf("") : sx * sw;
            if (call & 1) { base += odd; din += NR * IN_BYTES; }
            if (!direct && push)
                while (__builtin_amdgcn_s_memrealtime() - tstart < (u64)(HOLD0 + HOLD1 * (afirst - AGU))) __builtin_amdgcn_s_sleep(1);
            lastcall = call;
            unsigned rawbits = 0;
            const unsigned ta = (lane >> 3) * 128 + (((lane & 7) ^ (lane >> 3)) << 4);
            if (direct) {
#pragma unroll 1
                for (int i = wid; i < cnt * 8; i += 8) {
                    const u32x4 t0 = *(const u32x4*)(smem + i * 2048 + ta), t1 = *(const u32x4*)(smem + i * 2048 + 1024 + ta);
                    char* const o = base + ((i >> 3) * 32 + (i & 1) * 16) * (NS * 4) + ((i >> 1) & 3) * 128;
                    write_rows(t0, scale, o, lane);
                    write_rows(t1, scale, o + 8 * NS * 4, lane);
                }
            } else {
#pragma unroll 1
                for (int i = wid ^ 4; i < cnt * 4; i += 8) {
                    u32x4 t[4];
#pragma unroll
                    for (int j = 0; j < 4; ++j) t[j] = *(const u32x4*)(smem + i * 4096 + j * 1024 + ta);
                    if (GO_ITEM >= 8 && cls == GO_CLS && i == GO_ITEM && lane == 0) set_flag(din + IN_FLAG + pan * FSP, call, 0);
                    rawbits |= send_subtile(t, base, ((afirst + (i >> 2)) * 4 + pan) * 4 + (i & 3), lane, push && cnt > 1 ? 1 + (i >> 3) : 0,
                                            GO_ITEM < 8 && cls == GO_CLS && i == GO_ITEM ? din + IN_FLAG + pan * FSP : nullptr, call) << i;
                }
            }
            if (!direct) {
                if (rawbits && lane == 0) __hip_atomic_fetch_or(ctl, rawbits, __ATOMIC_RELAXED, __HIP_MEMORY_SCOPE_WORKGROUP);
                asm volatile("s_waitcnt vmcnt(0) lgkmcnt(0)" ::: "memory");
                BARRIER();
                STAMP(3);
                if (wid == 7 && lane == 0) set_flag(din + IN_FLAG + (b >> 3) * FSP, call, ctl[0]);
            }
            if (task + 1 == ntask) advance(ctl, myws + WS_CNT + b * 128, call, lane);
        }
        FRESH();
        if (!lastcall) {
            lastcall = CALL();
            advance(ctl, kp->myws + WS_CNT + b * 128, lastcall, lane);
        }
        const unsigned call = lastcall;
        if (!local) {
            const int src = kp->peer;
            const char* sin = kp->myws + WS_IN + (size_t)((call & 1) * NR + src) * IN_BYTES;
            char* const out = kp->out + ((size_t)src * MS * NS + pan * 128) * 4;
            const float scale = SCALE();
#pragma unroll 1
            for (int c = 1; c < 8; ++c) {
                if (((M_CLS >> (4 * c)) & 15) != cls) continue;
                BARRIER();
                if (wid == 1) poll_flags(sin + IN_FLAG + (c * 4 + pan) * FSP, lane == 0, ctl + 16 + c, kp->myws + WS_TMO, call);
                WAIT_LDS();
                BARRIER();
                const unsigned word = __builtin_amdgcn_readfirstlane(ctl[16 + c]);
                for (int i = wid; i < ((R_CNT >> (4 * c)) & 15) * 4; i += 8) {
                    const int unit = AGU + ((R_FIRST >> (4 * c)) & 15) + (i >> 2), cw = i & 3;
                    const bool raw = (word >> i) & 1, failed = word & FAILED;
                    if (!failed) fetch_subtile<63>(sin, (unit * 4 + pan) * 4 + cw, i * 4096, raw, lane);
                    WAIT_VM(0);
                    finish_half<0>(smem, i * 4096, raw, failed ? __builtin_nanf("") : scale, out + (unit * 32 * NS + cw * 32) * 4, lane);
                    finish_half<1>(smem, i * 4096, raw, failed ? __builtin_nanf("") : scale, out + (unit * 32 * NS + cw * 32) * 4, lane);
                }
            }
        }
    }
}
#undef grp
#undef cls
#undef pan
#undef local

extern "C" void kernel_launch(int rank, int nranks, ncclComm_t comm,
                              hipStream_t stream,
                              void* const* d_in, const int* in_sizes, int n_in,
                              void* d_out, int out_size,
                              const int* in_dtypes, int out_dtype,
                              void* const* d_out_all,
                              void* const* d_ws_all, size_t ws_bytes) {
    if (nranks != NR || (unsigned)rank >= (unsigned)NR || ws_bytes < WS_END) return;
    static bool init[NR] = {};
    if (!init[rank]) {
        HIP_CHECK(hipFuncSetAttribute((const void*)fused_kernel, hipFuncAttributeMaxDynamicSharedMemorySize, LDS_BYTES));
        init[rank] = true;
    }
    Params p;
    int peer[NR];
    {
        const unsigned quad = ((0x4d >> rank) & 1) ? 0x4du : 0xb2u;
        int lo = 0, hi = 4;
        for (int i = 1; i < NR; ++i) {
            const int q = (rank + i) & 7;
            if (lo == LOCAL_GRP) ++lo;
            if (TOPO == 0) { peer[lo++] = q; continue; }
            if ((quad >> q) & 1) peer[lo++] = q; else peer[hi++] = q;
        }
        peer[LOCAL_GRP] = rank;
    }
    for (int g = 0; g < NR; ++g) {
        p.g[g].x = (const char*)d_in[0];
        p.g[g].wblk = (const char*)d_in[1] + peer[g] * NS;
        p.g[g].myws = (char*)d_ws_all[rank];
        p.g[g].peerws = (char*)d_ws_all[peer[g]];
        p.g[g].out = (char*)d_out;
        p.g[g].sx = (const char*)d_in[2];
        p.g[g].sw = (const char*)d_in[3];
        p.g[g].rank = rank;
        p.g[g].peer = peer[g];
    }
    hipLaunchKernelGGL(fused_kernel, dim3(GRID), dim3(BLOCK), LDS_BYTES, stream, p);
    HIP_CHECK(hipGetLastError());
}
```

```cpp
#include <hip/hip_runtime.h>
#include <rccl/rccl.h>
#include <cstdio>

#define HIP_CHECK(expr) do { hipError_t _e = (expr); if (_e != hipSuccess) { \
    fprintf(stderr, "HIP error %s at %s:%d\n", hipGetErrorString(_e), __FILE__, __LINE__); \
    return; } } while (0)

typedef int v2i __attribute__((ext_vector_type(2)));
typedef int v4i __attribute__((ext_vector_type(4)));
typedef unsigned u32x4 __attribute__((ext_vector_type(4)));
typedef float f32x4 __attribute__((ext_vector_type(4)));
typedef unsigned long long u64;
#define LDS_AS __attribute__((address_space(3)))
#define GLB_AS __attribute__((address_space(1)))
#define CST_AS __attribute__((address_space(4)))

constexpr int AGU = 3;
constexpr unsigned R_CNT = 0x33210220;
constexpr unsigned R_FIRST = 0x63100b90;
constexpr unsigned R_PUSH = 0x70;
constexpr unsigned PULL_ORDER = 0x127;
constexpr unsigned M_CLS = 0x06570000;
constexpr unsigned A_CNT = 0x00012000;
constexpr unsigned A_FIRST = 0x00020000;
constexpr unsigned T_CNT = 0x00021000;
constexpr unsigned T_FIRST = 0x00010000;
constexpr unsigned PF_MASK = 0x0e;
constexpr int PF_XU = 10;
constexpr int PF2_FIRST = 9, PF2_XU = 4, PF2_LO = 8, PF2_N = 3;
constexpr int GO_CLS = 6, GO_ITEM = 7;
constexpr int LOCAL_GRP = 3;
constexpr unsigned L_CNT = 0x33330022;
constexpr unsigned L_FIRST = 0x963000ec;
constexpr unsigned LPF_MASK = 0x0f;
#define L_START 256
#define A_START 930
#define HOLD0 670
#define HOLD1 78
#define AGD 4
#define TLBW 5

constexpr int NR = 8, MS = 512, KD = 4096, ND = 4096, NS = 512;
constexpr int GRID = 256, BLOCK = 512;
constexpr int NSLOT = 4;
constexpr int SLOT_BYTES = 32768;
constexpr int CTL_OFF = NSLOT * SLOT_BYTES;
constexpr int MIR_OFF = CTL_OFF + 256;
constexpr int JUNK_OFF = MIR_OFF + 256;
constexpr int LDS_BYTES = JUNK_OFF + 256;
constexpr int BIAS = 1 << 21;
constexpr u64 DEADLINE_TICKS = 40000000ull;

constexpr unsigned BLK_BYTES = 2u << 20, HALF_BYTES = 1u << 20;
constexpr int SHORT_BYTES = 2816;
constexpr unsigned H_X = 144 * 4096;
constexpr unsigned H_FLAG = H_X + AGU * 32 * KD;
constexpr unsigned H_XFLAG = H_FLAG + 1024;
constexpr unsigned H_CNT = H_FLAG + 4096;
constexpr unsigned H_TMO = H_CNT + 4096;
constexpr unsigned H_LONG = NR * BLK_BYTES;
constexpr size_t WS_END = 2 * (size_t)NR * BLK_BYTES;
static_assert((16 - AGU) * 16 * SHORT_BYTES <= H_X && H_XFLAG + AGU * 4 * 128 <= H_CNT && H_TMO + 256 <= HALF_BYTES, "layout");
static_assert(PF_XU % 2 == 0 && AGU + PF_XU <= 16 && PF2_XU % 2 == 0 && AGU + PF2_FIRST + PF2_XU <= 16, "warm-up rows");
constexpr int nib(unsigned w, int c) { return (w >> 4 * c) & 15; }
constexpr bool plan_ok() {
    unsigned r = 0, a = 0, t = 0, l = 0;
    for (int c = 0; c < 8; ++c) {
        const int rc = nib(R_CNT, c), ac = nib(A_CNT, c), tc = nib(T_CNT, c), lc = nib(L_CNT, c);
        const bool push = (R_PUSH >> c) & 1;
        bool pulled = false;
        for (int i = 0; i < 3; ++i) pulled |= nib(PULL_ORDER, i) == c;
        if (rc > 3 || ac > 3 || tc > 3 || lc > 3 || !tc != !ac) return false;
        if (rc ? pulled == push : pulled || push) return false;
        if ((nib(M_CLS, c) != 0) != push || nib(M_CLS, c) > 7) return false;
        if ((c && !rc && !ac) || (!lc && !((LPF_MASK >> c) & 1))) return false;
        r += ((1u << rc) - 1) << nib(R_FIRST, c);
        a += ((1u << ac) - 1) << nib(A_FIRST, c);
        t += ((1u << tc) - 1) << nib(T_FIRST, c);
        l += ((1u << lc) - 1) << nib(L_FIRST, c);
    }
    return r == (1u << (16 - AGU)) - 1 && a == (1u << AGU) - 1 && t == a && l == 0xffffu && !nib(A_CNT, 0) && !(PF_MASK & 1)
           && ((R_PUSH >> GO_CLS) & 1) && nib(R_CNT, GO_CLS) > 1 && GO_ITEM < 8;
}
static_assert(plan_ok(), "the tiles must cover every unit once");
constexpr int NPU = nib(R_CNT, nib(PULL_ORDER, 0)) + nib(R_CNT, nib(PULL_ORDER, 1)) + nib(R_CNT, nib(PULL_ORDER, 2));
constexpr u64 pull_table() {
    u64 v = 0; int n = 0;
    for (int t = 0; t < 3; ++t)
        for (int u = 0; u < nib(R_CNT, nib(PULL_ORDER, t)); ++u) v |= (u64)(t << 6 | u << 4 | (nib(R_FIRST, nib(PULL_ORDER, t)) + u)) << 8 * n++;
    return v;
}
static_assert(NPU <= 8 && NPU * 16384 <= CTL_OFF, "pull buffers");

#define STAMP(i) do { } while (0)

struct Group {
    const char* x;
    const char* wblk;
    char* mine;
    char* theirs;
    char* out;
    const char* sx;
    const char* sw;
    int wown;
    int peer;
};
struct Params { Group g[NR]; };
static_assert(sizeof(Group) == 64, "one line");

#define WAIT_LDS() __builtin_amdgcn_s_waitcnt(0xC07F)
#define WAIT_VM(n) asm volatile("s_waitcnt vmcnt(%0)" :: "n"(n) : "memory")
#define PIN() __builtin_amdgcn_sched_barrier(0)
#define BARRIER() do { asm volatile("" ::: "memory"); __builtin_amdgcn_s_barrier(); asm volatile("" ::: "memory"); } while (0)
#define NOW() __builtin_amdgcn_s_memrealtime()

__device__ __forceinline__ void glds16(const char* sbase, unsigned voff, unsigned lds) {
    asm volatile("s_mov_b32 m0, %2\n\ts_nop 0\n\tglobal_load_lds_dwordx4 %0, %1" :: "v"(voff), "s"(sbase), "s"(lds) : "memory");
}
__device__ __forceinline__ void glds16_sys(const char* sbase, unsigned voff, unsigned lds) {
    asm volatile("s_mov_b32 m0, %2\n\ts_nop 0\n\tglobal_load_lds_dwordx4 %0, %1 sc0 sc1" :: "v"(voff), "s"(sbase), "s"(lds) : "memory");
}
__device__ __forceinline__ void glds4(const char* sbase, unsigned voff) {
    asm volatile("s_mov_b32 m0, %2\n\ts_nop 0\n\tglobal_load_lds_dword %0, %1" :: "v"(voff), "s"(sbase), "s"(JUNK_OFF) : "memory");
}
__device__ __forceinline__ void store_sys(char* sbase, unsigned voff, u32x4 v) {
    asm volatile("global_store_dwordx4 %0, %1, %2 sc0 sc1\n\ts_nop 1" :: "v"(voff), "v"(v), "s"(sbase) : "memory");
}
template <int STEP>
__device__ __forceinline__ void store_sys(char* sbase, unsigned voff, u32x4 v, int i) {
    if (i == 0) store_sys(sbase, voff, v);
    else if (i == 1) asm volatile("global_store_dwordx4 %0, %1, %2 offset:%3 sc0 sc1\n\ts_nop 1" :: "v"(voff), "v"(v), "s"(sbase), "n"(STEP) : "memory");
    else asm volatile("global_store_dwordx4 %0, %1, %2 offset:%3 sc0 sc1\n\ts_nop 1" :: "v"(voff), "v"(v), "s"(sbase), "n"(2 * STEP) : "memory");
}
__device__ __forceinline__ void set_flag(char* flag, unsigned call, unsigned bits) {
    __hip_atomic_store((GLB_AS u64*)flag, ((u64)call << 32) | (call << 16) | bits, __ATOMIC_RELAXED, __HIP_MEMORY_SCOPE_SYSTEM);
}

__device__ __forceinline__ unsigned w_offset(int lane, int wid) {
    return (unsigned)(wid * 32 + (lane >> 3)) * ND + (((lane & 7) ^ (((lane >> 4) & 3) << 1)) << 4);
}
__device__ __forceinline__ unsigned x_offset(int lane, int wid) {
    return (unsigned)(wid * 8 + (lane >> 3)) * KD + (((lane & 7) ^ ((wid * 4 + (lane >> 4)) & 7)) << 4);
}
template <int U, int LA>
__device__ __forceinline__ void load_tile(const char* wp, const char* xp, unsigned woff, unsigned xoff, int wid) {
    static_assert(LA < NSLOT, "ring too small");
    unsigned slot = SLOT_BYTES;
#pragma unroll 1
    for (int s = 1; s < 32; ++s) {
        if (s >= LA) {
            WAIT_VM((LA - 1) * (4 + U));
            BARRIER();
        }
#pragma unroll
        for (int j = 0; j < 4; ++j) glds16(wp + (size_t)(s * 128 + j * 8) * ND, j >= 2 ? woff ^ 16 : woff, slot + wid * 4096 + j * 1024);
#pragma unroll
        for (int u = 0; u < U; ++u) glds16_sys(xp + u * 32 * KD + s * 128, xoff, slot + 16384 + u * 4096 + wid * 1024);
        slot = (slot + SLOT_BYTES) & (NSLOT * SLOT_BYTES - 1);
    }
#pragma unroll
    for (int r = LA - 1; r >= 0; --r) {
        WAIT_VM(r * (4 + U));
        BARRIER();
    }
}

template <int U> struct Frag { v2i w[4][2]; v4i x[U][4]; };
struct Addr { unsigned w0, w1, x0, x1; };

template <int U>
__device__ __forceinline__ void read_frag(Frag<U>& f, char* smem, const Addr& a, int r) {
    const int kk = r / (2 + U), i = r % (2 + U);
    if (i < 2) f.w[kk][i] = __builtin_amdgcn_ds_read_tr8_b64_v2i32((LDS_AS v2i*)(smem + (kk & 1 ? a.w1 : a.w0) + (kk >> 1) * 8192 + i * 1024));
    else f.x[i - 2][kk] = *(const v4i*)(smem + (kk >> 1 ? a.x1 : a.x0) + (kk & 1) * 2048 + (i - 2) * 4096);
}
template <int U>
__device__ __forceinline__ void mfma(const Frag<U>& f, v4i (&acc)[U][4], int m) {
    const int k2 = m / (4 * U), u = m / 4 % U, i = m % 4;
    const v2i lo = f.w[k2 * 2 + (i >> 1)][0], hi = f.w[k2 * 2 + (i >> 1)][1];
    asm volatile("v_mfma_i32_16x16x64_i8 %0, %1, %2, %0" : "+v"(acc[u][i]) : "v"(v4i{lo.x, lo.y, hi.x, hi.y}), "v"(f.x[u][k2 * 2 + (i & 1)]));
}
template <int U>
__device__ __forceinline__ void stage(const Frag<U>& cur, Frag<U>& nxt, v4i (&acc)[U][4], char* smem, const Addr& a) {
    constexpr int NRD = 4 * (2 + U), PER = U == 1 ? 2 : 1;
#pragma unroll
    for (int m = 0; m < 8 * U; ++m) {
        mfma<U>(cur, acc, m);
#pragma unroll
        for (int r = m * PER; r < (m + 1) * PER && r < NRD; ++r) read_frag<U>(nxt, smem, a, r);
        PIN();
    }
}
template <int U>
__device__ __forceinline__ void tile_compute(char* smem, int lane, int cw, unsigned wad, unsigned xad, unsigned keep0, float keep1, float keep2) {
    const int g = lane >> 4;
    Addr a = {wad, wad ^ 16, xad, xad ^ 64};
    auto next = [&]() {
        constexpr unsigned M = NSLOT * SLOT_BYTES - 1;
        a.w0 = (a.w0 + SLOT_BYTES) & M; a.w1 = a.w0 ^ 16; a.x0 = (a.x0 + SLOT_BYTES) & M; a.x1 = a.x0 ^ 64;
        PIN();
    };
    v4i acc[U][4];
#pragma unroll
    for (int u = 0; u < U; ++u)
#pragma unroll
        for (int i = 0; i < 4; ++i) acc[u][i] = v4i{BIAS, BIAS, BIAS, BIAS};
    Frag<U> fa, fb;
    BARRIER();
#pragma unroll
    for (int r = 0; r < 4 * (2 + U); ++r) read_frag<U>(fa, smem, a, r);
    asm volatile("" :: "s"(keep0), "s"(keep1), "s"(keep2));
#pragma unroll 1
    for (int s = 1; s < 31; s += 2) {
        next();
        WAIT_LDS();
        BARRIER();
        stage<U>(fa, fb, acc, smem, a);
        next();
        WAIT_LDS();
        BARRIER();
        stage<U>(fb, fa, acc, smem, a);
    }
    next();
    WAIT_LDS();
    BARRIER();
    stage<U>(fa, fb, acc, smem, a);
    char* const d0 = smem + cw * 4096 + (lane & 15) * 128 + ((g ^ (lane & 7)) << 4);
    char* const d1 = smem + cw * 4096 + (lane & 15) * 128 + (((4 + g) ^ (lane & 7)) << 4);
#pragma unroll
    for (int u = 0; u <= U; ++u) {
        if (u == U) asm volatile("s_nop 7" ::: "memory");
#pragma unroll
        for (int i = 0; i < 8; ++i) {
            if (u < U) mfma<U>(fb, acc, (i >> 2) * 4 * U + u * 4 + (i & 3));
            if (u > 0 && i >= 4) *(v4i*)((i & 2 ? d1 : d0) + (u - 1) * 16384 + (i & 1) * 2048) = acc[u - 1][i & 3];
            PIN();
        }
    }
    WAIT_LDS();
}
__device__ __forceinline__ bool tight(const char* p) { return (((unsigned)(size_t)p >> 7) & 7) == 3 || (((unsigned)(size_t)p >> 7) & 31) == 31; }

__device__ __forceinline__ void write_rows(const u32x4& t, float scale, char* obase, int lane) {
    f32x4 v;
#pragma unroll
    for (int e = 0; e < 4; ++e) v[e] = (float)(int)(t[e] - BIAS) * scale;
    store_sys(obase, (lane >> 3) * (NS * 4) + (lane & 7) * 16, __builtin_bit_cast(u32x4, v));
}

template <int R>
__device__ __forceinline__ unsigned send_subtile(const char* src, char* dst, char* far, int lane, bool paced, char* go, unsigned call) {
    const unsigned off = lane * 16;
    u32x4 t[4];
#pragma unroll
    for (int i = 0; i < 4; ++i) t[i] = *(const u32x4*)(src + i * 1024);
#pragma unroll
    for (int i = 0; i < 2; ++i) {
        const u32x4 lo = {__builtin_amdgcn_perm(t[2 * i][1], t[2 * i][0], 0x05040100u), __builtin_amdgcn_perm(t[2 * i][3], t[2 * i][2], 0x05040100u),
                          __builtin_amdgcn_perm(t[2 * i + 1][1], t[2 * i + 1][0], 0x05040100u), __builtin_amdgcn_perm(t[2 * i + 1][3], t[2 * i + 1][2], 0x05040100u)};
        if (R == 1 && paced) WAIT_VM(3);
        store_sys<1024>(dst, off, lo, i);
        PIN();
    }
    u32x4 hi[2];
#pragma unroll
    for (int i = 0; i < 2; ++i)
        hi[i] = u32x4{__builtin_amdgcn_perm(t[2 * i][1], t[2 * i][0], 0x07060302u), __builtin_amdgcn_perm(t[2 * i][3], t[2 * i][2], 0x07060302u),
                      __builtin_amdgcn_perm(t[2 * i + 1][1], t[2 * i + 1][0], 0x07060302u), __builtin_amdgcn_perm(t[2 * i + 1][3], t[2 * i + 1][2], 0x07060302u)};
    const u32x4 o = hi[0] | hi[1];
    const bool wide = __any(((o[0] | o[1] | o[2] | o[3]) & 0xffc0ffc0u) != 0);
    unsigned b[4];
#pragma unroll
    for (int i = 0; i < 4; ++i) b[i] = __builtin_amdgcn_perm(hi[i >> 1][(i & 1) * 2 + 1], hi[i >> 1][(i & 1) * 2], 0x06040200u);
    const unsigned h0 = b[0] | ((b[3] << 6) & 0xc0c0c0c0u), h1 = b[1] | ((b[3] << 4) & 0xc0c0c0c0u), h2 = b[2] | ((b[3] << 2) & 0xc0c0c0c0u);
    auto s1 = __builtin_amdgcn_permlane16_swap(h0, h1, false, false);
    auto s2 = __builtin_amdgcn_permlane16_swap(h2, h2, false, false);
    auto s3 = __builtin_amdgcn_permlane32_swap(s1[1], s2[1], false, false);
    const u32x4 h = {h0, h1, h2, s3[1]};
    if (paced) {
        if (R == 0) WAIT_VM(1); else WAIT_VM(3);
        if (R == 0 && go && lane == 0) set_flag(go, call, 0);
    }
    if (wide) {
        store_sys<1024>(far, off, hi[0], 0);
        store_sys<1024>(far, off, hi[1], 1);
    } else if (lane < 48) {
        store_sys<1024>(dst, off, h, 2);
    }
    return wide;
}
template <int D>
__device__ __forceinline__ void fetch_subtile(const char* base, int n, unsigned buf, bool raw, int lane) {
    const char* const src = base + n * SHORT_BYTES;
    WAIT_VM(D);
    glds16_sys(src, lane * 16, buf);
    WAIT_VM(D);
    glds16_sys(src + 1024, lane * 16, buf + 1024);
    WAIT_VM(D);
    glds16_sys(raw ? base + H_LONG + n * 2048 : src + 2048, min(lane, raw ? 63 : 47) * 16, buf + 2048);
    if (raw) glds16_sys(base + H_LONG + n * 2048 + 1024, lane * 16, buf + 3072);
}
template <int H>
__device__ __forceinline__ void finish_half(char* smem, unsigned buf, bool raw, float scale, char* obase, int lane) {
    const u32x4 lo = *(const u32x4*)(smem + buf + H * 1024 + lane * 16);
    const unsigned off = (lane >> 3) * (NS * 4) + (lane & 7) * 16;
    f32x4 v[2];
    if (raw) {
        const u32x4 hi = *(const u32x4*)(smem + buf + 2048 + H * 1024 + lane * 16);
#pragma unroll
        for (int e = 0; e < 4; ++e) {
            v[e >> 1][(e & 1) * 2] = (float)(int)(__builtin_amdgcn_perm(hi[e], lo[e], 0x05040100u) - BIAS) * scale;
            v[e >> 1][(e & 1) * 2 + 1] = (float)(int)(__builtin_amdgcn_perm(hi[e], lo[e], 0x07060302u) - BIAS) * scale;
        }
    } else {
        const unsigned ha = buf + 2048 + (lane < 48 ? lane * 16 : (lane - 48) * 16 + 12), hs = lane < 48 ? 4 : 256;
        const unsigned h0 = *(const unsigned*)(smem + ha), h1 = *(const unsigned*)(smem + ha + hs);
        unsigned b[2];
        if (H == 0) {
            b[0] = h0 & 0x3f3f3f3fu; b[1] = h1 & 0x3f3f3f3fu;
        } else {
            const unsigned h2 = *(const unsigned*)(smem + ha + 2 * hs);
            b[0] = h2 & 0x3f3f3f3fu;
            b[1] = ((h0 >> 6) & 0x03030303u) | ((h1 >> 4) & 0x0c0c0c0cu) | ((h2 >> 2) & 0x30303030u);
        }
#pragma unroll
        for (int i = 0; i < 2; ++i) {
            v[i][0] = ((float)__builtin_amdgcn_perm(b[i], lo[2 * i], 0x0c040100u) - (float)BIAS) * scale;
            v[i][1] = ((float)__builtin_amdgcn_perm(b[i], lo[2 * i], 0x0c050302u) - (float)BIAS) * scale;
            v[i][2] = ((float)__builtin_amdgcn_perm(b[i], lo[2 * i + 1], 0x0c060100u) - (float)BIAS) * scale;
            v[i][3] = ((float)__builtin_amdgcn_perm(b[i], lo[2 * i + 1], 0x0c070302u) - (float)BIAS) * scale;
        }
    }
    store_sys(obase + (2 * H) * 8 * (NS * 4), off, __builtin_bit_cast(u32x4, v[0]));
    store_sys(obase + (2 * H + 1) * 8 * (NS * 4), off, __builtin_bit_cast(u32x4, v[1]));
}

constexpr unsigned READY = 1u << 31, FAILED = 1u << 30;
__device__ __forceinline__ bool give_up(u64 t0, char* tmo, unsigned call) {
    const u64 dt = NOW() - t0;
    if (dt > 10000 && (dt > DEADLINE_TICKS || __hip_atomic_load((GLB_AS unsigned*)tmo, __ATOMIC_RELAXED, __HIP_MEMORY_SCOPE_AGENT) == call)) {
        __hip_atomic_store((GLB_AS unsigned*)tmo, call, __ATOMIC_RELAXED, __HIP_MEMORY_SCOPE_AGENT);
        return true;
    }
    return false;
}
__device__ __forceinline__ unsigned poll_flags(const char* flag, char* tmo, unsigned call) {
    const u64 t0 = NOW();
    while (true) {
        const u64 v = __hip_atomic_load((GLB_AS u64*)flag, __ATOMIC_RELAXED, __HIP_MEMORY_SCOPE_SYSTEM);
        if (__all((unsigned)(v >> 32) >= call)) return (unsigned)v & 0xffffu;
        if (give_up(t0, tmo, call)) return FAILED;
        __builtin_amdgcn_s_sleep(8);
    }
}

template <int PER, int NW, int STEP>
__device__ __forceinline__ void warm_l2(const char* wblk, const char* xf, int me, int lane) {
    const unsigned wo = (lane >> 2) * ND + (lane & 3) * 128;
#pragma unroll 1
    for (int n = me; n < 32 * PER; n += STEP) {
        const int kb = n / PER, j = n - kb * PER;
        const bool isw = j < NW;
        glds4(isw ? wblk + (size_t)(kb * 128 + j * 16) * ND : xf + (size_t)(j - NW) * 64 * KD + kb * 128, isw ? wo : (unsigned)lane * KD);
    }
}
__device__ __forceinline__ void advance(LDS_AS unsigned* ctl, char* counter, unsigned call, int lane) {
    asm volatile("" :: "s"(call) : "memory");
    if (lane == 0 && __hip_atomic_fetch_add(ctl + 12, 1u, __ATOMIC_RELAXED, __HIP_MEMORY_SCOPE_WORKGROUP) == 7)
        __hip_atomic_store((GLB_AS unsigned*)counter, call, __ATOMIC_RELAXED, __HIP_MEMORY_SCOPE_AGENT);
}

typedef const CST_AS Group* KArgs;
#define FRESH() KArgs kp = kp0; int b = b0, wid = wid0, lane = lane0; \
    asm volatile("" : "+s"(kp), "+s"(b), "+s"(wid) :: "memory"); asm volatile("" : "+v"(lane))
#define COUNTER(mine) ((mine) + H_CNT + (b >> 3) * 128)
#define CALL() (*(const CST_AS unsigned*)COUNTER(kp->mine) + 1u)
#define SCALE() (*(const CST_AS float*)kp->sx * *(const CST_AS float*)kp->sw)
#define TMO() (kp->mine - (size_t)kp->peer * BLK_BYTES + H_TMO)
#define cls (b >> 5)
#define pan ((b >> 3) & 3)
#define local ((b & 7) == LOCAL_GRP)
#define NIB(word) (((unsigned)(word) >> (4 * cls)) & 15)

__global__ void __launch_bounds__(BLOCK, 2) fused_kernel(Params) {
    extern __shared__ __attribute__((aligned(16))) char smem[];
    const KArgs kp = (KArgs)__builtin_amdgcn_kernarg_segment_ptr() + (blockIdx.x & 7), kp0 = kp;
    const int lane = threadIdx.x & 63, wid = __builtin_amdgcn_readfirstlane(threadIdx.x >> 6), b = blockIdx.x;
    const int lane0 = lane, wid0 = wid, b0 = b;
    const char *const x0 = kp->x, *const w0 = kp->wblk;
    char *const mine0 = kp->mine, *const theirs0 = kp->theirs, *const out0 = kp->out;
    const char *const sx0 = kp->sx, *const sw0 = kp->sw;
    const u64 tstart = NOW();
    PIN();
    if ((TLBW & 4) && wid == 6 && lane == 0 && b >= 32) {
        const int k = b >> 3;
        const char* p = k & 1 ? w0 : x0;
        if (k == 4) p = theirs0 + H_TMO + 128;
        if (k == 6) p = mine0 + H_TMO + 128;
        if (k >= 9 && k < 16) p = w0 + (size_t)(k - 8) * (2u << 20);
        glds4(p, 0);
    }
    LDS_AS unsigned* ctl = (LDS_AS unsigned*)(smem + CTL_OFF);
    const bool link = cls == 0 && !local;

    if (link) {
        constexpr int NP = AGU * 8;
        static_assert(AGD >= 2 && AGD <= 8, "staging ring");
        const unsigned agoff = (pan * 4 + (wid & 3)) * 1024 + lane * 16;
        unsigned done = *(const CST_AS unsigned*)COUNTER(mine0);
        PIN();
        if (wid < 4) {
#pragma unroll
            for (int i = 0; i < AGD; ++i) glds16(x0 + i * 16384, agoff, (wid * 8 + i) * 1024);
            if (lane < 4) glds4((TLBW & 1) ? theirs0 + H_TMO + 128 : x0, 0);
        }
        if (wid == 7) {
            ctl[lane] = 0;
            ctl[64 + lane] = 0;
            WAIT_LDS();
        }
        BARRIER();
        asm volatile("" : "+s"(done));
        const unsigned call = done + 1;
        char* const theirs = theirs0 + (call & 1) * HALF_BYTES;
        STAMP(1);
        if (wid < 4) {
            int r = 0;
            WAIT_VM(1);
#pragma unroll 1
            for (int i = 0; i < NP; ++i) {
                WAIT_VM(2 * AGD - 2);
                if (i >= AGD && ((i - AGD) & 7) == 7 && lane == 0)
                    __hip_atomic_fetch_add(ctl + 8 + ((i - AGD) >> 3), 1u, __ATOMIC_RELAXED, __HIP_MEMORY_SCOPE_WORKGROUP);
                const u32x4 v = *(const u32x4*)(smem + (wid * 8 + r) * 1024 + lane * 16);
                store_sys(theirs + H_X + i * 16384, agoff, v);
                glds16(x0 + (i + AGD < NP ? i + AGD : 0) * 16384, agoff, (wid * 8 + r) * 1024);
                r = r == AGD - 1 ? 0 : r + 1;
            }
            WAIT_VM(0);
            if (lane == 0) __hip_atomic_fetch_add(ctl + 8 + AGU - 1, 1u, __ATOMIC_RELAXED, __HIP_MEMORY_SCOPE_WORKGROUP);
            STAMP(3);
        }
        advance(ctl, COUNTER(mine0), call, lane);
        FRESH();
        const float scale = SCALE();
        LDS_AS u64* mirror = (LDS_AS u64*)(ctl + (MIR_OFF - CTL_OFF) / 4);
        auto flag_word = [&](int i) {
            const u64 v = __hip_atomic_load(mirror + i * 2, __ATOMIC_ACQUIRE, __HIP_MEMORY_SCOPE_WORKGROUP);
            const unsigned fail = __hip_atomic_load(ctl + 14, __ATOMIC_ACQUIRE, __HIP_MEMORY_SCOPE_WORKGROUP);
            const bool ok = (unsigned)(v >> 32) >= call && ((unsigned)v >> 16) == (call & 0xffffu);
            return (ok ? ((unsigned)v & 0xffffu) | READY : 0u) | fail;
        };
        if (wid == 7) {
            for (int u = 0; u < AGU; ++u) {
                while (__hip_atomic_load(ctl + 8 + u, __ATOMIC_RELAXED, __HIP_MEMORY_SCOPE_WORKGROUP) < 4) __builtin_amdgcn_s_sleep(2);
                if (lane == 0) set_flag(theirs + H_XFLAG + (u * 4 + pan) * 128, call, 0);
            }
        } else if (wid == 6) {
            const char* const flags = mine0 + (call & 1) * HALF_BYTES + H_FLAG;
            const unsigned fo = (((PULL_ORDER >> (lane & 12)) & 15) * 4 + (lane & 3)) * 16;
            while (__hip_atomic_load(ctl + 12, __ATOMIC_RELAXED, __HIP_MEMORY_SCOPE_WORKGROUP) < 8) __builtin_amdgcn_s_sleep(4);
            const u64 t0 = NOW();
            while (true) {
                WAIT_VM(7);
                if (lane < 16) glds16_sys(flags, fo, MIR_OFF);
                __builtin_amdgcn_s_sleep(12);
                if (__all(flag_word(lane & 15) & READY)) break;
                if (give_up(t0, TMO(), call)) {
                    __hip_atomic_store(ctl + 14, READY | FAILED, __ATOMIC_RELEASE, __HIP_MEMORY_SCOPE_WORKGROUP);
                    break;
                }
            }
        }
        constexpr int N0 = 4 * nib(R_CNT, nib(PULL_ORDER, 0)), N1 = N0 + 3 * (NPU - N0 / 4), LAST = (NPU - 1) * 4;
        int tp = 3, ej, ep;
        for (int p = 2; p >= 0; --p) if (tight(w0 + kp->wown + p * 128)) tp = p;
        const int em = min(lane, NPU * 4 - 1);
        if (em < N0) { ej = em >> 2; ep = em & 3; }
        else if (em < N1) { ej = (em - N0) / 3; ep = em - N0 - 3 * ej; ep += ep >= tp; ej += N0 / 4; }
        else { ej = em - N1 + N0 / 4; ep = tp; }
        const int ee = (int)(pull_table() >> (8 * ej)) & 255;
        const unsigned eout = ((AGU + (ee & 15)) * 32 * NS + ep * 128 + pan * 32) * 4;
        char* const out = kp->out;
        LDS_AS unsigned* landed = ctl + 16;
        if (wid < 4) {
            const int en = ((ee & 15) * 4 + ep) * 4 + pan, ef = (ee >> 6) * 4 + ep, eb = ((ee >> 4) & 3) * 4 + pan;
            while (!(flag_word(12 + pan) & READY)) __builtin_amdgcn_s_sleep(1);
            unsigned words = 0, marks = 0;
#pragma unroll 1
            for (int j = 0; j < NPU; ++j) {
                const int m = j * 4 + wid;
                unsigned word = __builtin_amdgcn_readlane(words, m);
                for (int k = 0; !(word & READY); ++k) {
                    if (k == 1) WAIT_VM(2);
                    else if (k == 2) WAIT_VM(1);
                    else if (k == 3) {
                        WAIT_VM(0);
                        if (lane == 0) __hip_atomic_store(landed + wid, marks | j, __ATOMIC_RELEASE, __HIP_MEMORY_SCOPE_WORKGROUP);
                    } else if (k) __builtin_amdgcn_s_sleep(1);
                    const unsigned w = flag_word(ef);
                    words = (w & (READY | FAILED)) | ((w >> eb) & 1);
                    word = __builtin_amdgcn_readlane(words, m);
                }
                marks |= ((word & 1) << 8 | (word & FAILED ? 1u << 16 : 0u)) << j;
                if (word & FAILED) WAIT_VM(0);
                else fetch_subtile<2>(theirs, __builtin_amdgcn_readlane(en, m), m * 4096, word & 1, lane);
                if (lane == 0) __hip_atomic_store(landed + wid, marks | j, __ATOMIC_RELEASE, __HIP_MEMORY_SCOPE_WORKGROUP);
            }
            char* o = out + __builtin_amdgcn_readlane(eout, LAST + wid);
            float sc = (marks >> (15 + NPU)) & 1 ? __builtin_nanf("") : scale;
            asm volatile("" : "+s"(o));
            asm volatile("" : "+v"(sc));
            WAIT_VM(0);
            if (lane == 0) __hip_atomic_store(landed + wid, marks | NPU, __ATOMIC_RELEASE, __HIP_MEMORY_SCOPE_WORKGROUP);
            STAMP(2);
            finish_half<0>(smem, (LAST + wid) * 4096, (marks >> (7 + NPU)) & 1, sc, o, lane);
        } else {
            for (bool more = true; more;) {
                unsigned m = 0;
                if (lane == 0) m = __hip_atomic_fetch_add(ctl + 13, 1u, __ATOMIC_RELAXED, __HIP_MEMORY_SCOPE_WORKGROUP);
                m = __builtin_amdgcn_readfirstlane(m);
                more = m < LAST;
                if (!more) m = LAST + wid - 4;
                char* o = out + __builtin_amdgcn_readlane(eout, m);
                asm volatile("" : "+s"(o));
                unsigned w;
                while (((w = __builtin_amdgcn_readfirstlane(__hip_atomic_load(landed + (m & 3), __ATOMIC_ACQUIRE, __HIP_MEMORY_SCOPE_WORKGROUP))) & 255) <= m >> 2)
                    if (more) __builtin_amdgcn_s_sleep(1);
                w >>= m >> 2;
                const float sc = (w >> 16) & 1 ? __builtin_nanf("") : scale;
                finish_half<1>(smem, m * 4096, (w >> 8) & 1, sc, o, lane);
                if (more) finish_half<0>(smem, m * 4096, (w >> 8) & 1, sc, o, lane);
            }
        }
    } else {
        int cnt = NIB(local ? L_CNT : R_CNT);
        if (wid == 7) {
            ctl[lane] = 0;
            WAIT_LDS();
        }
        if (((local ? LPF_MASK : PF_MASK) >> cls) & 1) {
            if (wid == 6) asm volatile("" :: "s"(CALL()), "v"(SCALE()));
            if ((wid >> 1) == 2) {
                const int me = (__builtin_popcount((local ? LPF_MASK : PF_MASK) & ((1 << cls) - 1)) * 4 + pan) * 2 + (wid & 1);
                if (local) {
                    warm_l2<8, 0, __builtin_popcount(LPF_MASK) * 8>(w0, x0, me, lane);
                    warm_l2<8, 8, __builtin_popcount(LPF_MASK) * 8>(w0, x0, me, lane);
                } else {
                    warm_l2<8 + PF_XU / 2, 8, __builtin_popcount(PF_MASK) * 8>(w0, x0 + AGU * 32 * KD, me, lane);
                    if ((unsigned)(me - PF2_LO * 2) < PF2_N * 2)
                        warm_l2<PF2_XU / 2, 0, PF2_N * 2>(w0, x0 + (AGU + PF2_FIRST) * 32 * KD, me - PF2_LO * 2, lane);
                }
            }
            BARRIER();
        } else if (local) {
            while (NOW() - tstart < L_START) __builtin_amdgcn_s_sleep(2);
        }
        int task = cnt ? 0 : 1;
        unsigned lastcall = 0;
        const int ntask = !local && NIB(A_CNT) ? 2 : 1;
#pragma unroll 1
        for (; task < ntask; ++task) {
            unsigned bad = 0, first = local ? NIB(L_FIRST) : AGU + NIB(R_FIRST);
            const char *wp1 = nullptr, *xp1 = nullptr;
            if (task == 1) {
                FRESH();
                const unsigned call = CALL();
                const char* const mine = kp->mine + (call & 1) * HALF_BYTES;
                wp1 = kp->wblk + kp->wown;
                xp1 = mine + H_X;
                const bool tp = tight(wp1 + pan * 128);
                cnt = NIB(tp ? T_CNT : A_CNT);
                first = NIB(tp ? T_FIRST : A_FIRST);
                if (wid == 1) {
                    if (!tp) while (NOW() - tstart < A_START) __builtin_amdgcn_s_sleep(4);
                    if (lane < 4) ctl[1] = poll_flags(mine + H_XFLAG + ((first + cnt - 1) * 4 + lane) * 128, TMO(), call);
                    WAIT_LDS();
                }
                BARRIER();
                bad = __builtin_amdgcn_readfirstlane(ctl[1]);
                STAMP(4);
            }
            FRESH();
            const int g = lane >> 4, q = (lane & 15) >> 1;
            unsigned woff = wid < 4 ? w_offset(lane, wid) + pan * 128
                                    : (16 * g + q) * 128 + (((2 * wid - 8) ^ (((q >> 1) & 3) << 1) ^ (g & 1)) << 4) + (lane & 1) * 8;
            unsigned xoff = wid < 4 ? x_offset(lane, wid) + first * 32 * KD : (lane & 15) * 128 + ((g ^ ((lane >> 1) & 7)) << 4) + 16384;
            const bool direct = local || task == 1, push = (R_PUSH >> cls) & 1;
            const unsigned n0 = ((first - AGU + ((wid ^ 4) >> 2)) * 4 + pan) * 4 + (wid & 3);
            unsigned ofar = H_LONG + n0 * 2048, ocnt = H_CNT + (b >> 3) * 128;
            unsigned odst = direct ? ((first * 32 + (wid & 1) * 16) * NS + pan * 128 + (wid >> 1) * 32) * 4 : n0 * SHORT_BYTES;
            unsigned ta = (lane >> 3) * 128 + (((lane & 7) ^ (lane >> 3)) << 4) + (direct ? wid * 2048 : (wid ^ 4) * 4096);
            int mode = direct ? 0 : (wid ^ 4) >= cnt * 4 ? 1 : (wid ^ 4) + 8 >= cnt * 4 ? 2 : 3;
            asm volatile("" : "+v"(woff), "+v"(xoff), "+v"(ta));
            asm volatile("" : "+s"(ofar), "+s"(ocnt), "+s"(odst), "+s"(mode));
            PIN();
            const char *wp = task ? wp1 : w0, *xp = task ? xp1 : x0;
            if (wid < 4 && !bad) {
#pragma unroll
                for (int j = 0; j < 4; ++j) glds16(wp + j * 8 * ND, j >= 2 ? woff ^ 16 : woff, wid * 4096 + j * 1024);
                glds16_sys(xp, xoff, 16384 + wid * 1024);
                if (cnt > 1) glds16_sys(xp + 32 * KD, xoff, 16384 + 4096 + wid * 1024);
                if (cnt > 2) glds16_sys(xp + 64 * KD, xoff, 16384 + 8192 + wid * 1024);
            }
            const u64 t0 = NOW();
            char* const counter = mine0 + ocnt;
            const char *sxp = sx0, *swp = sw0;
            asm volatile("" : "+s"(sxp), "+s"(swp));
            const unsigned done = *(const CST_AS unsigned*)counter;
            const float sx = *(const CST_AS float*)sxp, sw = *(const CST_AS float*)swp;
            char* far = (push ? theirs0 : mine0) + ofar;
            char* dst = (mode == 0 ? out0 : push ? theirs0 : mine0) + odst;
            char* flags = theirs0;
            asm volatile("" : "+s"(far), "+s"(dst), "+s"(flags));
            if (bad) {
            } else if (wid < 4) {
                if (cnt == 1) load_tile<1, 3>(wp, xp, woff, xoff, wid);
                else if (cnt == 2) load_tile<2, 3>(wp, xp, woff, xoff, wid);
                else load_tile<3, 2>(wp, xp, woff, xoff, wid);
                if (mode && push) while (NOW() - t0 < HOLD0 + HOLD1 * (first - AGU)) __builtin_amdgcn_s_sleep(1);
            } else if (cnt == 1) tile_compute<1>(smem, lane, wid - 4, woff, xoff, done, sx, sw);
            else if (cnt == 2) tile_compute<2>(smem, lane, wid - 4, woff, xoff, done, sx, sw);
            else tile_compute<3>(smem, lane, wid - 4, woff, xoff, done, sx, sw);
            const unsigned call = done + 1, half = (call & 1) * HALF_BYTES;
            lastcall = call;
            if (mode) { dst += half; far += half; flags += half + H_FLAG; }
            asm volatile("" : "+s"(far), "+s"(dst), "+s"(flags));
            BARRIER();
            if (mode == 0) {
                const float scale = bad ? __builtin_nanf("") : sx * sw;
#pragma unroll 1
                for (int u = 0; u < cnt; ++u) {
                    const u32x4 t0 = *(const u32x4*)(smem + ta + u * 16384), t1 = *(const u32x4*)(smem + ta + u * 16384 + 1024);
                    write_rows(t0, scale, dst + u * (32 * NS * 4), lane);
                    write_rows(t1, scale, dst + u * (32 * NS * 4) + 8 * NS * 4, lane);
                }
            } else {
                unsigned marks = 0;
                if (mode > 1)
                    marks = send_subtile<0>(smem + ta, dst, far, lane, push && cnt > 1,
                                            cls == GO_CLS && (wid ^ 4) == GO_ITEM ? flags + pan * 16 : nullptr, call);
                if (mode > 2) marks |= send_subtile<1>(smem + ta + 32768, dst + 32 * SHORT_BYTES, far + 32 * 2048, lane, push, nullptr, call) << 8;
                if (marks && lane == 0) __hip_atomic_fetch_or(ctl, marks << (wid ^ 4), __ATOMIC_RELAXED, __HIP_MEMORY_SCOPE_WORKGROUP);
                flags += (b >> 3) * 16;
                asm volatile("" : "+s"(flags));
                asm volatile("s_waitcnt vmcnt(0) lgkmcnt(0)" ::: "memory");
                BARRIER();
                STAMP(3);
                if (wid == 7 && lane == 0) set_flag(flags, call, ctl[0]);
            }
            if (task + 1 == ntask) advance(ctl, counter, call, lane);
        }
        FRESH();
        if (!lastcall) advance(ctl, COUNTER(kp->mine), CALL(), lane);
        if (!local) {
            const char* const mine = kp->mine + (lastcall & 1) * HALF_BYTES;
            char* const out = kp->out + pan * 128 * 4;
            const float scale = SCALE();
#pragma unroll 1
            for (int c = 1; c < 8; ++c) {
                if (((M_CLS >> (4 * c)) & 15) != cls) continue;
                BARRIER();
                if (wid == 1) {
                    if (lane == 0) ctl[1] = poll_flags(mine + H_FLAG + (c * 4 + pan) * 16, TMO(), lastcall);
                    WAIT_LDS();
                }
                BARRIER();
                const unsigned word = __builtin_amdgcn_readfirstlane(ctl[1]);
                const float sc = word & FAILED ? __builtin_nanf("") : scale;
                for (int i = wid; i < ((R_CNT >> (4 * c)) & 15) * 4; i += 8) {
                    const int u = ((R_FIRST >> (4 * c)) & 15) + (i >> 2);
                    if (!(word & FAILED)) fetch_subtile<63>(mine, (u * 4 + pan) * 4 + (i & 3), i * 4096, (word >> i) & 1, lane);
                    WAIT_VM(0);
                    char* const o = out + ((AGU + u) * 32 * NS + (i & 3) * 32) * 4;
                    finish_half<0>(smem, i * 4096, (word >> i) & 1, sc, o, lane);
                    finish_half<1>(smem, i * 4096, (word >> i) & 1, sc, o, lane);
                }
            }
        }
    }
}
#undef cls
#undef pan
#undef local

extern "C" void kernel_launch(int rank, int nranks, ncclComm_t comm,
                              hipStream_t stream,
                              void* const* d_in, const int* in_sizes, int n_in,
                              void* d_out, int out_size,
                              const int* in_dtypes, int out_dtype,
                              void* const* d_out_all,
                              void* const* d_ws_all, size_t ws_bytes) {
    if (nranks != NR || (unsigned)rank >= (unsigned)NR || ws_bytes < WS_END || n_in != 4 || in_sizes[0] != MS * KD
        || in_sizes[1] != KD * ND || out_size != NR * MS * NS) return;
    static bool init[NR] = {};
    if (!init[rank]) {
        HIP_CHECK(hipFuncSetAttribute((const void*)fused_kernel, hipFuncAttributeMaxDynamicSharedMemorySize, LDS_BYTES));
        init[rank] = true;
    }
    static_assert(LOCAL_GRP == 3, "three peers in the own quad");
    int peer[NR], lo = 0, hi = 4;
    const unsigned quad = ((0x4d >> rank) & 1) ? 0x4du : 0xb2u;
    for (int i = 1; i < NR; ++i) {
        const int q = (rank + i) & 7;
        if ((quad >> q) & 1) peer[lo++] = q; else peer[hi++] = q;
    }
    peer[LOCAL_GRP] = rank;
    Params p;
    for (int g = 0; g < NR; ++g) {
        p.g[g].x = (const char*)d_in[0];
        p.g[g].wblk = (const char*)d_in[1] + peer[g] * NS;
        p.g[g].mine = (char*)d_ws_all[rank] + (size_t)peer[g] * BLK_BYTES;
        p.g[g].theirs = (char*)d_ws_all[peer[g]] + (size_t)rank * BLK_BYTES;
        p.g[g].out = (char*)d_out + (size_t)peer[g] * MS * NS * 4;
        p.g[g].sx = (const char*)d_in[2];
        p.g[g].sw = (const char*)d_in[3];
        p.g[g].wown = (rank - peer[g]) * NS;
        p.g[g].peer = peer[g];
    }
    hipLaunchKernelGGL(fused_kernel, dim3(GRID), dim3(BLOCK), LDS_BYTES, stream, p);
    HIP_CHECK(hipGetLastError());
}
```

```cpp
#include <hip/hip_runtime.h>
#include <rccl/rccl.h>
#include <cstdio>

#define HIP_CHECK(expr) do { hipError_t _e = (expr); if (_e != hipSuccess) { \
    fprintf(stderr, "HIP error %s at %s:%d\n", hipGetErrorString(_e), __FILE__, __LINE__); \
    return; } } while (0)

typedef int v2i __attribute__((ext_vector_type(2)));
typedef int v4i __attribute__((ext_vector_type(4)));
typedef unsigned u32x4 __attribute__((ext_vector_type(4)));
typedef float f32x4 __attribute__((ext_vector_type(4)));
typedef unsigned long long u64;
#define LDS_AS __attribute__((address_space(3)))
#define GLB_AS __attribute__((address_space(1)))
#define CST_AS __attribute__((address_space(4)))

constexpr int AGU = 3;
constexpr unsigned R_CNT = 0x33210220;
constexpr unsigned R_FIRST = 0x63100b90;
constexpr unsigned R_PUSH = 0x70;
constexpr unsigned PULL_ORDER = 0x127;
constexpr unsigned M_CLS = 0x06570000;
constexpr unsigned A_CNT = 0x00012000;
constexpr unsigned A_FIRST = 0x00020000;
constexpr unsigned T_CNT = 0x00021000;
constexpr unsigned T_FIRST = 0x00010000;
constexpr unsigned PF_MASK = 0x0e;
constexpr int PF_XU = 10;
constexpr int PF2_FIRST = 9, PF2_XU = 4, PF2_LO = 8, PF2_N = 3;
constexpr int GO_CLS = 6, GO_ITEM = 7;
constexpr int LOCAL_GRP = 3;
constexpr unsigned L_CNT = 0x33330022;
constexpr unsigned L_FIRST = 0x963000ec;
constexpr unsigned LPF_MASK = 0x0f;
#define L_START 256
#define A_START 930
#define HOLD0 655
#define HOLD1 78
#define AGD 4
#define CODEW 1
constexpr int CODE_BYTES = 18432, CODE_PIECES = 3;
#define TLBW 5

constexpr int NR = 8, MS = 512, KD = 4096, ND = 4096, NS = 512;
constexpr int GRID = 256, BLOCK = 512;
constexpr int NSLOT = 4;
constexpr int SLOT_BYTES = 32768;
constexpr int CTL_OFF = NSLOT * SLOT_BYTES;
constexpr int MIR_OFF = CTL_OFF + 256;
constexpr int JUNK_OFF = MIR_OFF + 256;
constexpr int LDS_BYTES = JUNK_OFF + 256;
constexpr int BIAS = 1 << 21;
constexpr u64 DEADLINE_TICKS = 40000000ull;

constexpr unsigned BLK_BYTES = 2u << 20, HALF_BYTES = 1u << 20;
constexpr int SHORT_BYTES = 2816;
constexpr unsigned H_X = 144 * 4096;
constexpr unsigned H_FLAG = H_X + AGU * 32 * KD;
constexpr unsigned H_XFLAG = H_FLAG + 1024;
constexpr unsigned H_CNT = H_FLAG + 4096;
constexpr unsigned H_TMO = H_CNT + 4096;
constexpr unsigned H_LONG = NR * BLK_BYTES;
constexpr size_t WS_END = 2 * (size_t)NR * BLK_BYTES;
static_assert((16 - AGU) * 16 * SHORT_BYTES <= H_X && H_XFLAG + AGU * 4 * 128 <= H_CNT && H_TMO + 256 <= HALF_BYTES, "layout");
static_assert(PF_XU % 2 == 0 && AGU + PF_XU <= 16 && PF2_XU % 2 == 0 && AGU + PF2_FIRST + PF2_XU <= 16, "warm-up rows");
constexpr int nib(unsigned w, int c) { return (w >> 4 * c) & 15; }
constexpr bool plan_ok() {
    unsigned r = 0, a = 0, t = 0, l = 0;
    for (int c = 0; c < 8; ++c) {
        const int rc = nib(R_CNT, c), ac = nib(A_CNT, c), tc = nib(T_CNT, c), lc = nib(L_CNT, c);
        const bool push = (R_PUSH >> c) & 1;
        bool pulled = false;
        for (int i = 0; i < 3; ++i) pulled |= nib(PULL_ORDER, i) == c;
        if (rc > 3 || ac > 3 || tc > 3 || lc > 3 || !tc != !ac) return false;
        if (rc ? pulled == push : pulled || push) return false;
        if ((nib(M_CLS, c) != 0) != push || nib(M_CLS, c) > 7) return false;
        if ((c && !rc && !ac) || (!lc && !((LPF_MASK >> c) & 1))) return false;
        r += ((1u << rc) - 1) << nib(R_FIRST, c);
        a += ((1u << ac) - 1) << nib(A_FIRST, c);
        t += ((1u << tc) - 1) << nib(T_FIRST, c);
        l += ((1u << lc) - 1) << nib(L_FIRST, c);
    }
    return r == (1u << (16 - AGU)) - 1 && a == (1u << AGU) - 1 && t == a && l == 0xffffu && !nib(A_CNT, 0) && !(PF_MASK & 1)
           && ((R_PUSH >> GO_CLS) & 1) && nib(R_CNT, GO_CLS) > 1 && GO_ITEM < 8;
}
static_assert(plan_ok(), "the tiles must cover every unit once");
constexpr int NPU = nib(R_CNT, nib(PULL_ORDER, 0)) + nib(R_CNT, nib(PULL_ORDER, 1)) + nib(R_CNT, nib(PULL_ORDER, 2));
constexpr u64 pull_table() {
    u64 v = 0; int n = 0;
    for (int t = 0; t < 3; ++t)
        for (int u = 0; u < nib(R_CNT, nib(PULL_ORDER, t)); ++u) v |= (u64)(t << 6 | u << 4 | (nib(R_FIRST, nib(PULL_ORDER, t)) + u)) << 8 * n++;
    return v;
}
static_assert(NPU <= 8 && NPU * 16384 <= CTL_OFF, "pull buffers");

#define STAMP(i) do { } while (0)

struct Group {
    const char* x;
    const char* wblk;
    char* mine;
    char* theirs;
    char* out;
    const char* sx;
    const char* sw;
    int wown;
    int peer;
};
struct Params { Group g[NR]; };
static_assert(sizeof(Group) == 64, "one line");

#define WAIT_LDS() __builtin_amdgcn_s_waitcnt(0xC07F)
#define WAIT_VM(n) asm volatile("s_waitcnt vmcnt(%0)" :: "n"(n) : "memory")
#define PIN() __builtin_amdgcn_sched_barrier(0)
#define BARRIER() do { asm volatile("" ::: "memory"); __builtin_amdgcn_s_barrier(); asm volatile("" ::: "memory"); } while (0)
#define NOW() __builtin_amdgcn_s_memrealtime()
#define RARE(c) __builtin_expect(!!(c), 0)

__device__ __forceinline__ void glds16(const char* sbase, unsigned voff, unsigned lds) {
    asm volatile("s_mov_b32 m0, %2\n\ts_nop 0\n\tglobal_load_lds_dwordx4 %0, %1" :: "v"(voff), "s"(sbase), "s"(lds) : "memory");
}
__device__ __forceinline__ void glds16_sys(const char* sbase, unsigned voff, unsigned lds) {
    asm volatile("s_mov_b32 m0, %2\n\ts_nop 0\n\tglobal_load_lds_dwordx4 %0, %1 sc0 sc1" :: "v"(voff), "s"(sbase), "s"(lds) : "memory");
}
__device__ __forceinline__ void glds4(const char* sbase, unsigned voff) {
    asm volatile("s_mov_b32 m0, %2\n\ts_nop 0\n\tglobal_load_lds_dword %0, %1" :: "v"(voff), "s"(sbase), "s"(JUNK_OFF) : "memory");
}
__device__ __forceinline__ void store_sys(char* sbase, unsigned voff, u32x4 v) {
    asm volatile("global_store_dwordx4 %0, %1, %2 sc0 sc1\n\ts_nop 1" :: "v"(voff), "v"(v), "s"(sbase) : "memory");
}
template <int STEP>
__device__ __forceinline__ void store_sys(char* sbase, unsigned voff, u32x4 v, int i) {
    if (i == 0) store_sys(sbase, voff, v);
    else if (i == 1) asm volatile("global_store_dwordx4 %0, %1, %2 offset:%3 sc0 sc1\n\ts_nop 1" :: "v"(voff), "v"(v), "s"(sbase), "n"(STEP) : "memory");
    else asm volatile("global_store_dwordx4 %0, %1, %2 offset:%3 sc0 sc1\n\ts_nop 1" :: "v"(voff), "v"(v), "s"(sbase), "n"(2 * STEP) : "memory");
}
__device__ __forceinline__ void set_flag(char* flag, unsigned call, unsigned bits) {
    __hip_atomic_store((GLB_AS u64*)flag, ((u64)call << 32) | (call << 16) | bits, __ATOMIC_RELAXED, __HIP_MEMORY_SCOPE_SYSTEM);
}

__device__ __forceinline__ unsigned w_offset(int lane, int wid) {
    return (unsigned)(wid * 32 + (lane >> 3)) * ND + (((lane & 7) ^ (((lane >> 4) & 3) << 1)) << 4);
}
__device__ __forceinline__ unsigned x_offset(int lane, int wid) {
    return (unsigned)(wid * 8 + (lane >> 3)) * KD + (((lane & 7) ^ ((wid * 4 + (lane >> 4)) & 7)) << 4);
}
template <int U, int LA>
__device__ __forceinline__ void load_tile(const char* wp, const char* xp, unsigned woff, unsigned xoff, int wid) {
    static_assert(LA < NSLOT, "ring too small");
    unsigned slot = SLOT_BYTES;
#pragma unroll 1
    for (int s = 1; s < 32; ++s) {
        if (s >= LA) {
            WAIT_VM((LA - 1) * (4 + U));
            BARRIER();
        }
#pragma unroll
        for (int j = 0; j < 4; ++j) glds16(wp + (size_t)(s * 128 + j * 8) * ND, j >= 2 ? woff ^ 16 : woff, slot + wid * 4096 + j * 1024);
#pragma unroll
        for (int u = 0; u < U; ++u) glds16_sys(xp + u * 32 * KD + s * 128, xoff, slot + 16384 + u * 4096 + wid * 1024);
        slot = (slot + SLOT_BYTES) & (NSLOT * SLOT_BYTES - 1);
    }
#pragma unroll
    for (int r = LA - 1; r >= 0; --r) {
        WAIT_VM(r * (4 + U));
        BARRIER();
    }
}

template <int U> struct Frag { v2i w[4][2]; v4i x[U][4]; };
struct Addr { unsigned w0, w1, x0, x1; };

template <int U>
__device__ __forceinline__ void read_frag(Frag<U>& f, char* smem, const Addr& a, int r) {
    const int kk = r / (2 + U), i = r % (2 + U);
    if (i < 2) f.w[kk][i] = __builtin_amdgcn_ds_read_tr8_b64_v2i32((LDS_AS v2i*)(smem + (kk & 1 ? a.w1 : a.w0) + (kk >> 1) * 8192 + i * 1024));
    else f.x[i - 2][kk] = *(const v4i*)(smem + (kk >> 1 ? a.x1 : a.x0) + (kk & 1) * 2048 + (i - 2) * 4096);
}
template <int U>
__device__ __forceinline__ void mfma(const Frag<U>& f, v4i (&acc)[U][4], int m) {
    const int k2 = m / (4 * U), u = m / 4 % U, i = m % 4;
    const v2i lo = f.w[k2 * 2 + (i >> 1)][0], hi = f.w[k2 * 2 + (i >> 1)][1];
    asm volatile("v_mfma_i32_16x16x64_i8 %0, %1, %2, %0" : "+v"(acc[u][i]) : "v"(v4i{lo.x, lo.y, hi.x, hi.y}), "v"(f.x[u][k2 * 2 + (i & 1)]));
}
template <int U>
__device__ __forceinline__ void stage(const Frag<U>& cur, Frag<U>& nxt, v4i (&acc)[U][4], char* smem, const Addr& a) {
    constexpr int NRD = 4 * (2 + U), PER = U == 1 ? 2 : 1;
#pragma unroll
    for (int m = 0; m < 8 * U; ++m) {
        mfma<U>(cur, acc, m);
#pragma unroll
        for (int r = m * PER; r < (m + 1) * PER && r < NRD; ++r) read_frag<U>(nxt, smem, a, r);
        PIN();
    }
}
template <int U>
__device__ __forceinline__ void tile_compute(char* smem, int lane, int cw, unsigned keep0, float keep1, float keep2) {
    const int g = lane >> 4, q = (lane & 15) >> 1;
    const unsigned wad = (16 * g + q) * 128 + (((2 * cw) ^ (((q >> 1) & 3) << 1) ^ (g & 1)) << 4) + (lane & 1) * 8;
    const unsigned xad = (lane & 15) * 128 + ((g ^ ((lane >> 1) & 7)) << 4) + 16384;
    Addr a = {wad, wad ^ 16, xad, xad ^ 64};
    auto next = [&]() {
        constexpr unsigned M = NSLOT * SLOT_BYTES - 1;
        a.w0 = (a.w0 + SLOT_BYTES) & M; a.w1 = a.w0 ^ 16; a.x0 = (a.x0 + SLOT_BYTES) & M; a.x1 = a.x0 ^ 64;
        PIN();
    };
    v4i acc[U][4];
#pragma unroll
    for (int u = 0; u < U; ++u)
#pragma unroll
        for (int i = 0; i < 4; ++i) acc[u][i] = v4i{BIAS, BIAS, BIAS, BIAS};
    Frag<U> fa, fb;
    BARRIER();
#pragma unroll
    for (int r = 0; r < 4 * (2 + U); ++r) read_frag<U>(fa, smem, a, r);
    asm volatile("" :: "s"(keep0), "s"(keep1), "s"(keep2));
#pragma unroll 1
    for (int s = 1; s < 31; s += 2) {
        next();
        WAIT_LDS();
        BARRIER();
        stage<U>(fa, fb, acc, smem, a);
        next();
        WAIT_LDS();
        BARRIER();
        stage<U>(fb, fa, acc, smem, a);
    }
    next();
    WAIT_LDS();
    BARRIER();
    stage<U>(fa, fb, acc, smem, a);
    char* const d0 = smem + cw * 4096 + (lane & 15) * 128 + ((g ^ (lane & 7)) << 4);
    char* const d1 = smem + cw * 4096 + (lane & 15) * 128 + (((4 + g) ^ (lane & 7)) << 4);
#pragma unroll
    for (int u = 0; u <= U; ++u) {
        if (u == U) asm volatile("s_nop 7" ::: "memory");
#pragma unroll
        for (int i = 0; i < 8; ++i) {
            if (u < U) mfma<U>(fb, acc, (i >> 2) * 4 * U + u * 4 + (i & 3));
            if (u > 0 && i >= 4) *(v4i*)((i & 2 ? d1 : d0) + (u - 1) * 16384 + (i & 1) * 2048) = acc[u - 1][i & 3];
            PIN();
        }
    }
    WAIT_LDS();
}
__device__ __forceinline__ bool tight(const char* p) { return (((unsigned)(size_t)p >> 7) & 7) == 3 || (((unsigned)(size_t)p >> 7) & 31) == 31; }

__device__ __forceinline__ void write_rows(const u32x4& t, float scale, char* obase, int lane) {
    f32x4 v;
#pragma unroll
    for (int e = 0; e < 4; ++e) v[e] = (float)(int)(t[e] - BIAS) * scale;
    store_sys(obase, (lane >> 3) * (NS * 4) + (lane & 7) * 16, __builtin_bit_cast(u32x4, v));
}

template <int R>
__device__ __forceinline__ unsigned send_subtile(const char* src, char* dst, char* far, int lane, bool paced, char* go, unsigned call) {
    const unsigned off = lane * 16;
    u32x4 t[4];
#pragma unroll
    for (int i = 0; i < 4; ++i) t[i] = *(const u32x4*)(src + i * 1024);
#pragma unroll
    for (int i = 0; i < 2; ++i) {
        const u32x4 lo = {__builtin_amdgcn_perm(t[2 * i][1], t[2 * i][0], 0x05040100u), __builtin_amdgcn_perm(t[2 * i][3], t[2 * i][2], 0x05040100u),
                          __builtin_amdgcn_perm(t[2 * i + 1][1], t[2 * i + 1][0], 0x05040100u), __builtin_amdgcn_perm(t[2 * i + 1][3], t[2 * i + 1][2], 0x05040100u)};
        if (R == 1 && paced) WAIT_VM(3);
        store_sys<1024>(dst, off, lo, i);
        PIN();
    }
    u32x4 hi[2];
#pragma unroll
    for (int i = 0; i < 2; ++i)
        hi[i] = u32x4{__builtin_amdgcn_perm(t[2 * i][1], t[2 * i][0], 0x07060302u), __builtin_amdgcn_perm(t[2 * i][3], t[2 * i][2], 0x07060302u),
                      __builtin_amdgcn_perm(t[2 * i + 1][1], t[2 * i + 1][0], 0x07060302u), __builtin_amdgcn_perm(t[2 * i + 1][3], t[2 * i + 1][2], 0x07060302u)};
    const u32x4 o = hi[0] | hi[1];
    const bool wide = __any(((o[0] | o[1] | o[2] | o[3]) & 0xffc0ffc0u) != 0);
    unsigned b[4];
#pragma unroll
    for (int i = 0; i < 4; ++i) b[i] = __builtin_amdgcn_perm(hi[i >> 1][(i & 1) * 2 + 1], hi[i >> 1][(i & 1) * 2], 0x06040200u);
    const unsigned h0 = b[0] | ((b[3] << 6) & 0xc0c0c0c0u), h1 = b[1] | ((b[3] << 4) & 0xc0c0c0c0u), h2 = b[2] | ((b[3] << 2) & 0xc0c0c0c0u);
    auto s1 = __builtin_amdgcn_permlane16_swap(h0, h1, false, false);
    auto s2 = __builtin_amdgcn_permlane16_swap(h2, h2, false, false);
    auto s3 = __builtin_amdgcn_permlane32_swap(s1[1], s2[1], false, false);
    const u32x4 h = {h0, h1, h2, s3[1]};
    if (paced) {
        if (R == 0) WAIT_VM(1); else WAIT_VM(3);
        if (R == 0 && go && lane == 0) set_flag(go, call, 0);
    }
    if (wide) {
        store_sys<1024>(far, off, hi[0], 0);
        store_sys<1024>(far, off, hi[1], 1);
    } else if (lane < 48) {
        store_sys<1024>(dst, off, h, 2);
    }
    return wide;
}
template <int D>
__device__ __forceinline__ void fetch_subtile(const char* base, int n, unsigned buf, bool raw, int lane) {
    const char* const src = base + n * SHORT_BYTES;
    WAIT_VM(D);
    glds16_sys(src, lane * 16, buf);
    WAIT_VM(D);
    glds16_sys(src + 1024, lane * 16, buf + 1024);
    WAIT_VM(D);
    glds16_sys(raw ? base + H_LONG + n * 2048 : src + 2048, min(lane, raw ? 63 : 47) * 16, buf + 2048);
    if (raw) glds16_sys(base + H_LONG + n * 2048 + 1024, lane * 16, buf + 3072);
}
template <int H>
__device__ __forceinline__ void finish_half(char* smem, unsigned buf, bool raw, float scale, char* obase, int lane) {
    const u32x4 lo = *(const u32x4*)(smem + buf + H * 1024 + lane * 16);
    const unsigned off = (lane >> 3) * (NS * 4) + (lane & 7) * 16;
    f32x4 v[2];
    if (raw) {
        const u32x4 hi = *(const u32x4*)(smem + buf + 2048 + H * 1024 + lane * 16);
#pragma unroll
        for (int e = 0; e < 4; ++e) {
            v[e >> 1][(e & 1) * 2] = (float)(int)(__builtin_amdgcn_perm(hi[e], lo[e], 0x05040100u) - BIAS) * scale;
            v[e >> 1][(e & 1) * 2 + 1] = (float)(int)(__builtin_amdgcn_perm(hi[e], lo[e], 0x07060302u) - BIAS) * scale;
        }
    } else {
        const unsigned ha = buf + 2048 + (lane < 48 ? lane * 16 : (lane - 48) * 16 + 12), hs = lane < 48 ? 4 : 256;
        const unsigned h0 = *(const unsigned*)(smem + ha), h1 = *(const unsigned*)(smem + ha + hs);
        unsigned b[2];
        if (H == 0) {
            b[0] = h0 & 0x3f3f3f3fu; b[1] = h1 & 0x3f3f3f3fu;
        } else {
            const unsigned h2 = *(const unsigned*)(smem + ha + 2 * hs);
            b[0] = h2 & 0x3f3f3f3fu;
            b[1] = ((h0 >> 6) & 0x03030303u) | ((h1 >> 4) & 0x0c0c0c0cu) | ((h2 >> 2) & 0x30303030u);
        }
#pragma unroll
        for (int i = 0; i < 2; ++i) {
            v[i][0] = ((float)__builtin_amdgcn_perm(b[i], lo[2 * i], 0x0c040100u) - (float)BIAS) * scale;
            v[i][1] = ((float)__builtin_amdgcn_perm(b[i], lo[2 * i], 0x0c050302u) - (float)BIAS) * scale;
            v[i][2] = ((float)__builtin_amdgcn_perm(b[i], lo[2 * i + 1], 0x0c060100u) - (float)BIAS) * scale;
            v[i][3] = ((float)__builtin_amdgcn_perm(b[i], lo[2 * i + 1], 0x0c070302u) - (float)BIAS) * scale;
        }
    }
    store_sys(obase + (2 * H) * 8 * (NS * 4), off, __builtin_bit_cast(u32x4, v[0]));
    store_sys(obase + (2 * H + 1) * 8 * (NS * 4), off, __builtin_bit_cast(u32x4, v[1]));
}

constexpr unsigned READY = 1u << 31, FAILED = 1u << 30;
__device__ __forceinline__ bool give_up(u64 t0, char* tmo, unsigned call) {
    const u64 dt = NOW() - t0;
    if (dt > 10000 && (dt > DEADLINE_TICKS || __hip_atomic_load((GLB_AS unsigned*)tmo, __ATOMIC_RELAXED, __HIP_MEMORY_SCOPE_AGENT) == call)) {
        __hip_atomic_store((GLB_AS unsigned*)tmo, call, __ATOMIC_RELAXED, __HIP_MEMORY_SCOPE_AGENT);
        return true;
    }
    return false;
}
__device__ __forceinline__ unsigned poll_flags(const char* flag, char* tmo, unsigned call) {
    const u64 t0 = NOW();
    while (true) {
        const u64 v = __hip_atomic_load((GLB_AS u64*)flag, __ATOMIC_RELAXED, __HIP_MEMORY_SCOPE_SYSTEM);
        if (__all((unsigned)(v >> 32) >= call)) return (unsigned)v & 0xffffu;
        if (give_up(t0, tmo, call)) return FAILED;
        __builtin_amdgcn_s_sleep(8);
    }
}

template <int PER, int NW, int STEP>
__device__ __forceinline__ void warm_l2(const char* wblk, const char* xf, int me, int lane) {
    const unsigned wo = (lane >> 2) * ND + (lane & 3) * 128;
#pragma unroll 1
    for (int n = me; n < 32 * PER; n += STEP) {
        const int kb = n / PER, j = n - kb * PER;
        const bool isw = j < NW;
        glds4(isw ? wblk + (size_t)(kb * 128 + j * 16) * ND : xf + (size_t)(j - NW) * 64 * KD + kb * 128, isw ? wo : (unsigned)lane * KD);
    }
}
__device__ __forceinline__ void advance(LDS_AS unsigned* ctl, char* counter, unsigned call, int lane) {
    asm volatile("" :: "s"(call) : "memory");
    if (lane == 0 && __hip_atomic_fetch_add(ctl + 12, 1u, __ATOMIC_RELAXED, __HIP_MEMORY_SCOPE_WORKGROUP) == 7)
        __hip_atomic_store((GLB_AS unsigned*)counter, call, __ATOMIC_RELAXED, __HIP_MEMORY_SCOPE_AGENT);
}

typedef const CST_AS Group* KArgs;
#define FRESH() KArgs kp = kp0; int b = b0, wid = wid0, lane = lane0; \
    asm volatile("" : "+s"(kp), "+s"(b), "+s"(wid) :: "memory"); asm volatile("" : "+v"(lane))
#define COUNTER(mine) ((mine) + H_CNT + (b >> 3) * 128)
#define CALL() (*(const CST_AS unsigned*)COUNTER(kp->mine) + 1u)
#define SCALE() (*(const CST_AS float*)kp->sx * *(const CST_AS float*)kp->sw)
#define TMO() (kp->mine - (size_t)kp->peer * BLK_BYTES + H_TMO)
#define cls (b >> 5)
#define pan ((b >> 3) & 3)
#define local ((b & 7) == LOCAL_GRP)
#define NIB(word) (((unsigned)(word) >> (4 * cls)) & 15)

__global__ void __launch_bounds__(BLOCK, 2) fused_kernel(Params) {
    extern __shared__ __attribute__((aligned(16))) char smem[];
    const KArgs kp = (KArgs)__builtin_amdgcn_kernarg_segment_ptr() + (blockIdx.x & 7), kp0 = kp;
    const int lane = threadIdx.x & 63, wid = __builtin_amdgcn_readfirstlane(threadIdx.x >> 6), b = blockIdx.x;
    const int lane0 = lane, wid0 = wid, b0 = b;
    const char *const x0 = kp->x, *const w0 = kp->wblk;
    char *const mine0 = kp->mine, *const theirs0 = kp->theirs;
    const u64 tstart = NOW();
    if (CODEW) {
        u64 pc;
        asm volatile("s_cmp_lt_u32 %2, %3\n\ts_cbranch_scc0 .Lnocode%=\n\ts_getpc_b64 %0\n\ts_andn2_b64 %0, %0, 0xff\n\ts_mov_b32 m0, %4\n\ts_nop 0\n\t"
                     "global_load_lds_dword %1, %0\n.Lnocode%=:"
                     : "=&s"(pc) : "v"(min((b >> 3) * 8192 + lane * 128, CODE_BYTES - 4)), "s"(b + (7 - wid) * 256), "n"(8 * CODE_PIECES), "n"(JUNK_OFF)
                     : "scc", "memory");
    }
    PIN();
    ((LDS_AS u64*)(smem + CTL_OFF))[lane] = 0;
    LDS_AS unsigned* ctl = (LDS_AS unsigned*)(smem + CTL_OFF);
    const bool link = cls == 0 && !local;

    if (link) {
        constexpr int NP = AGU * 8;
        static_assert(AGD >= 2 && AGD <= 8, "staging ring");
        const unsigned agoff = (pan * 4 + (wid & 3)) * 1024 + lane * 16;
        unsigned done = *(const CST_AS unsigned*)COUNTER(mine0);
        PIN();
        if (wid < 4) {
#pragma unroll
            for (int i = 0; i < AGD; ++i) glds16(x0 + i * 16384, agoff, (wid * 8 + i) * 1024);
            if (lane < 4) glds4((TLBW & 1) ? theirs0 + H_TMO + 128 : x0, 0);
        }
        WAIT_LDS();
        BARRIER();
        asm volatile("" : "+s"(done));
        const unsigned call = done + 1;
        char* const theirs = theirs0 + (call & 1) * HALF_BYTES;
        STAMP(1);
        if (wid < 4) {
            int r = 0;
            WAIT_VM(1);
#pragma unroll 1
            for (int i = 0; i < NP; ++i) {
                WAIT_VM(2 * AGD - 2);
                if (i >= AGD && ((i - AGD) & 7) == 7 && lane == 0)
                    __hip_atomic_fetch_add(ctl + 8 + ((i - AGD) >> 3), 1u, __ATOMIC_RELAXED, __HIP_MEMORY_SCOPE_WORKGROUP);
                const u32x4 v = *(const u32x4*)(smem + (wid * 8 + r) * 1024 + lane * 16);
                store_sys(theirs + H_X + i * 16384, agoff, v);
                glds16(x0 + (i + AGD < NP ? i + AGD : 0) * 16384, agoff, (wid * 8 + r) * 1024);
                r = r == AGD - 1 ? 0 : r + 1;
            }
            WAIT_VM(0);
            if (lane == 0) __hip_atomic_fetch_add(ctl + 8 + AGU - 1, 1u, __ATOMIC_RELAXED, __HIP_MEMORY_SCOPE_WORKGROUP);
            STAMP(3);
        }
        advance(ctl, COUNTER(mine0), call, lane);
        FRESH();
        const float scale = SCALE();
        LDS_AS u64* mirror = (LDS_AS u64*)(ctl + (MIR_OFF - CTL_OFF) / 4);
        auto flag_word = [&](int i) {
            const u64 v = __hip_atomic_load(mirror + i * 2, __ATOMIC_ACQUIRE, __HIP_MEMORY_SCOPE_WORKGROUP);
            const unsigned fail = __hip_atomic_load(ctl + 14, __ATOMIC_ACQUIRE, __HIP_MEMORY_SCOPE_WORKGROUP);
            const bool ok = (unsigned)(v >> 32) >= call && ((unsigned)v >> 16) == (call & 0xffffu);
            return (ok ? ((unsigned)v & 0xffffu) | READY : 0u) | fail;
        };
        if (wid == 7) {
            for (int u = 0; u < AGU; ++u) {
                while (__hip_atomic_load(ctl + 8 + u, __ATOMIC_RELAXED, __HIP_MEMORY_SCOPE_WORKGROUP) < 4) __builtin_amdgcn_s_sleep(2);
                if (lane == 0) set_flag(theirs + H_XFLAG + (u * 4 + pan) * 128, call, 0);
            }
        } else if (wid == 6) {
            const char* const flags = mine0 + (call & 1) * HALF_BYTES + H_FLAG;
            const unsigned fo = (((PULL_ORDER >> (lane & 12)) & 15) * 4 + (lane & 3)) * 16;
            while (__hip_atomic_load(ctl + 12, __ATOMIC_RELAXED, __HIP_MEMORY_SCOPE_WORKGROUP) < 8) __builtin_amdgcn_s_sleep(4);
            const u64 t0 = NOW();
            while (true) {
                WAIT_VM(7);
                if (lane < 16) glds16_sys(flags, fo, MIR_OFF);
                __builtin_amdgcn_s_sleep(12);
                if (__all(flag_word(lane & 15) & READY)) break;
                if (give_up(t0, TMO(), call)) {
                    __hip_atomic_store(ctl + 14, READY | FAILED, __ATOMIC_RELEASE, __HIP_MEMORY_SCOPE_WORKGROUP);
                    break;
                }
            }
        }
        constexpr int N0 = 4 * nib(R_CNT, nib(PULL_ORDER, 0)), N1 = N0 + 3 * (NPU - N0 / 4), LAST = (NPU - 1) * 4;
        int tp = 3, ej, ep;
        for (int p = 2; p >= 0; --p) if (tight(w0 + kp->wown + p * 128)) tp = p;
        const int em = min(lane, NPU * 4 - 1);
        if (em < N0) { ej = em >> 2; ep = em & 3; }
        else if (em < N1) { ej = (em - N0) / 3; ep = em - N0 - 3 * ej; ep += ep >= tp; ej += N0 / 4; }
        else { ej = em - N1 + N0 / 4; ep = tp; }
        const int ee = (int)(pull_table() >> (8 * ej)) & 255;
        const unsigned eout = ((AGU + (ee & 15)) * 32 * NS + ep * 128 + pan * 32) * 4;
        char* const out = kp->out;
        LDS_AS unsigned* landed = ctl + 16;
        if (wid < 4) {
            const int en = ((ee & 15) * 4 + ep) * 4 + pan, ef = (ee >> 6) * 4 + ep, eb = ((ee >> 4) & 3) * 4 + pan;
            while (!(flag_word(12 + pan) & READY)) __builtin_amdgcn_s_sleep(1);
            unsigned words = 0, marks = 0;
#pragma unroll 1
            for (int j = 0; j < NPU; ++j) {
                const int m = j * 4 + wid;
                unsigned word = __builtin_amdgcn_readlane(words, m);
                for (int k = 0; !(word & READY); ++k) {
                    if (k == 1) WAIT_VM(2);
                    else if (k == 2) WAIT_VM(1);
                    else if (k == 3) {
                        WAIT_VM(0);
                        if (lane == 0) __hip_atomic_store(landed + wid, marks | j, __ATOMIC_RELEASE, __HIP_MEMORY_SCOPE_WORKGROUP);
                    } else if (k) __builtin_amdgcn_s_sleep(1);
                    const unsigned w = flag_word(ef);
                    words = (w & (READY | FAILED)) | ((w >> eb) & 1);
                    word = __builtin_amdgcn_readlane(words, m);
                }
                marks |= ((word & 1) << 8 | (word & FAILED ? 1u << 16 : 0u)) << j;
                if (word & FAILED) WAIT_VM(0);
                else fetch_subtile<2>(theirs, __builtin_amdgcn_readlane(en, m), m * 4096, word & 1, lane);
                if (lane == 0) __hip_atomic_store(landed + wid, marks | j, __ATOMIC_RELEASE, __HIP_MEMORY_SCOPE_WORKGROUP);
            }
            char* o = out + __builtin_amdgcn_readlane(eout, LAST + wid);
            float sc = (marks >> (15 + NPU)) & 1 ? __builtin_nanf("") : scale;
            asm volatile("" : "+s"(o));
            asm volatile("" : "+v"(sc));
            WAIT_VM(0);
            if (lane == 0) __hip_atomic_store(landed + wid, marks | NPU, __ATOMIC_RELEASE, __HIP_MEMORY_SCOPE_WORKGROUP);
            STAMP(2);
            finish_half<0>(smem, (LAST + wid) * 4096, (marks >> (7 + NPU)) & 1, sc, o, lane);
        } else {
            for (bool more = true; more;) {
                unsigned m = 0;
                if (lane == 0) m = __hip_atomic_fetch_add(ctl + 13, 1u, __ATOMIC_RELAXED, __HIP_MEMORY_SCOPE_WORKGROUP);
                m = __builtin_amdgcn_readfirstlane(m);
                more = m < LAST;
                if (!more) m = LAST + wid - 4;
                char* o = out + __builtin_amdgcn_readlane(eout, m);
                asm volatile("" : "+s"(o));
                unsigned w;
                while (((w = __builtin_amdgcn_readfirstlane(__hip_atomic_load(landed + (m & 3), __ATOMIC_ACQUIRE, __HIP_MEMORY_SCOPE_WORKGROUP))) & 255) <= m >> 2)
                    if (more) __builtin_amdgcn_s_sleep(1);
                w >>= m >> 2;
                const float sc = (w >> 16) & 1 ? __builtin_nanf("") : scale;
                finish_half<1>(smem, m * 4096, (w >> 8) & 1, sc, o, lane);
                if (more) finish_half<0>(smem, m * 4096, (w >> 8) & 1, sc, o, lane);
            }
        }
    } else {
        int cnt = NIB(local ? L_CNT : R_CNT);
        if (RARE(((local ? LPF_MASK : PF_MASK) >> cls) & 1)) {
            if ((wid >> 1) == 2) {
                const int me = (__builtin_popcount((local ? LPF_MASK : PF_MASK) & ((1 << cls) - 1)) * 4 + pan) * 2 + (wid & 1);
                if (RARE(local)) {
                    warm_l2<8, 0, __builtin_popcount(LPF_MASK) * 8>(w0, x0, me, lane);
                    warm_l2<8, 8, __builtin_popcount(LPF_MASK) * 8>(w0, x0, me, lane);
                } else {
                    warm_l2<8 + PF_XU / 2, 8, __builtin_popcount(PF_MASK) * 8>(w0, x0 + AGU * 32 * KD, me, lane);
                    if ((unsigned)(me - PF2_LO * 2) < PF2_N * 2)
                        warm_l2<PF2_XU / 2, 0, PF2_N * 2>(w0, x0 + (AGU + PF2_FIRST) * 32 * KD, me - PF2_LO * 2, lane);
                }
            } else if (wid == 6) {
                const int k = b >> 3;
                if ((TLBW & 4) && lane == 0) glds4(k == 4 ? theirs0 + H_TMO + 128 : k < 8 ? mine0 + H_TMO + 128 : w0 + (size_t)(k - 8) * (2u << 20), 0);
                asm volatile("" :: "s"(CALL()), "v"(SCALE()));
            }
            WAIT_LDS();
            BARRIER();
        } else if (RARE(local)) {
            while (NOW() - tstart < L_START) __builtin_amdgcn_s_sleep(2);
        }
        int task = cnt ? 0 : 1;
        unsigned lastcall = 0;
        const int ntask = !local && NIB(A_CNT) ? 2 : 1;
#pragma unroll 1
        for (; task < ntask; ++task) {
            unsigned bad = 0, first = local ? NIB(L_FIRST) : AGU + NIB(R_FIRST);
            const char *wp1 = nullptr, *xp1 = nullptr;
            if (RARE(task == 1)) {
                FRESH();
                const unsigned call = CALL();
                const char* const mine = kp->mine + (call & 1) * HALF_BYTES;
                wp1 = kp->wblk + kp->wown;
                xp1 = mine + H_X;
                const bool tp = tight(wp1 + pan * 128);
                cnt = NIB(tp ? T_CNT : A_CNT);
                first = NIB(tp ? T_FIRST : A_FIRST);
                if (wid == 1) {
                    if (!tp) while (NOW() - tstart < A_START) __builtin_amdgcn_s_sleep(4);
                    if (lane < 4) ctl[1] = poll_flags(mine + H_XFLAG + ((first + cnt - 1) * 4 + lane) * 128, TMO(), call);
                    WAIT_LDS();
                }
                BARRIER();
                bad = __builtin_amdgcn_readfirstlane(ctl[1]);
                STAMP(4);
            }
            FRESH();
            unsigned woff = w_offset(lane, wid) + pan * 128, xoff = x_offset(lane, wid) + first * 32 * KD;
            asm volatile("" : "+v"(woff), "+v"(xoff));
            PIN();
            WAIT_LDS();
            const char *wp = task ? wp1 : w0, *xp = task ? xp1 : x0;
            if (wid < 4 && !bad) {
#pragma unroll
                for (int j = 0; j < 4; ++j) glds16(wp + j * 8 * ND, j >= 2 ? woff ^ 16 : woff, wid * 4096 + j * 1024);
                glds16_sys(xp, xoff, 16384 + wid * 1024);
                if (cnt > 1) glds16_sys(xp + 32 * KD, xoff, 16384 + 4096 + wid * 1024);
                if (cnt > 2) glds16_sys(xp + 64 * KD, xoff, 16384 + 8192 + wid * 1024);
            }
            const u64 t0 = NOW();
            const bool direct = local || task == 1, push = (R_PUSH >> cls) & 1;
            char *mine = kp->mine, *flags = kp->theirs, *out = kp->out;
            const char *sxp = kp->sx, *swp = kp->sw;
            asm volatile("" : "+s"(mine), "+s"(flags), "+s"(out), "+s"(sxp), "+s"(swp));
            const unsigned done = *(const CST_AS unsigned*)COUNTER(mine);
            const float sx = *(const CST_AS float*)sxp, sw = *(const CST_AS float*)swp;
            const unsigned n0 = ((first - AGU + ((wid ^ 4) >> 2)) * 4 + pan) * 4 + (wid & 3);
            char* far = (push ? flags : mine) + H_LONG + n0 * 2048;
            char* dst = direct ? out + ((first * 32 + (wid & 1) * 16) * NS + pan * 128 + (wid >> 1) * 32) * 4 : (push ? flags : mine) + n0 * SHORT_BYTES;
            flags += H_FLAG;
            unsigned ta = (lane >> 3) * 128 + (((lane & 7) ^ (lane >> 3)) << 4) + (direct ? wid * 2048 : (wid ^ 4) * 4096);
            int mode = direct ? 0 : (wid ^ 4) >= cnt * 4 ? 1 : (wid ^ 4) + 8 >= cnt * 4 ? 2 : 3;
            asm volatile("" : "+s"(far), "+s"(dst), "+s"(flags), "+s"(mode));
            asm volatile("" : "+v"(ta));
            if (bad) {
            } else if (wid < 4) {
                if (cnt == 1) load_tile<1, 3>(wp, xp, woff, xoff, wid);
                else if (cnt == 2) load_tile<2, 3>(wp, xp, woff, xoff, wid);
                else load_tile<3, 2>(wp, xp, woff, xoff, wid);
                if (!direct && push) while (NOW() - t0 < HOLD0 + HOLD1 * (first - AGU)) __builtin_amdgcn_s_sleep(1);
            } else if (cnt == 1) tile_compute<1>(smem, lane, wid - 4, done, sx, sw);
            else if (cnt == 2) tile_compute<2>(smem, lane, wid - 4, done, sx, sw);
            else tile_compute<3>(smem, lane, wid - 4, done, sx, sw);
            const unsigned call = done + 1, half = (call & 1) * HALF_BYTES;
            lastcall = call;
            if (mode) { dst += half; far += half; flags += half; }
            asm volatile("" : "+s"(far), "+s"(dst), "+s"(flags));
            BARRIER();
            if (mode == 0) {
                const float scale = bad ? __builtin_nanf("") : sx * sw;
#pragma unroll 1
                for (int u = 0; u < cnt; ++u) {
                    const u32x4 t0 = *(const u32x4*)(smem + ta + u * 16384), t1 = *(const u32x4*)(smem + ta + u * 16384 + 1024);
                    write_rows(t0, scale, dst + u * (32 * NS * 4), lane);
                    write_rows(t1, scale, dst + u * (32 * NS * 4) + 8 * NS * 4, lane);
                }
            } else {
                unsigned marks = 0;
                if (mode > 1)
                    marks = send_subtile<0>(smem + ta, dst, far, lane, push && cnt > 1,
                                            cls == GO_CLS && (wid ^ 4) == GO_ITEM ? flags + pan * 16 : nullptr, call);
                if (mode > 2) marks |= send_subtile<1>(smem + ta + 32768, dst + 32 * SHORT_BYTES, far + 32 * 2048, lane, push, nullptr, call) << 8;
                if (marks && lane == 0) __hip_atomic_fetch_or(ctl, marks << (wid ^ 4), __ATOMIC_RELAXED, __HIP_MEMORY_SCOPE_WORKGROUP);
                flags += (b >> 3) * 16;
                asm volatile("" : "+s"(flags));
                asm volatile("s_waitcnt vmcnt(0) lgkmcnt(0)" ::: "memory");
                BARRIER();
                STAMP(3);
                if (wid == 7 && lane == 0) set_flag(flags, call, ctl[0]);
            }
            if (task + 1 == ntask) advance(ctl, COUNTER(mine), call, lane);
        }
        FRESH();
        if (!lastcall) advance(ctl, COUNTER(kp->mine), CALL(), lane);
        if (!local) {
            const char* const mine = kp->mine + (lastcall & 1) * HALF_BYTES;
            char* const out = kp->out + pan * 128 * 4;
            const float scale = SCALE();
#pragma unroll 1
            for (int c = 1; c < 8; ++c) {
                if (((M_CLS >> (4 * c)) & 15) != cls) continue;
                BARRIER();
                if (wid == 1) {
                    if (lane == 0) ctl[1] = poll_flags(mine + H_FLAG + (c * 4 + pan) * 16, TMO(), lastcall);
                    WAIT_LDS();
                }
                BARRIER();
                const unsigned word = __builtin_amdgcn_readfirstlane(ctl[1]);
                const float sc = word & FAILED ? __builtin_nanf("") : scale;
                for (int i = wid; i < ((R_CNT >> (4 * c)) & 15) * 4; i += 8) {
                    const int u = ((R_FIRST >> (4 * c)) & 15) + (i >> 2);
                    if (!(word & FAILED)) fetch_subtile<63>(mine, (u * 4 + pan) * 4 + (i & 3), i * 4096, (word >> i) & 1, lane);
                    WAIT_VM(0);
                    char* const o = out + ((AGU + u) * 32 * NS + (i & 3) * 32) * 4;
                    finish_half<0>(smem, i * 4096, (word >> i) & 1, sc, o, lane);
                    finish_half<1>(smem, i * 4096, (word >> i) & 1, sc, o, lane);
                }
            }
        }
    }
}
#undef cls
#undef pan
#undef local

extern "C" void kernel_launch(int rank, int nranks, ncclComm_t comm,
                              hipStream_t stream,
                              void* const* d_in, const int* in_sizes, int n_in,
                              void* d_out, int out_size,
                              const int* in_dtypes, int out_dtype,
                              void* const* d_out_all,
                              void* const* d_ws_all, size_t ws_bytes) {
    if (nranks != NR || (unsigned)rank >= (unsigned)NR || ws_bytes < WS_END || n_in != 4 || in_sizes[0] != MS * KD
        || in_sizes[1] != KD * ND || out_size != NR * MS * NS) return;
    static bool init[NR] = {};
    if (!init[rank]) {
        HIP_CHECK(hipFuncSetAttribute((const void*)fused_kernel, hipFuncAttributeMaxDynamicSharedMemorySize, LDS_BYTES));
        init[rank] = true;
    }
    static_assert(LOCAL_GRP == 3, "three peers in the own quad");
    int peer[NR], lo = 0, hi = 4;
    const unsigned quad = ((0x4d >> rank) & 1) ? 0x4du : 0xb2u;
    for (int i = 1; i < NR; ++i) {
        const int q = (rank + i) & 7;
        if ((quad >> q) & 1) peer[lo++] = q; else peer[hi++] = q;
    }
    peer[LOCAL_GRP] = rank;
    Params p;
    for (int g = 0; g < NR; ++g) {
        p.g[g].x = (const char*)d_in[0];
        p.g[g].wblk = (const char*)d_in[1] + peer[g] * NS;
        p.g[g].mine = (char*)d_ws_all[rank] + (size_t)peer[g] * BLK_BYTES;
        p.g[g].theirs = (char*)d_ws_all[peer[g]] + (size_t)rank * BLK_BYTES;
        p.g[g].out = (char*)d_out + (size_t)peer[g] * MS * NS * 4;
        p.g[g].sx = (const char*)d_in[2];
        p.g[g].sw = (const char*)d_in[3];
        p.g[g].wown = (rank - peer[g]) * NS;
        p.g[g].peer = peer[g];
    }
    hipLaunchKernelGGL(fused_kernel, dim3(GRID), dim3(BLOCK), LDS_BYTES, stream, p);
    HIP_CHECK(hipGetLastError());
}
```

```cpp
#include <hip/hip_runtime.h>
#include <rccl/rccl.h>
#include <cstdio>

#define HIP_CHECK(expr) do { hipError_t _e = (expr); if (_e != hipSuccess) { \
    fprintf(stderr, "HIP error %s at %s:%d\n", hipGetErrorString(_e), __FILE__, __LINE__); \
    return; } } while (0)

typedef int v2i __attribute__((ext_vector_type(2)));
typedef int v4i __attribute__((ext_vector_type(4)));
typedef unsigned u32x4 __attribute__((ext_vector_type(4)));
typedef float f32x4 __attribute__((ext_vector_type(4)));
typedef unsigned long long u64;
#define LDS_AS __attribute__((address_space(3)))
#define GLB_AS __attribute__((address_space(1)))
#define CST_AS __attribute__((address_space(4)))

constexpr int AGU = 3;
constexpr unsigned R_CNT = 0x33210220;
constexpr unsigned R_FIRST = 0x63100b90;
constexpr unsigned R_PUSH = 0x70;
constexpr unsigned PULL_ORDER = 0x127;
constexpr unsigned M_CLS = 0x06570000;
constexpr unsigned A_CNT = 0x00012000;
constexpr unsigned A_FIRST = 0x00020000;
constexpr unsigned T_CNT = 0x00021000;
constexpr unsigned T_FIRST = 0x00010000;
constexpr unsigned PF_MASK = 0x0e;
constexpr int PF_XU = 9;
constexpr int PF2_FIRST = 9, PF2_XU = 4, PF2_LO = 8, PF2_N = 3;
constexpr int GO_CLS = 6, GO_ITEM = 7;
constexpr int LOCAL_GRP = 3;
constexpr unsigned L_CNT = 0x33330022;
constexpr unsigned L_FIRST = 0x963000ec;
constexpr unsigned LPF_MASK = 0x0f;
#define L_START 256
#define A_START 930
#define HOLD0 700
#define HOLD1 78
#define AGD 4
#define CODEW 1
constexpr int CODE_BYTES = 18432, CODE_PIECES = 3;
#define TLBW 5

constexpr int NR = 8, MS = 512, KD = 4096, ND = 4096, NS = 512;
constexpr int GRID = 256, BLOCK = 512;
constexpr int NSLOT = 4;
constexpr int SLOT_BYTES = 32768;
constexpr int CTL_OFF = NSLOT * SLOT_BYTES;
constexpr int MIR_OFF = CTL_OFF + 256;
constexpr int JUNK_OFF = MIR_OFF + 256;
constexpr int LDS_BYTES = JUNK_OFF + 256;
constexpr int BIAS = 1 << 21;
constexpr u64 DEADLINE_TICKS = 40000000ull;

constexpr unsigned BLK_BYTES = 2u << 20, HALF_BYTES = 1u << 20;
constexpr int SHORT_BYTES = 2816;
constexpr unsigned H_X = 144 * 4096;
constexpr unsigned H_FLAG = H_X + AGU * 32 * KD;
constexpr unsigned H_XFLAG = H_FLAG + 1024;
constexpr unsigned H_CNT = H_FLAG + 4096;
constexpr unsigned H_TMO = H_CNT + 4096;
constexpr unsigned H_LONG = NR * BLK_BYTES;
constexpr size_t WS_END = 2 * (size_t)NR * BLK_BYTES;
static_assert((16 - AGU) * 16 * SHORT_BYTES <= H_X && H_XFLAG + AGU * 4 * 128 <= H_CNT && H_TMO + 256 <= HALF_BYTES, "layout");
static_assert(PF_MASK == 0x0e && LPF_MASK == 0x0f && PF_XU <= 12 && AGU + PF_XU <= 16 && PF2_XU % 2 == 0 && AGU + PF2_FIRST + PF2_XU <= 16, "warm-up");
constexpr int nib(unsigned w, int c) { return (w >> 4 * c) & 15; }
constexpr bool plan_ok() {
    unsigned r = 0, a = 0, t = 0, l = 0;
    for (int c = 0; c < 8; ++c) {
        const int rc = nib(R_CNT, c), ac = nib(A_CNT, c), tc = nib(T_CNT, c), lc = nib(L_CNT, c);
        const bool push = (R_PUSH >> c) & 1;
        bool pulled = false;
        for (int i = 0; i < 3; ++i) pulled |= nib(PULL_ORDER, i) == c;
        if (rc > 3 || ac > 3 || tc > 3 || lc > 3 || !tc != !ac) return false;
        if (rc ? pulled == push : pulled || push) return false;
        if ((nib(M_CLS, c) != 0) != push || nib(M_CLS, c) > 7) return false;
        if ((c && !rc && !ac) || (!lc && !((LPF_MASK >> c) & 1))) return false;
        r += ((1u << rc) - 1) << nib(R_FIRST, c);
        a += ((1u << ac) - 1) << nib(A_FIRST, c);
        t += ((1u << tc) - 1) << nib(T_FIRST, c);
        l += ((1u << lc) - 1) << nib(L_FIRST, c);
    }
    return r == (1u << (16 - AGU)) - 1 && a == (1u << AGU) - 1 && t == a && l == 0xffffu && !nib(A_CNT, 0) && !(PF_MASK & 1)
           && ((R_PUSH >> GO_CLS) & 1) && nib(R_CNT, GO_CLS) > 1 && GO_ITEM < 8;
}
static_assert(plan_ok(), "the tiles must cover every unit once");
constexpr int NPU = nib(R_CNT, nib(PULL_ORDER, 0)) + nib(R_CNT, nib(PULL_ORDER, 1)) + nib(R_CNT, nib(PULL_ORDER, 2));
constexpr u64 pull_table() {
    u64 v = 0; int n = 0;
    for (int t = 0; t < 3; ++t)
        for (int u = 0; u < nib(R_CNT, nib(PULL_ORDER, t)); ++u) v |= (u64)(t << 6 | u << 4 | (nib(R_FIRST, nib(PULL_ORDER, t)) + u)) << 8 * n++;
    return v;
}
static_assert(NPU <= 8 && NPU * 16384 <= CTL_OFF, "pull buffers");

#define STAMP(i) do { } while (0)

struct Group {
    const char* x;
    const char* wblk;
    char* mine;
    char* theirs;
    char* out;
    const char* sx;
    const char* sw;
    int wown;
    int peer;
};
struct Params { Group g[NR]; };
static_assert(sizeof(Group) == 64, "one line");

#define WAIT_LDS() __builtin_amdgcn_s_waitcnt(0xC07F)
#define WAIT_VM(n) asm volatile("s_waitcnt vmcnt(%0)" :: "n"(n) : "memory")
#define PIN() __builtin_amdgcn_sched_barrier(0)
#define BARRIER() do { asm volatile("" ::: "memory"); __builtin_amdgcn_s_barrier(); asm volatile("" ::: "memory"); } while (0)
#define NOW() __builtin_amdgcn_s_memrealtime()
#define RARE(c) __builtin_expect(!!(c), 0)

__device__ __forceinline__ void glds16(const char* sbase, unsigned voff, unsigned lds) {
    asm volatile("s_mov_b32 m0, %2\n\ts_nop 0\n\tglobal_load_lds_dwordx4 %0, %1" :: "v"(voff), "s"(sbase), "s"(lds) : "memory");
}
__device__ __forceinline__ void glds16_sys(const char* sbase, unsigned voff, unsigned lds) {
    asm volatile("s_mov_b32 m0, %2\n\ts_nop 0\n\tglobal_load_lds_dwordx4 %0, %1 sc0 sc1" :: "v"(voff), "s"(sbase), "s"(lds) : "memory");
}
__device__ __forceinline__ void glds4(const char* sbase, unsigned voff) {
    asm volatile("s_mov_b32 m0, %2\n\ts_nop 0\n\tglobal_load_lds_dword %0, %1" :: "v"(voff), "s"(sbase), "s"(JUNK_OFF) : "memory");
}
__device__ __forceinline__ void store_sys(char* sbase, unsigned voff, u32x4 v) {
    asm volatile("global_store_dwordx4 %0, %1, %2 sc0 sc1\n\ts_nop 1" :: "v"(voff), "v"(v), "s"(sbase) : "memory");
}
template <int STEP>
__device__ __forceinline__ void store_sys(char* sbase, unsigned voff, u32x4 v, int i) {
    if (i == 0) store_sys(sbase, voff, v);
    else if (i == 1) asm volatile("global_store_dwordx4 %0, %1, %2 offset:%3 sc0 sc1\n\ts_nop 1" :: "v"(voff), "v"(v), "s"(sbase), "n"(STEP) : "memory");
    else asm volatile("global_store_dwordx4 %0, %1, %2 offset:%3 sc0 sc1\n\ts_nop 1" :: "v"(voff), "v"(v), "s"(sbase), "n"(2 * STEP) : "memory");
}
__device__ __forceinline__ void set_flag(char* flag, unsigned call, unsigned bits) {
    __hip_atomic_store((GLB_AS u64*)flag, ((u64)call << 32) | (call << 16) | bits, __ATOMIC_RELAXED, __HIP_MEMORY_SCOPE_SYSTEM);
}

__device__ __forceinline__ unsigned w_offset(int lane, int wid) {
    return (unsigned)(wid * 32 + (lane >> 3)) * ND + (((lane & 7) ^ (((lane >> 4) & 3) << 1)) << 4);
}
__device__ __forceinline__ unsigned x_offset(int lane, int wid) {
    return (unsigned)(wid * 8 + (lane >> 3)) * KD + (((lane & 7) ^ ((wid * 4 + (lane >> 4)) & 7)) << 4);
}
template <int U, int LA>
__device__ __forceinline__ void load_tile(const char* wp, const char* xp, unsigned woff, unsigned xoff, int wid) {
    static_assert(LA < NSLOT, "ring too small");
    unsigned slot = SLOT_BYTES;
#pragma unroll 1
    for (int s = 1; s < 32; ++s) {
        if (s >= LA) {
            WAIT_VM((LA - 1) * (4 + U));
            BARRIER();
        }
#pragma unroll
        for (int j = 0; j < 4; ++j) glds16(wp + (size_t)(s * 128 + j * 8) * ND, j >= 2 ? woff ^ 16 : woff, slot + wid * 4096 + j * 1024);
#pragma unroll
        for (int u = 0; u < U; ++u) glds16_sys(xp + u * 32 * KD + s * 128, xoff, slot + 16384 + u * 4096 + wid * 1024);
        slot = (slot + SLOT_BYTES) & (NSLOT * SLOT_BYTES - 1);
    }
#pragma unroll
    for (int r = LA - 1; r >= 0; --r) {
        WAIT_VM(r * (4 + U));
        BARRIER();
    }
}

template <int U> struct Frag { v2i w[4][2]; v4i x[U][4]; };
struct Addr { unsigned w0, w1, x0, x1; };

template <int U>
__device__ __forceinline__ void read_frag(Frag<U>& f, char* smem, const Addr& a, int r) {
    const int kk = r / (2 + U), i = r % (2 + U);
    if (i < 2) f.w[kk][i] = __builtin_amdgcn_ds_read_tr8_b64_v2i32((LDS_AS v2i*)(smem + (kk & 1 ? a.w1 : a.w0) + (kk >> 1) * 8192 + i * 1024));
    else f.x[i - 2][kk] = *(const v4i*)(smem + (kk >> 1 ? a.x1 : a.x0) + (kk & 1) * 2048 + (i - 2) * 4096);
}
template <int U>
__device__ __forceinline__ void mfma(const Frag<U>& f, v4i (&acc)[U][4], int m) {
    const int k2 = m / (4 * U), u = m / 4 % U, i = m % 4;
    const v2i lo = f.w[k2 * 2 + (i >> 1)][0], hi = f.w[k2 * 2 + (i >> 1)][1];
    asm volatile("v_mfma_i32_16x16x64_i8 %0, %1, %2, %0" : "+v"(acc[u][i]) : "v"(v4i{lo.x, lo.y, hi.x, hi.y}), "v"(f.x[u][k2 * 2 + (i & 1)]));
}
template <int U>
__device__ __forceinline__ void stage(const Frag<U>& cur, Frag<U>& nxt, v4i (&acc)[U][4], char* smem, const Addr& a) {
    constexpr int NRD = 4 * (2 + U), PER = U == 1 ? 2 : 1;
#pragma unroll
    for (int m = 0; m < 8 * U; ++m) {
        mfma<U>(cur, acc, m);
#pragma unroll
        for (int r = m * PER; r < (m + 1) * PER && r < NRD; ++r) read_frag<U>(nxt, smem, a, r);
        PIN();
    }
}
template <int U>
__device__ __forceinline__ void tile_compute(char* smem, int lane, int cw, unsigned keep0, float keep1, float keep2) {
    const int g = lane >> 4, q = (lane & 15) >> 1;
    const unsigned wad = (16 * g + q) * 128 + (((2 * cw) ^ (((q >> 1) & 3) << 1) ^ (g & 1)) << 4) + (lane & 1) * 8;
    const unsigned xad = (lane & 15) * 128 + ((g ^ ((lane >> 1) & 7)) << 4) + 16384;
    Addr a = {wad, wad ^ 16, xad, xad ^ 64};
    auto next = [&]() {
        constexpr unsigned M = NSLOT * SLOT_BYTES - 1;
        a.w0 = (a.w0 + SLOT_BYTES) & M; a.w1 = a.w0 ^ 16; a.x0 = (a.x0 + SLOT_BYTES) & M; a.x1 = a.x0 ^ 64;
        PIN();
    };
    v4i acc[U][4];
#pragma unroll
    for (int u = 0; u < U; ++u)
#pragma unroll
        for (int i = 0; i < 4; ++i) acc[u][i] = v4i{BIAS, BIAS, BIAS, BIAS};
    Frag<U> fa, fb;
    BARRIER();
#pragma unroll
    for (int r = 0; r < 4 * (2 + U); ++r) read_frag<U>(fa, smem, a, r);
    asm volatile("" :: "s"(keep0), "s"(keep1), "s"(keep2));
#pragma unroll 1
    for (int s = 1; s < 31; s += 2) {
        next();
        WAIT_LDS();
        BARRIER();
        stage<U>(fa, fb, acc, smem, a);
        next();
        WAIT_LDS();
        BARRIER();
        stage<U>(fb, fa, acc, smem, a);
    }
    next();
    WAIT_LDS();
    BARRIER();
    stage<U>(fa, fb, acc, smem, a);
    char* const d0 = smem + cw * 4096 + (lane & 15) * 128 + ((g ^ (lane & 7)) << 4);
    char* const d1 = smem + cw * 4096 + (lane & 15) * 128 + (((4 + g) ^ (lane & 7)) << 4);
#pragma unroll
    for (int u = 0; u <= U; ++u) {
        if (u == U) asm volatile("s_nop 7" ::: "memory");
#pragma unroll
        for (int i = 0; i < 8; ++i) {
            if (u < U) mfma<U>(fb, acc, (i >> 2) * 4 * U + u * 4 + (i & 3));
            if (u > 0 && i >= 4) *(v4i*)((i & 2 ? d1 : d0) + (u - 1) * 16384 + (i & 1) * 2048) = acc[u - 1][i & 3];
            PIN();
        }
    }
    WAIT_LDS();
}
__device__ __forceinline__ bool tight(const char* p) { return (((unsigned)(size_t)p >> 7) & 7) == 3 || (((unsigned)(size_t)p >> 7) & 31) == 31; }

__device__ __forceinline__ void write_rows(const u32x4& t, float scale, char* obase, int lane) {
    f32x4 v;
#pragma unroll
    for (int e = 0; e < 4; ++e) v[e] = (float)(int)(t[e] - BIAS) * scale;
    store_sys(obase, (lane >> 3) * (NS * 4) + (lane & 7) * 16, __builtin_bit_cast(u32x4, v));
}

template <int R>
__device__ __forceinline__ unsigned send_subtile(const char* src, char* dst, char* far, int lane, bool paced, char* go, unsigned call) {
    const unsigned off = lane * 16;
    u32x4 t[4];
#pragma unroll
    for (int i = 0; i < 4; ++i) t[i] = *(const u32x4*)(src + i * 1024);
#pragma unroll
    for (int i = 0; i < 2; ++i) {
        const u32x4 lo = {__builtin_amdgcn_perm(t[2 * i][1], t[2 * i][0], 0x05040100u), __builtin_amdgcn_perm(t[2 * i][3], t[2 * i][2], 0x05040100u),
                          __builtin_amdgcn_perm(t[2 * i + 1][1], t[2 * i + 1][0], 0x05040100u), __builtin_amdgcn_perm(t[2 * i + 1][3], t[2 * i + 1][2], 0x05040100u)};
        if (R == 1 && paced) WAIT_VM(3);
        store_sys<1024>(dst, off, lo, i);
        PIN();
    }
    u32x4 hi[2];
#pragma unroll
    for (int i = 0; i < 2; ++i)
        hi[i] = u32x4{__builtin_amdgcn_perm(t[2 * i][1], t[2 * i][0], 0x07060302u), __builtin_amdgcn_perm(t[2 * i][3], t[2 * i][2], 0x07060302u),
                      __builtin_amdgcn_perm(t[2 * i + 1][1], t[2 * i + 1][0], 0x07060302u), __builtin_amdgcn_perm(t[2 * i + 1][3], t[2 * i + 1][2], 0x07060302u)};
    const u32x4 o = hi[0] | hi[1];
    const bool wide = __any(((o[0] | o[1] | o[2] | o[3]) & 0xffc0ffc0u) != 0);
    unsigned b[4];
#pragma unroll
    for (int i = 0; i < 4; ++i) b[i] = __builtin_amdgcn_perm(hi[i >> 1][(i & 1) * 2 + 1], hi[i >> 1][(i & 1) * 2], 0x06040200u);
    const unsigned h0 = b[0] | ((b[3] << 6) & 0xc0c0c0c0u), h1 = b[1] | ((b[3] << 4) & 0xc0c0c0c0u), h2 = b[2] | ((b[3] << 2) & 0xc0c0c0c0u);
    auto s1 = __builtin_amdgcn_permlane16_swap(h0, h1, false, false);
    auto s2 = __builtin_amdgcn_permlane16_swap(h2, h2, false, false);
    auto s3 = __builtin_amdgcn_permlane32_swap(s1[1], s2[1], false, false);
    const u32x4 h = {h0, h1, h2, s3[1]};
    if (paced) {
        if (R == 0) WAIT_VM(1); else WAIT_VM(3);
        if (R == 0 && go && lane == 0) set_flag(go, call, 0);
    }
    if (wide) {
        store_sys<1024>(far, off, hi[0], 0);
        store_sys<1024>(far, off, hi[1], 1);
    } else if (lane < 48) {
        store_sys<1024>(dst, off, h, 2);
    }
    return wide;
}
template <int D>
__device__ __forceinline__ void fetch_subtile(const char* base, int n, unsigned buf, bool raw, int lane) {
    const char* const src = base + n * SHORT_BYTES;
    WAIT_VM(D);
    glds16_sys(src, lane * 16, buf);
    WAIT_VM(D);
    glds16_sys(src + 1024, lane * 16, buf + 1024);
    WAIT_VM(D);
    glds16_sys(raw ? base + H_LONG + n * 2048 : src + 2048, min(lane, raw ? 63 : 47) * 16, buf + 2048);
    if (raw) glds16_sys(base + H_LONG + n * 2048 + 1024, lane * 16, buf + 3072);
}
template <int H>
__device__ __forceinline__ void finish_half(char* smem, unsigned buf, bool raw, float scale, char* obase, int lane) {
    const u32x4 lo = *(const u32x4*)(smem + buf + H * 1024 + lane * 16);
    const unsigned off = (lane >> 3) * (NS * 4) + (lane & 7) * 16;
    f32x4 v[2];
    if (raw) {
        const u32x4 hi = *(const u32x4*)(smem + buf + 2048 + H * 1024 + lane * 16);
#pragma unroll
        for (int e = 0; e < 4; ++e) {
            v[e >> 1][(e & 1) * 2] = (float)(int)(__builtin_amdgcn_perm(hi[e], lo[e], 0x05040100u) - BIAS) * scale;
            v[e >> 1][(e & 1) * 2 + 1] = (float)(int)(__builtin_amdgcn_perm(hi[e], lo[e], 0x07060302u) - BIAS) * scale;
        }
    } else {
        const unsigned ha = buf + 2048 + (lane < 48 ? lane * 16 : (lane - 48) * 16 + 12), hs = lane < 48 ? 4 : 256;
        const unsigned h0 = *(const unsigned*)(smem + ha), h1 = *(const unsigned*)(smem + ha + hs);
        unsigned b[2];
        if (H == 0) {
            b[0] = h0 & 0x3f3f3f3fu; b[1] = h1 & 0x3f3f3f3fu;
        } else {
            const unsigned h2 = *(const unsigned*)(smem + ha + 2 * hs);
            b[0] = h2 & 0x3f3f3f3fu;
            b[1] = ((h0 >> 6) & 0x03030303u) | ((h1 >> 4) & 0x0c0c0c0cu) | ((h2 >> 2) & 0x30303030u);
        }
#pragma unroll
        for (int i = 0; i < 2; ++i) {
            v[i][0] = ((float)__builtin_amdgcn_perm(b[i], lo[2 * i], 0x0c040100u) - (float)BIAS) * scale;
            v[i][1] = ((float)__builtin_amdgcn_perm(b[i], lo[2 * i], 0x0c050302u) - (float)BIAS) * scale;
            v[i][2] = ((float)__builtin_amdgcn_perm(b[i], lo[2 * i + 1], 0x0c060100u) - (float)BIAS) * scale;
            v[i][3] = ((float)__builtin_amdgcn_perm(b[i], lo[2 * i + 1], 0x0c070302u) - (float)BIAS) * scale;
        }
    }
    store_sys(obase + (2 * H) * 8 * (NS * 4), off, __builtin_bit_cast(u32x4, v[0]));
    store_sys(obase + (2 * H + 1) * 8 * (NS * 4), off, __builtin_bit_cast(u32x4, v[1]));
}

constexpr unsigned READY = 1u << 31, FAILED = 1u << 30;
__device__ __forceinline__ bool give_up(u64 t0, char* tmo, unsigned call) {
    const u64 dt = NOW() - t0;
    if (dt > 10000 && (dt > DEADLINE_TICKS || __hip_atomic_load((GLB_AS unsigned*)tmo, __ATOMIC_RELAXED, __HIP_MEMORY_SCOPE_AGENT) == call)) {
        __hip_atomic_store((GLB_AS unsigned*)tmo, call, __ATOMIC_RELAXED, __HIP_MEMORY_SCOPE_AGENT);
        return true;
    }
    return false;
}
__device__ __forceinline__ unsigned poll_flags(const char* flag, char* tmo, unsigned call) {
    const u64 t0 = NOW();
    while (true) {
        const u64 v = __hip_atomic_load((GLB_AS u64*)flag, __ATOMIC_RELAXED, __HIP_MEMORY_SCOPE_SYSTEM);
        if (__all((unsigned)(v >> 32) >= call)) return (unsigned)v & 0xffffu;
        if (give_up(t0, tmo, call)) return FAILED;
        __builtin_amdgcn_s_sleep(8);
    }
}

__device__ __forceinline__ void warm_l2(const char* p, unsigned vo, int shift, int n0, int dn, int n1) {
    p += (unsigned)n0 << shift;
#pragma unroll 1
    for (int n = n0; n < n1; n += dn, p += (unsigned)dn << shift) glds4(p, vo);
}
__device__ __forceinline__ void advance(LDS_AS unsigned* ctl, char* counter, unsigned call, int lane) {
    asm volatile("" :: "s"(call) : "memory");
    if (lane == 0 && __hip_atomic_fetch_add(ctl + 12, 1u, __ATOMIC_RELAXED, __HIP_MEMORY_SCOPE_WORKGROUP) == 7)
        __hip_atomic_store((GLB_AS unsigned*)counter, call, __ATOMIC_RELAXED, __HIP_MEMORY_SCOPE_AGENT);
}

typedef const CST_AS Group* KArgs;
#define FRESH() KArgs kp = kp0; int b = b0, wid = wid0, lane = lane0; \
    asm volatile("" : "+s"(kp), "+s"(b), "+s"(wid) :: "memory"); asm volatile("" : "+v"(lane))
#define COUNTER(mine) ((mine) + H_CNT + (b >> 3) * 128)
#define CALL() (*(const CST_AS unsigned*)COUNTER(kp->mine) + 1u)
#define SCALE() (*(const CST_AS float*)kp->sx * *(const CST_AS float*)kp->sw)
#define TMO() (kp->mine - (size_t)kp->peer * BLK_BYTES + H_TMO)
#define cls (b >> 5)
#define pan ((b >> 3) & 3)
#define local ((b & 7) == LOCAL_GRP)
#define NIB(word) (((unsigned)(word) >> (4 * cls)) & 15)

__global__ void __launch_bounds__(BLOCK, 2) fused_kernel(Params) {
    extern __shared__ __attribute__((aligned(16))) char smem[];
    const KArgs kp = (KArgs)__builtin_amdgcn_kernarg_segment_ptr() + (blockIdx.x & 7), kp0 = kp;
    const int lane = threadIdx.x & 63, wid = __builtin_amdgcn_readfirstlane(threadIdx.x >> 6), b = blockIdx.x;
    const int lane0 = lane, wid0 = wid, b0 = b;
    const char *const x0 = kp->x, *const w0 = kp->wblk;
    char *const mine0 = kp->mine, *const theirs0 = kp->theirs;
    const u64 tstart = NOW();
    if (CODEW) {
        u64 pc;
        asm volatile("s_cmp_lt_u32 %2, %3\n\ts_cbranch_scc0 .Lnocode%=\n\ts_getpc_b64 %0\n\ts_andn2_b64 %0, %0, 0xff\n\ts_mov_b32 m0, %4\n\ts_nop 0\n\t"
                     "global_load_lds_dword %1, %0\n.Lnocode%=:"
                     : "=&s"(pc) : "v"(min((b >> 3) * 8192 + lane * 128, CODE_BYTES - 4)), "s"(b + (7 - wid) * 256), "n"(8 * CODE_PIECES), "n"(JUNK_OFF)
                     : "scc", "memory");
    }
    PIN();
    ((LDS_AS u64*)(smem + CTL_OFF))[lane] = 0;
    LDS_AS unsigned* ctl = (LDS_AS unsigned*)(smem + CTL_OFF);
    const bool link = cls == 0 && !local;
    const unsigned agoff = (pan * 4 + (wid & 3)) * 1024 + lane * 16;
    unsigned done0 = 0;
    if (b < 128) {
        if (link) {
            done0 = *(const CST_AS unsigned*)COUNTER(mine0);
            PIN();
            if (wid < 4) {
#pragma unroll
                for (int i = 0; i < AGD; ++i) glds16(x0 + i * 16384, agoff, (wid * 8 + i) * 1024);
                if (lane < 4) glds4(theirs0 + H_TMO + 128, 0);
            }
        } else if ((wid >> 1) == 2) {
            const int i = (b >> 3) - (local ? 0 : 4);
            const bool isx = local || (wid == 5 && i < PF_XU);
            constexpr int NWW = 24 - PF_XU;
            const int n0 = local ? wid & 1 : isx ? 0 : wid == 4 ? i : i + 12 - PF_XU, dn = local ? 2 : isx ? 1 : NWW;
            const unsigned vo = isx ? ((local ? i : AGU + i) * 32 + (lane >> 1)) * KD + (lane & 1) * 128 : (lane >> 2) * ND + (lane & 3) * 128;
            warm_l2(isx ? x0 : w0, vo, isx ? 8 : 16, n0, dn, isx ? 16 : 256);
        }
    }

    if (link) {
        constexpr int NP = AGU * 8;
        static_assert(AGD >= 2 && AGD <= 8, "staging ring");
        unsigned done = done0;
        WAIT_LDS();
        BARRIER();
        asm volatile("" : "+s"(done));
        const unsigned call = done + 1;
        char* const theirs = theirs0 + (call & 1) * HALF_BYTES;
        STAMP(1);
        if (wid < 4) {
            int r = 0;
            WAIT_VM(1);
#pragma unroll 1
            for (int i = 0; i < NP; ++i) {
                WAIT_VM(2 * AGD - 2);
                if (i >= AGD && ((i - AGD) & 7) == 7 && lane == 0)
                    __hip_atomic_fetch_add(ctl + 8 + ((i - AGD) >> 3), 1u, __ATOMIC_RELAXED, __HIP_MEMORY_SCOPE_WORKGROUP);
                const u32x4 v = *(const u32x4*)(smem + (wid * 8 + r) * 1024 + lane * 16);
                store_sys(theirs + H_X + i * 16384, agoff, v);
                glds16(x0 + (i + AGD < NP ? i + AGD : 0) * 16384, agoff, (wid * 8 + r) * 1024);
                r = r == AGD - 1 ? 0 : r + 1;
            }
            WAIT_VM(0);
            if (lane == 0) __hip_atomic_fetch_add(ctl + 8 + AGU - 1, 1u, __ATOMIC_RELAXED, __HIP_MEMORY_SCOPE_WORKGROUP);
            STAMP(3);
        }
        advance(ctl, COUNTER(mine0), call, lane);
        FRESH();
        const float scale = SCALE();
        LDS_AS u64* mirror = (LDS_AS u64*)(ctl + (MIR_OFF - CTL_OFF) / 4);
        auto flag_word = [&](int i) {
            const u64 v = __hip_atomic_load(mirror + i * 2, __ATOMIC_ACQUIRE, __HIP_MEMORY_SCOPE_WORKGROUP);
            const unsigned fail = __hip_atomic_load(ctl + 14, __ATOMIC_ACQUIRE, __HIP_MEMORY_SCOPE_WORKGROUP);
            const bool ok = (unsigned)(v >> 32) >= call && ((unsigned)v >> 16) == (call & 0xffffu);
            return (ok ? ((unsigned)v & 0xffffu) | READY : 0u) | fail;
        };
        if (wid == 7) {
            for (int u = 0; u < AGU; ++u) {
                while (__hip_atomic_load(ctl + 8 + u, __ATOMIC_RELAXED, __HIP_MEMORY_SCOPE_WORKGROUP) < 4) __builtin_amdgcn_s_sleep(2);
                if (lane == 0) set_flag(theirs + H_XFLAG + (u * 4 + pan) * 128, call, 0);
            }
        } else if (wid == 6) {
            const char* const flags = mine0 + (call & 1) * HALF_BYTES + H_FLAG;
            const unsigned fo = (((PULL_ORDER >> (lane & 12)) & 15) * 4 + (lane & 3)) * 16;
            while (__hip_atomic_load(ctl + 12, __ATOMIC_RELAXED, __HIP_MEMORY_SCOPE_WORKGROUP) < 8) __builtin_amdgcn_s_sleep(4);
            const u64 t0 = NOW();
            while (true) {
                WAIT_VM(7);
                if (lane < 16) glds16_sys(flags, fo, MIR_OFF);
                __builtin_amdgcn_s_sleep(12);
                if (__all(flag_word(lane & 15) & READY)) break;
                if (give_up(t0, TMO(), call)) {
                    __hip_atomic_store(ctl + 14, READY | FAILED, __ATOMIC_RELEASE, __HIP_MEMORY_SCOPE_WORKGROUP);
                    break;
                }
            }
        }
        constexpr int N0 = 4 * nib(R_CNT, nib(PULL_ORDER, 0)), N1 = N0 + 3 * (NPU - N0 / 4), LAST = (NPU - 1) * 4;
        int tp = 3, ej, ep;
        for (int p = 2; p >= 0; --p) if (tight(w0 + kp->wown + p * 128)) tp = p;
        const int em = min(lane, NPU * 4 - 1);
        if (em < N0) { ej = em >> 2; ep = em & 3; }
        else if (em < N1) { ej = (em - N0) / 3; ep = em - N0 - 3 * ej; ep += ep >= tp; ej += N0 / 4; }
        else { ej = em - N1 + N0 / 4; ep = tp; }
        const int ee = (int)(pull_table() >> (8 * ej)) & 255;
        const unsigned eout = ((AGU + (ee & 15)) * 32 * NS + ep * 128 + pan * 32) * 4;
        char* const out = kp->out;
        LDS_AS unsigned* landed = ctl + 16;
        if (wid < 4) {
            const int en = ((ee & 15) * 4 + ep) * 4 + pan, ef = (ee >> 6) * 4 + ep, eb = ((ee >> 4) & 3) * 4 + pan;
            while (!(flag_word(12 + pan) & READY)) __builtin_amdgcn_s_sleep(1);
            unsigned words = 0, marks = 0;
#pragma unroll 1
            for (int j = 0; j < NPU; ++j) {
                const int m = j * 4 + wid;
                unsigned word = __builtin_amdgcn_readlane(words, m);
                for (int k = 0; !(word & READY); ++k) {
                    if (k == 1) WAIT_VM(2);
                    else if (k == 2) WAIT_VM(1);
                    else if (k == 3) {
                        WAIT_VM(0);
                        if (lane == 0) __hip_atomic_store(landed + wid, marks | j, __ATOMIC_RELEASE, __HIP_MEMORY_SCOPE_WORKGROUP);
                    } else if (k) __builtin_amdgcn_s_sleep(1);
                    const unsigned w = flag_word(ef);
                    words = (w & (READY | FAILED)) | ((w >> eb) & 1);
                    word = __builtin_amdgcn_readlane(words, m);
                }
                marks |= ((word & 1) << 8 | (word & FAILED ? 1u << 16 : 0u)) << j;
                if (word & FAILED) WAIT_VM(0);
                else fetch_subtile<2>(theirs, __builtin_amdgcn_readlane(en, m), m * 4096, word & 1, lane);
                if (lane == 0) __hip_atomic_store(landed + wid, marks | j, __ATOMIC_RELEASE, __HIP_MEMORY_SCOPE_WORKGROUP);
            }
            char* o = out + __builtin_amdgcn_readlane(eout, LAST + wid);
            float sc = (marks >> (15 + NPU)) & 1 ? __builtin_nanf("") : scale;
            asm volatile("" : "+s"(o));
            asm volatile("" : "+v"(sc));
            WAIT_VM(0);
            if (lane == 0) __hip_atomic_store(landed + wid, marks | NPU, __ATOMIC_RELEASE, __HIP_MEMORY_SCOPE_WORKGROUP);
            STAMP(2);
            finish_half<0>(smem, (LAST + wid) * 4096, (marks >> (7 + NPU)) & 1, sc, o, lane);
        } else {
            for (bool more = true; more;) {
                unsigned m = 0;
                if (lane == 0) m = __hip_atomic_fetch_add(ctl + 13, 1u, __ATOMIC_RELAXED, __HIP_MEMORY_SCOPE_WORKGROUP);
                m = __builtin_amdgcn_readfirstlane(m);
                more = m < LAST;
                if (!more) m = LAST + wid - 4;
                char* o = out + __builtin_amdgcn_readlane(eout, m);
                asm volatile("" : "+s"(o));
                unsigned w;
                while (((w = __builtin_amdgcn_readfirstlane(__hip_atomic_load(landed + (m & 3), __ATOMIC_ACQUIRE, __HIP_MEMORY_SCOPE_WORKGROUP))) & 255) <= m >> 2)
                    if (more) __builtin_amdgcn_s_sleep(1);
                w >>= m >> 2;
                const float sc = (w >> 16) & 1 ? __builtin_nanf("") : scale;
                finish_half<1>(smem, m * 4096, (w >> 8) & 1, sc, o, lane);
                if (more) finish_half<0>(smem, m * 4096, (w >> 8) & 1, sc, o, lane);
            }
        }
    } else {
        int cnt = NIB(local ? L_CNT : R_CNT);
        if (RARE(((local ? LPF_MASK : PF_MASK) >> cls) & 1)) {
            if ((wid >> 1) == 2) {
                const int me = (b >> 3) * 2 + (wid & 1);
                static_assert(PF2_XU == 4, "wave 4: the first 64 rows, wave 5: the other 64");
                if (local) warm_l2(w0, (lane >> 2) * ND + (lane & 3) * 128, 16, me, 32, 256);
                else if ((unsigned)((b >> 3) - 4 - PF2_LO) < PF2_N)
                    warm_l2(x0, ((AGU + PF2_FIRST) * 32 + (wid & 1) * 64 + lane) * KD, 7, (b >> 3) - 4 - PF2_LO, PF2_N, 32);
            } else if (wid == 6) {
                const int k = b >> 3;
                if ((TLBW & 4) && lane == 0) glds4(k == 4 ? theirs0 + H_TMO + 128 : k < 8 ? mine0 + H_TMO + 128 : w0 + (size_t)(k - 8) * (2u << 20), 0);
                asm volatile("" :: "s"(CALL()), "v"(SCALE()));
            }
            WAIT_LDS();
            BARRIER();
        } else if (RARE(local)) {
            while (NOW() - tstart < L_START) __builtin_amdgcn_s_sleep(2);
        }
        int task = cnt ? 0 : 1;
        unsigned lastcall = 0;
        const int ntask = !local && NIB(A_CNT) ? 2 : 1;
#pragma unroll 1
        for (; task < ntask; ++task) {
            unsigned bad = 0, first = local ? NIB(L_FIRST) : AGU + NIB(R_FIRST);
            const char *wp1 = nullptr, *xp1 = nullptr;
            if (RARE(task == 1)) {
                FRESH();
                const unsigned call = CALL();
                const char* const mine = kp->mine + (call & 1) * HALF_BYTES;
                wp1 = kp->wblk + kp->wown;
                xp1 = mine + H_X;
                const bool tp = tight(wp1 + pan * 128);
                cnt = NIB(tp ? T_CNT : A_CNT);
                first = NIB(tp ? T_FIRST : A_FIRST);
                if (wid == 1) {
                    if (!tp) while (NOW() - tstart < A_START) __builtin_amdgcn_s_sleep(4);
                    if (lane < 4) ctl[1] = poll_flags(mine + H_XFLAG + ((first + cnt - 1) * 4 + lane) * 128, TMO(), call);
                    WAIT_LDS();
                }
                BARRIER();
                bad = __builtin_amdgcn_readfirstlane(ctl[1]);
                STAMP(4);
            }
            if (wid < 4) {
                FRESH();
                unsigned woff = w_offset(lane, wid) + pan * 128, xoff = x_offset(lane, wid) + first * 32 * KD;
                asm volatile("" : "+v"(woff), "+v"(xoff));
                PIN();
                WAIT_LDS();
                const char *wp = task ? wp1 : w0, *xp = task ? xp1 : x0;
                if (!bad) {
#pragma unroll
                    for (int j = 0; j < 4; ++j) glds16(wp + j * 8 * ND, j >= 2 ? woff ^ 16 : woff, wid * 4096 + j * 1024);
                    glds16_sys(xp, xoff, 16384 + wid * 1024);
                    if (cnt > 1) glds16_sys(xp + 32 * KD, xoff, 16384 + 4096 + wid * 1024);
                    if (cnt > 2) glds16_sys(xp + 64 * KD, xoff, 16384 + 8192 + wid * 1024);
                    if (cnt == 1) load_tile<1, 3>(wp, xp, woff, xoff, wid);
                    else if (cnt == 2) load_tile<2, 3>(wp, xp, woff, xoff, wid);
                    else load_tile<3, 2>(wp, xp, woff, xoff, wid);
                }
            }
            FRESH();
            const bool direct = local || task == 1, push = (R_PUSH >> cls) & 1;
            char *mine = kp->mine, *flags = kp->theirs, *out = kp->out;
            const char *sxp = kp->sx, *swp = kp->sw;
            asm volatile("" : "+s"(mine), "+s"(flags), "+s"(out), "+s"(sxp), "+s"(swp));
            const unsigned done = *(const CST_AS unsigned*)COUNTER(mine);
            const float sx = *(const CST_AS float*)sxp, sw = *(const CST_AS float*)swp;
            const unsigned n0 = ((first - AGU + ((wid ^ 4) >> 2)) * 4 + pan) * 4 + (wid & 3);
            char* far = (push ? flags : mine) + H_LONG + n0 * 2048;
            char* dst = direct ? out + ((first * 32 + (wid & 1) * 16) * NS + pan * 128 + (wid >> 1) * 32) * 4 : (push ? flags : mine) + n0 * SHORT_BYTES;
            flags += H_FLAG;
            unsigned ta = (lane >> 3) * 128 + (((lane & 7) ^ (lane >> 3)) << 4) + (direct ? wid * 2048 : (wid ^ 4) * 4096);
            int mode = direct ? 0 : (wid ^ 4) >= cnt * 4 ? 1 : (wid ^ 4) + 8 >= cnt * 4 ? 2 : 3;
            asm volatile("" : "+s"(far), "+s"(dst), "+s"(flags), "+s"(mode));
            asm volatile("" : "+v"(ta));
            if (bad) {
            } else if (wid < 4) {
                if (!direct && push) while (NOW() - tstart < HOLD0 + HOLD1 * (first - AGU)) __builtin_amdgcn_s_sleep(1);
            } else if (cnt == 1) tile_compute<1>(smem, lane, wid - 4, done, sx, sw);
            else if (cnt == 2) tile_compute<2>(smem, lane, wid - 4, done, sx, sw);
            else tile_compute<3>(smem, lane, wid - 4, done, sx, sw);
            const unsigned call = done + 1, half = (call & 1) * HALF_BYTES;
            lastcall = call;
            if (mode) { dst += half; far += half; flags += half; }
            asm volatile("" : "+s"(far), "+s"(dst), "+s"(flags));
            BARRIER();
            if (mode == 0) {
                const float scale = bad ? __builtin_nanf("") : sx * sw;
#pragma unroll 1
                for (int u = 0; u < cnt; ++u) {
                    const u32x4 t0 = *(const u32x4*)(smem + ta + u * 16384), t1 = *(const u32x4*)(smem + ta + u * 16384 + 1024);
                    write_rows(t0, scale, dst + u * (32 * NS * 4), lane);
                    write_rows(t1, scale, dst + u * (32 * NS * 4) + 8 * NS * 4, lane);
                }
            } else {
                unsigned marks = 0;
                if (mode > 1)
                    marks = send_subtile<0>(smem + ta, dst, far, lane, push && cnt > 1,
                                            cls == GO_CLS && (wid ^ 4) == GO_ITEM ? flags + pan * 16 : nullptr, call);
                if (mode > 2) marks |= send_subtile<1>(smem + ta + 32768, dst + 32 * SHORT_BYTES, far + 32 * 2048, lane, push, nullptr, call) << 8;
                if (marks && lane == 0) __hip_atomic_fetch_or(ctl, marks << (wid ^ 4), __ATOMIC_RELAXED, __HIP_MEMORY_SCOPE_WORKGROUP);
                flags += (b >> 3) * 16;
                asm volatile("" : "+s"(flags));
                asm volatile("s_waitcnt vmcnt(0) lgkmcnt(0)" ::: "memory");
                BARRIER();
                STAMP(3);
                if (wid == 7 && lane == 0) set_flag(flags, call, ctl[0]);
            }
            if (task + 1 == ntask) advance(ctl, COUNTER(mine), call, lane);
        }
        FRESH();
        if (!lastcall) advance(ctl, COUNTER(kp->mine), CALL(), lane);
        if (!local) {
            const char* const mine = kp->mine + (lastcall & 1) * HALF_BYTES;
            char* const out = kp->out + pan * 128 * 4;
            const float scale = SCALE();
#pragma unroll 1
            for (int c = 1; c < 8; ++c) {
                if (((M_CLS >> (4 * c)) & 15) != cls) continue;
                BARRIER();
                if (wid == 1) {
                    if (lane == 0) ctl[1] = poll_flags(mine + H_FLAG + (c * 4 + pan) * 16, TMO(), lastcall);
                    WAIT_LDS();
                }
                BARRIER();
                const unsigned word = __builtin_amdgcn_readfirstlane(ctl[1]);
                const float sc = word & FAILED ? __builtin_nanf("") : scale;
                for (int i = wid; i < ((R_CNT >> (4 * c)) & 15) * 4; i += 8) {
                    const int u = ((R_FIRST >> (4 * c)) & 15) + (i >> 2);
                    if (!(word & FAILED)) fetch_subtile<63>(mine, (u * 4 + pan) * 4 + (i & 3), i * 4096, (word >> i) & 1, lane);
                    WAIT_VM(0);
                    char* const o = out + ((AGU + u) * 32 * NS + (i & 3) * 32) * 4;
                    finish_half<0>(smem, i * 4096, (word >> i) & 1, sc, o, lane);
                    finish_half<1>(smem, i * 4096, (word >> i) & 1, sc, o, lane);
                }
            }
        }
    }
}
#undef cls
#undef pan
#undef local

extern "C" void kernel_launch(int rank, int nranks, ncclComm_t comm,
                              hipStream_t stream,
                              void* const* d_in, const int* in_sizes, int n_in,
                              void* d_out, int out_size,
                              const int* in_dtypes, int out_dtype,
                              void* const* d_out_all,
                              void* const* d_ws_all, size_t ws_bytes) {
    if (nranks != NR || (unsigned)rank >= (unsigned)NR || ws_bytes < WS_END || n_in != 4 || in_sizes[0] != MS * KD
        || in_sizes[1] != KD * ND || out_size != NR * MS * NS) return;
    static bool init[NR] = {};
    if (!init[rank]) {
        HIP_CHECK(hipFuncSetAttribute((const void*)fused_kernel, hipFuncAttributeMaxDynamicSharedMemorySize, LDS_BYTES));
        init[rank] = true;
    }
    static_assert(LOCAL_GRP == 3, "three peers in the own quad");
    int peer[NR], lo = 0, hi = 4;
    const unsigned quad = ((0x4d >> rank) & 1) ? 0x4du : 0xb2u;
    for (int i = 1; i < NR; ++i) {
        const int q = (rank + i) & 7;
        if ((quad >> q) & 1) peer[lo++] = q; else peer[hi++] = q;
    }
    peer[LOCAL_GRP] = rank;
    Params p;
    for (int g = 0; g < NR; ++g) {
        p.g[g].x = (const char*)d_in[0];
        p.g[g].wblk = (const char*)d_in[1] + peer[g] * NS;
        p.g[g].mine = (char*)d_ws_all[rank] + (size_t)peer[g] * BLK_BYTES;
        p.g[g].theirs = (char*)d_ws_all[peer[g]] + (size_t)rank * BLK_BYTES;
        p.g[g].out = (char*)d_out + (size_t)peer[g] * MS * NS * 4;
        p.g[g].sx = (const char*)d_in[2];
        p.g[g].sw = (const char*)d_in[3];
        p.g[g].wown = (rank - peer[g]) * NS;
        p.g[g].peer = peer[g];
    }
    hipLaunchKernelGGL(fused_kernel, dim3(GRID), dim3(BLOCK), LDS_BYTES, stream, p);
    HIP_CHECK(hipGetLastError());
}
```
